```python
import numpy as np

kernel_name = "gpu_gemm_rs_m4096_k4096_n32768_f32_gelu"


ABSMAX_RTOL = 0.001

COMPUTE_DTYPE = "float32"
EPILOGUE = "gelu"
IN_QUANT = None
OUT_QUANT = None
IN_DTYPES = {"x": "float32", "w_mat": "float32"}
WORLD_SIZE = 8

def setup_inputs():

    x = np.random.randn(4096, 4096).astype(np.float32)
    w_mat = (np.random.randn(4096, 32768) / np.sqrt(4096)).astype(np.float32)
    return {"x": x, "w_mat": w_mat}

def _epilogue(y):
    c = 0.7978845608028654
    return (0.5 * y * (1.0 + np.tanh(c * (y + 0.044715 * y ** 3)))).astype(np.float32)

def reference(x, w_mat):
    y = (x @ w_mat).astype(np.float32)
    return _epilogue(y)
```

```cpp
#include <hip/hip_runtime.h>
#include <rccl/rccl.h>
#include <cstdio>

#define HIP_CHECK(expr) do { hipError_t _e = (expr); if (_e != hipSuccess) { \
    fprintf(stderr, "HIP error %s at %s:%d\n", hipGetErrorString(_e), __FILE__, __LINE__); \
    return; } } while (0)

#define SF 2
#define EPI_LDS 1
#define EW1 2
#define EF1 2
#define EF2 2
#define POLL_SLEEP 16
#define GLDS_AUX 17

namespace {

typedef _Float16 h8 __attribute__((ext_vector_type(8)));
typedef _Float16 h2 __attribute__((ext_vector_type(2)));
typedef float f4 __attribute__((ext_vector_type(4)));
typedef unsigned u4 __attribute__((ext_vector_type(4)));
typedef unsigned long long u64;
typedef __attribute__((address_space(1))) unsigned gu32;
typedef __attribute__((address_space(1))) f4 gf4;
#define LOADF4(p) (*(p))
#define LAS __attribute__((address_space(3)))

constexpr int NR = 8;
constexpr int MG = 4096, KG = 4096, NG = 32768;
constexpr int KS = KG / NR;
constexpr int NS = NG / NR;
constexpr int MS = MG / NR;
constexpr int NWG = 256, NTHR = 512;
constexpr int NSEND = 56, NGEM = NWG - NSEND;
constexpr int TILE_B = 256 * 64 * 2;
constexpr int HALF_B = TILE_B / 2;
constexpr int NKT = KG / 64;
constexpr int NIMG = 256;
constexpr int NB1 = 13;
constexpr int PER1 = 16 + NB1, NIMG1 = 8 * PER1;
constexpr unsigned OP_B = 16u * NKT * TILE_B;
constexpr unsigned CTL_B = 1u << 20;
constexpr unsigned SLOT_B = 2u * OP_B;
constexpr size_t WS_NEED = (size_t)CTL_B + 2 * (size_t)SLOT_B;
constexpr int W_TMO = 0;
constexpr int W_FLAG = 32;
constexpr int W_DONE = W_FLAG + 256 * 32;
constexpr int W_TICK = W_DONE + 8 * 32;
constexpr int W_CALL = W_TICK + 8 * 32;
constexpr int W_STAMP = 131072;
static_assert(W_CALL + NWG * 32 <= W_STAMP, "control layout");
constexpr u64 DEADLINE = 40000000ull;

constexpr int LDS_CTL = 8 * HALF_B;
constexpr int LDS_BYTES = LDS_CTL + 256;
constexpr int L_CNT = 8, L_BAD = 40;

struct Params {
    const float* x; const float* w;
    float* out[NR]; char* ws[NR];
    int rank; int pad;
};

#define WAIT_V(n) asm volatile("s_waitcnt vmcnt(" #n ")" ::: "memory")
#define WAIT_L(n) asm volatile("s_waitcnt lgkmcnt(" #n ")" ::: "memory")
#define BAR __builtin_amdgcn_s_barrier()
#define SCHED __builtin_amdgcn_sched_barrier(0)
#define CFENCE asm volatile("" ::: "memory")
template <int N> __device__ __forceinline__ void wait_v() { asm volatile("s_waitcnt vmcnt(%0)" :: "i"(N) : "memory"); }

__device__ __forceinline__ void st4(unsigned voff, u4 v, const void* sbase) {
    asm volatile("global_store_dwordx4 %0, %1, %2 sc0 sc1\n\ts_nop 1" :: "v"(voff), "v"(v), "s"(sbase) : "memory");
}
__device__ __forceinline__ unsigned pk(float a, float b) {
    h2 r; r.x = (_Float16)a; r.y = (_Float16)b; return __builtin_bit_cast(unsigned, r);
}
__device__ __forceinline__ float gelu(float y) {
    const float k1 = -2.0f * 0.7978845608028654f * 1.4426950408889634f, k2 = k1 * 0.044715f;
    const float e = __builtin_amdgcn_exp2f(y * (k1 + k2 * y * y));
    return y * __builtin_amdgcn_rcpf(1.0f + e);
}

#define STAMP(i) do {} while (0)
#define STAMPW(i) do {} while (0)

__device__ __forceinline__ bool wave_wait(gu32* pa, gu32* pb, u64 need, u64& ra, u64& rb, unsigned target, gu32* tmo, unsigned call, int lane) {
    const u64 t0 = __builtin_amdgcn_s_memrealtime();
    for (;;) {
        const unsigned va = __hip_atomic_load(pa, __ATOMIC_RELAXED, __HIP_MEMORY_SCOPE_SYSTEM);
        const unsigned vb = __hip_atomic_load(pb, __ATOMIC_RELAXED, __HIP_MEMORY_SCOPE_SYSTEM);
        const unsigned vt = __hip_atomic_load(tmo, __ATOMIC_RELAXED, __HIP_MEMORY_SCOPE_AGENT);
        if ((unsigned)__builtin_amdgcn_readfirstlane(vt) == call) return true;
        ra = __ballot((int)(va - target) >= 0); rb = __ballot((int)(vb - target) >= 0);
        if ((ra & rb & need) == need) return false;
        if (__builtin_amdgcn_s_memrealtime() - t0 > DEADLINE) {
            if (lane == 0) __hip_atomic_store(tmo, call, __ATOMIC_RELAXED, __HIP_MEMORY_SCOPE_AGENT);
            return true;
        }
        __builtin_amdgcn_s_sleep(POLL_SLEEP);
    }
}

__device__ __forceinline__ void image_of(int i, bool& isA, int& panel, int& j) {
    if (i < NIMG1) { j = i / PER1; const int g = i - j * PER1; isA = g < 16; panel = isA ? g : g - 16; }
    else { const int q = i - NIMG1; j = q / (16 - NB1); isA = false; panel = NB1 + q - j * (16 - NB1); }
}
__device__ __forceinline__ void job_loads(f4 (&L)[8], const char* x, const char* w, int i, int dest, int e, unsigned vxe, unsigned vl) {
    bool isA; int panel, j; image_of(i, isA, panel, j);
    const char* b = isA ? x + ((size_t)(panel * 256 + e * 32) * KS + j * 64) * 4 : w + ((size_t)(j * 64 + e * 8) * NG + dest * NS + panel * 256) * 4;
    const unsigned ve = isA ? vxe : vl, vo = isA ? (vxe ^ 128u) : vl;
    const unsigned s2 = isA ? 8 * KS * 4 : 2 * NG * 4, s1 = isA ? 16 : NG * 4;
#pragma unroll
    for (int r = 0; r < 8; ++r) L[r] = LOADF4((const gf4*)(b + ((r >> 1) * s2 + (r & 1) * s1) + ((r & 2) ? vo : ve)));
}
__device__ __forceinline__ void job_pack(const f4 (&L)[8], u4 (&Q)[4], bool isA) {
    if (isA) {
#pragma unroll
        for (int u = 0; u < 4; ++u) { Q[u].x = pk(L[2 * u].x, L[2 * u].y); Q[u].y = pk(L[2 * u].z, L[2 * u].w); Q[u].z = pk(L[2 * u + 1].x, L[2 * u + 1].y); Q[u].w = pk(L[2 * u + 1].z, L[2 * u + 1].w); }
    } else {
#pragma unroll
        for (int t = 0; t < 4; ++t) { Q[t].x = pk(L[0][t], L[1][t]); Q[t].y = pk(L[2][t], L[3][t]); Q[t].z = pk(L[4][t], L[5][t]); Q[t].w = pk(L[6][t], L[7][t]); }
    }
}

__global__ __launch_bounds__(NTHR, 2) void gemm_rs_kernel(const Params P) {
    __shared__ __attribute__((aligned(16))) unsigned char smem[LDS_BYTES];
    LAS unsigned char* lds = (LAS unsigned char*)smem;
    LAS unsigned* lctl = (LAS unsigned*)(lds + LDS_CTL);
    const int tid = threadIdx.x, lane = tid & 63, wid = __builtin_amdgcn_readfirstlane(tid >> 6);
    const int bid = blockIdx.x, rank = P.rank;
    gu32* ctl = (gu32*)P.ws[rank];
    STAMP(0);

    const bool sender = bid < NSEND;
    const int sq = bid >> 3, sdest = sender ? (rank + 1 + sq) & 7 : rank;
    const char* xb = (const char*)P.x; const char* wb = (const char*)P.w;
    const unsigned vl = (unsigned)lane * 16u;
    const unsigned vxe = (unsigned)((lane >> 3) * (KS * 4) + (((lane & 7) ^ (lane >> 4)) * 32));
    const int ifirst = sender ? (bid & 7) : bid - NSEND;
    f4 L0[8], L1[8];
    job_loads(L0, xb, wb, ifirst, sdest, wid, vxe, vl);

    gu32* callp = ctl + W_CALL + bid * 32;
    const unsigned call = __builtin_amdgcn_readfirstlane(__hip_atomic_load(callp, __ATOMIC_RELAXED, __HIP_MEMORY_SCOPE_AGENT)) + 1u;
    if (tid < 64) lctl[tid] = tid < 3 ? (unsigned)(ifirst + 8 * tid) : 0u;
    const unsigned slot = CTL_B + (call & 1u) * SLOT_B;
    gu32* tmo = ctl + W_TMO;
    char* dws = P.ws[(unsigned)sdest & 7u];
#define FLAG_OF(isA, panel, j) ((gu32*)dws + (W_FLAG + ((((isA) ? 0 : 16) + (panel)) * 8 + rank) * 32 + (j)))
    if (!sender) {
        WAIT_L(0); BAR; CFENCE;
        if (tid == 0) __hip_atomic_store(callp, call, __ATOMIC_RELAXED, __HIP_MEMORY_SCOPE_AGENT);
        for (int i = ifirst; i < NIMG; i += NGEM) {
            bool isA; int panel, j; image_of(i, isA, panel, j);
            if (i != ifirst) job_loads(L0, xb, wb, i, rank, wid, vxe, vl);
            u4 Q[4]; job_pack(L0, Q, isA);
            const char* db = dws + (slot + (isA ? 0u : OP_B) + (unsigned)((panel * NKT + j) * TILE_B + wid * (isA ? 4096 : 2048)));
            const int hi = isA ? 2048 : HALF_B;
            st4(vl, Q[0], db); st4(vl, Q[1], db + 1024); st4(vl, Q[2], db + hi); st4(vl, Q[3], db + hi + 1024);
            WAIT_V(0); BAR; CFENCE;
            if (tid == 0) __hip_atomic_store(FLAG_OF(isA, panel, j), call, __ATOMIC_RELAXED, __HIP_MEMORY_SCOPE_SYSTEM);
        }
    } else {
        {
            bool isA; int panel, j; image_of(ifirst, isA, panel, j);
            u4 Q[4]; job_pack(L0, Q, isA);
            const int o = wid * (isA ? 4096 : 2048) + lane * 16, hi = isA ? 2048 : HALF_B;
            *(LAS u4*)(lds + o) = Q[0]; *(LAS u4*)(lds + o + 1024) = Q[1]; *(LAS u4*)(lds + o + hi) = Q[2]; *(LAS u4*)(lds + o + hi + 1024) = Q[3];
        }
        if (wid >= 4) { job_loads(L0, xb, wb, ifirst + 8, sdest, 2 * (wid - 4), vxe, vl); job_loads(L1, xb, wb, ifirst + 8, sdest, 2 * (wid - 4) + 1, vxe, vl); }
        WAIT_L(0); BAR; CFENCE;
        if (tid == 0) __hip_atomic_store(callp, call, __ATOMIC_RELAXED, __HIP_MEMORY_SCOPE_AGENT);
        const int pbase = 8 + ((rank - sdest - 1) & 7);
        if (wid < 4) {
            gu32* prev_flag = nullptr;
            for (int n = 0;; ++n) {
                const int i = __builtin_amdgcn_readfirstlane(lctl[n & 3]);
                if (i >= NIMG) break;
                u4 Q[8];
#pragma unroll
                for (int r = 0; r < 8; ++r) Q[r] = *(const LAS u4*)(lds + (n & 1) * TILE_B + wid * 8192 + r * 1024 + lane * 16);
                bool isA; int panel, j; image_of(i, isA, panel, j);
                const char* db = dws + (slot + (isA ? 0u : OP_B) + (unsigned)((panel * NKT + pbase + j * 7) * TILE_B + wid * 8192));
#pragma unroll
                for (int r = 0; r < 8; ++r) {
                    wait_v<SF - 1>();
                    if (r == SF - 1 && n > 0) {
                        unsigned old = 0;
                        if (lane == 0) old = __hip_atomic_fetch_add(lctl + L_CNT + ((n - 1) & 3), 1u, __ATOMIC_RELAXED, __HIP_MEMORY_SCOPE_WORKGROUP);
                        if ((__builtin_amdgcn_readfirstlane(old) & 3u) == 3u && lane == 0) __hip_atomic_store(prev_flag, call, __ATOMIC_RELAXED, __HIP_MEMORY_SCOPE_SYSTEM);
                    }
                    st4(vl, Q[r], db + r * 1024);
                }
                prev_flag = FLAG_OF(isA, panel, j);
                if (wid == 0 && (n & 3) == 3) STAMPW(21 + ((n >> 2) & 7));
                BAR; CFENCE;
            }
            WAIT_V(0); BAR; CFENCE;
            if (tid == 0) __hip_atomic_store(prev_flag, call, __ATOMIC_RELAXED, __HIP_MEMORY_SCOPE_SYSTEM);
        } else {
            gu32* tick = ctl + W_TICK + sq * 32;
            const unsigned tbase = (call - 1u) * (NIMG - 24 + 8) - 24u;
            bool more = true;
            for (int n = 0;; ++n) {
                const int i = __builtin_amdgcn_readfirstlane(lctl[n & 3]);
                if (i >= NIMG) break;
                const int i1 = __builtin_amdgcn_readfirstlane(lctl[(n + 1) & 3]), i2 = __builtin_amdgcn_readfirstlane(lctl[(n + 2) & 3]);
                unsigned t = NIMG;
                if (wid == 4 && more && lane == 0) t = __hip_atomic_fetch_add(tick, 1u, __ATOMIC_RELAXED, __HIP_MEMORY_SCOPE_AGENT) - tbase;
                if (i1 < NIMG) {
                    bool isA; int panel, j; image_of(i1, isA, panel, j);
                    u4 Q0[4], Q1[4]; job_pack(L0, Q0, isA); job_pack(L1, Q1, isA);
                    const int js = isA ? 4096 : 2048, o = ((n + 1) & 1) * TILE_B + 2 * (wid - 4) * js + lane * 16, hi = isA ? 2048 : HALF_B;
                    *(LAS u4*)(lds + o) = Q0[0]; *(LAS u4*)(lds + o + 1024) = Q0[1]; *(LAS u4*)(lds + o + hi) = Q0[2]; *(LAS u4*)(lds + o + hi + 1024) = Q0[3];
                    *(LAS u4*)(lds + o + js) = Q1[0]; *(LAS u4*)(lds + o + js + 1024) = Q1[1]; *(LAS u4*)(lds + o + js + hi) = Q1[2]; *(LAS u4*)(lds + o + js + hi + 1024) = Q1[3];
                }
                CFENCE;
                const int il = i2 < NIMG ? i2 : i;
                job_loads(L0, xb, wb, il, sdest, 2 * (wid - 4), vxe, vl); job_loads(L1, xb, wb, il, sdest, 2 * (wid - 4) + 1, vxe, vl);
                if (wid == 4) {
                    t = __builtin_amdgcn_readfirstlane(t);
                    more = t < (unsigned)NIMG;
                    if (lane == 0) lctl[(n + 3) & 3] = more ? t : (unsigned)NIMG;
                }
                WAIT_L(0); BAR; CFENCE;
            }
            WAIT_V(0); BAR; CFENCE;
        }
    }
    STAMP(16);
    int pm, pn;
    {
        const int g = sender ? bid : bid - NSEND, xcd = g & 7, idx = g >> 3;
        if (sender) { const int d = (xcd + idx + 1) & 7; pm = 2 * d + (idx >= 3); pn = idx >= 3 ? 9 + idx : 13 + idx; }
        else if (idx < 24) { pm = 2 * (idx & 7) + (xcd & 1); pn = 3 * (xcd >> 1) + (idx >> 3); }
        else { pm = 2 * xcd; pn = 12; }
    }

    const int wr = wid >> 2, wc = wid & 3, fr = lane & 15, fq = lane >> 4;
    gu32* fa = ctl + W_FLAG + (pm * 8 + (lane & 7)) * 32 + (lane >> 3);
    gu32* fb = ctl + W_FLAG + ((16 + pn) * 8 + (lane & 7)) * 32 + (lane >> 3);
    const u64 own = 0x0101010101010101ull << rank;
    u64 ra = 0, rb = 0;
    bool bad = false;
#define STEP_WAIT(need, last) do { if (!bad && ((last) || (ra & rb & (need)) != (need))) bad = wave_wait(fa, fb, (need), ra, rb, call, tmo, call, lane); } while (0)
    if (wid == 0) { STEP_WAIT(own, false); WAIT_V(0); STAMPW(1); }
    CFENCE; BAR; CFENCE;

    const char* cA = P.ws[rank] + (slot + (unsigned)(pm * NKT) * TILE_B);
    const char* cB = P.ws[rank] + (slot + OP_B + (unsigned)(pn * NKT) * TILE_B);
    const unsigned vst = (unsigned)tid * 16u;
    const unsigned ldsw = (unsigned)wid * 1024u;
    const int a0 = (wr * 64 + fr) * 128 + ((fq ^ (fr >> 1)) * 16), a1 = a0 ^ 64;
    const int b0 = fq * 2048 + (wc * 16 + fr) * 16;
#define SA(b, h) (((b) * 2 + (h)) * HALF_B)
#define SB(b, h) ((4 + (b) * 2 + (h)) * HALF_B)
#define STAGE(bufoff, gbase) do { _Pragma("unroll") for (int _i = 0; _i < 2; ++_i) \
        __builtin_amdgcn_global_load_lds((const unsigned*)((gbase) + vst + _i * 8192), (LAS unsigned*)(lds + (bufoff) + ldsw + _i * 8192), 16, 0, GLDS_AUX); } while (0)
#define LDA(dst, b, h) do { _Pragma("unroll") for (int m = 0; m < 4; ++m) { dst[m][0] = *(const LAS h8*)(lds + SA(b, h) + a0 + m * 2048); dst[m][1] = *(const LAS h8*)(lds + SA(b, h) + a1 + m * 2048); } } while (0)
#define LDB(dst, b, h) do { _Pragma("unroll") for (int n = 0; n < 2; ++n) _Pragma("unroll") for (int k = 0; k < 2; ++k) dst[n][k] = *(const LAS h8*)(lds + SB(b, h) + b0 + k * 8192 + n * 1024); } while (0)
#define MMA(ai, bj, At, Bt) do { __builtin_amdgcn_s_setprio(1); _Pragma("unroll") for (int m = 0; m < 4; ++m) _Pragma("unroll") for (int n = 0; n < 2; ++n) _Pragma("unroll") for (int k = 0; k < 2; ++k) \
        acc[ai][bj][m][n] = __builtin_amdgcn_mfma_f32_16x16x32_f16(At[m][k], Bt[n][k], acc[ai][bj][m][n], 0, 0, 0); __builtin_amdgcn_s_setprio(0); } while (0)
    f4 acc[2][2][4][2];
#pragma unroll
    for (int a = 0; a < 2; ++a)
#pragma unroll
        for (int b = 0; b < 2; ++b)
#pragma unroll
            for (int m = 0; m < 4; ++m)
#pragma unroll
                for (int n = 0; n < 2; ++n) acc[a][b][m][n] = (f4){0.f, 0.f, 0.f, 0.f};
    h8 At[4][2], B0[2][2], B1[2][2];
    STAGE(SB(0, 0), cB); STAGE(SA(0, 0), cA); STAGE(SB(0, 1), cB + HALF_B); STAGE(SA(0, 1), cA + HALF_B);
    if (wr == 1) BAR;
    WAIT_V(4); BAR;
    STAGE(SB(1, 0), cB + TILE_B); STAGE(SA(1, 0), cA + TILE_B); STAGE(SB(1, 1), cB + TILE_B + HALF_B);
    WAIT_V(6); BAR;
    int jn = 0, bnd = 8;
    for (int t = 0; t < NKT; t += 2) {
        const int t2 = (t + 2) & (NKT - 1);
        const char* a1p = cA + (size_t)(t + 1) * TILE_B;
        const char* a2 = cA + (size_t)t2 * TILE_B; const char* b2 = cB + (size_t)t2 * TILE_B;
        const char* a3 = a2 + TILE_B; const char* b3 = b2 + TILE_B;
        if (t + 3 >= bnd && jn < 8) {
            if (wid == 0) {
                STEP_WAIT((0xFFull << (8 * jn)) & ~own, jn == 7);
                if (jn == 7 && lane == 0) lctl[L_BAD] = bad ? 1u : 0u;
                STAMPW(2 + jn);
            }
            CFENCE; BAR; CFENCE;
            ++jn; bnd += 7;
        }
        LDB(B0, 0, 0); SCHED; LDA(At, 0, 0); STAGE(SA(1, 1), a1p + HALF_B);
        WAIT_L(8); BAR; WAIT_L(0); MMA(0, 0, At, B0); BAR; SCHED;
        LDB(B1, 0, 1); STAGE(SB(0, 0), b2);
        BAR; WAIT_L(0); MMA(0, 1, At, B1); BAR;
        LDA(At, 0, 1); STAGE(SA(0, 0), a2);
        BAR; WAIT_L(0); MMA(1, 0, At, B0); BAR; SCHED;
        STAGE(SB(0, 1), b2 + HALF_B);
        WAIT_V(6); BAR; MMA(1, 1, At, B1); BAR;
        LDB(B0, 1, 0); SCHED; LDA(At, 1, 0); STAGE(SA(0, 1), a2 + HALF_B);
        WAIT_L(8); BAR; WAIT_L(0); MMA(0, 0, At, B0); BAR; SCHED;
        LDB(B1, 1, 1); STAGE(SB(1, 0), b3);
        BAR; WAIT_L(0); MMA(0, 1, At, B1); BAR;
        LDA(At, 1, 1); STAGE(SA(1, 0), a3);
        BAR; WAIT_L(0); MMA(1, 0, At, B0); BAR; SCHED;
        STAGE(SB(1, 1), b3 + HALF_B);
        WAIT_V(6); BAR; MMA(1, 1, At, B1); BAR;
    }
    WAIT_V(0);
    if (wr == 0) BAR;
    CFENCE; BAR; CFENCE;
    STAMP(17);

    const unsigned dest = ((unsigned)pm >> 1) & 7u;
    {
        int tz = threadIdx.x; asm volatile("" : "+v"(tz));
        const int l2 = tz & 63;
        const bool poison = lctl[L_BAD] != 0u;
        const float qnan = __builtin_nanf("");
#define GV(a) gelu(a)
        const bool all8 = sender || dest == (unsigned)rank;
        const char* ob = (const char*)P.out[dest] + ((size_t)((pm & 1) * 256) * NG + rank * NS + pn * 256) * 4;
        const int wrow = (wr * 64 + (l2 >> 4) * 4) * 1024 + wc * 256 + (l2 & 15) * 16;
        const int nw = all8 ? 8 : EW1;
#pragma unroll
        for (int ai = 0; ai < 2; ++ai) {
#pragma unroll
            for (int m = 0; m < 4; ++m)
#pragma unroll
                for (int r = 0; r < 4; ++r) {
                    f4 v = {GV(acc[ai][0][m][0][r]), GV(acc[ai][0][m][1][r]), GV(acc[ai][1][m][0][r]), GV(acc[ai][1][m][1][r])};
                    if (poison) v = (f4){qnan, qnan, qnan, qnan};
                    acc[ai][0][m][0][r] = v.x; acc[ai][0][m][1][r] = v.y; acc[ai][1][m][0][r] = v.z; acc[ai][1][m][1][r] = v.w;
                }
#pragma unroll
            for (int m = 0; m < 4; ++m) asm volatile("" : "+v"(acc[ai][0][m][0]), "+v"(acc[ai][0][m][1]), "+v"(acc[ai][1][m][0]), "+v"(acc[ai][1][m][1]));
            if (ai == 1) { WAIT_L(0); BAR; CFENCE; }
#pragma unroll
            for (int m = 0; m < 4; ++m)
#pragma unroll
                for (int r = 0; r < 4; ++r)
                    *(LAS f4*)(lds + wrow + (m * 16 + r) * 1024) = (f4){acc[ai][0][m][0][r], acc[ai][0][m][1][r], acc[ai][1][m][0][r], acc[ai][1][m][1][r]};
            WAIT_L(0); BAR; CFENCE;
            if (wid < nw) {
                u4 v = *(const LAS u4*)(lds + wid * 1024 + l2 * 16);
                for (int row = wid; row < 128; row += nw) {
                    const int nx = row + nw < 128 ? row + nw : row;
                    const u4 vn = *(const LAS u4*)(lds + nx * 1024 + l2 * 16);
                    if (all8) wait_v<EF2 - 1>(); else wait_v<EF1 - 1>();
                    st4((unsigned)l2 * 16u, v, ob + (size_t)(ai * 128 + row) * (NG * 4));
                    v = vn;
                }
            }
        }
    }
    STAMP(18);
    WAIT_V(0);
    BAR; CFENCE;
    if (tid == 0 && dest != (unsigned)rank)
        __hip_atomic_fetch_add((gu32*)P.ws[dest] + W_DONE + rank * 32, 1u, __ATOMIC_RELAXED, __HIP_MEMORY_SCOPE_SYSTEM);
    STAMP(19);
    if (bid == 0 && wid == 0) {
        gu32* pd = ctl + W_DONE + (lane & 7) * 32;
        const bool late = wave_wait(pd, pd, 0xFFull & ~(1ull << rank), ra, rb, call * 32u, tmo, call, lane);
        if (late && lane == 0) P.out[rank][rank * NS] = __builtin_nanf("");
        STAMPW(20);
    }
}

}

extern "C" void kernel_launch(int rank, int nranks, ncclComm_t comm, hipStream_t stream,
                              void* const* d_in, const int* in_sizes, int n_in,
                              void* d_out, int out_size, const int* in_dtypes, int out_dtype,
                              void* const* d_out_all, void* const* d_ws_all, size_t ws_bytes) {
    (void)comm; (void)in_dtypes; (void)out_dtype; (void)d_out;
    if (nranks != NR || (unsigned)rank >= (unsigned)NR || n_in < 2 || ws_bytes < WS_NEED || in_sizes[0] != MG * KS || in_sizes[1] != KS * NG || out_size != MS * NG) {
        fprintf(stderr, "kernel_launch: unexpected problem shape\n");
        return;
    }
    Params P;
    P.x = (const float*)d_in[0]; P.w = (const float*)d_in[1];
    for (int p = 0; p < NR; ++p) { P.out[p] = (float*)d_out_all[p]; P.ws[p] = (char*)d_ws_all[p]; }
    P.rank = rank; P.pad = 0;
    hipLaunchKernelGGL(gemm_rs_kernel, dim3(NWG), dim3(NTHR), 0, stream, P);
    HIP_CHECK(hipGetLastError());
}
```

```cpp
#include <hip/hip_runtime.h>
#include <rccl/rccl.h>
#include <cstdio>

#define HIP_CHECK(expr) do { hipError_t _e = (expr); if (_e != hipSuccess) { \
    fprintf(stderr, "HIP error %s at %s:%d\n", hipGetErrorString(_e), __FILE__, __LINE__); \
    return; } } while (0)

#define SF 2
#define ACT1 2
#define ACT2 4
#define EF 2
#define POLL_SLEEP 16
#define GLDS_AUX 17

namespace {

typedef _Float16 h8 __attribute__((ext_vector_type(8)));
typedef _Float16 h2 __attribute__((ext_vector_type(2)));
typedef float f4 __attribute__((ext_vector_type(4)));
typedef unsigned u4 __attribute__((ext_vector_type(4)));
typedef unsigned long long u64;
typedef __attribute__((address_space(1))) unsigned gu32;
typedef __attribute__((address_space(1))) f4 gf4;
#define LOADF4(p) (*(p))
#define LAS __attribute__((address_space(3)))

constexpr int NR = 8;
constexpr int MG = 4096, KG = 4096, NG = 32768;
constexpr int KS = KG / NR;
constexpr int NS = NG / NR;
constexpr int MS = MG / NR;
constexpr int NWG = 256, NTHR = 512;
constexpr int NSEND = 56, NGEM = NWG - NSEND;
constexpr int TILE_B = 256 * 64 * 2;
constexpr int HALF_B = TILE_B / 2;
constexpr int NKT = KG / 64;
constexpr int NIMG = 256;
constexpr int NB1 = 12;
constexpr int PER1 = 16 + NB1, NIMG1 = 8 * PER1;
constexpr unsigned OP_B = 16u * NKT * TILE_B;
constexpr unsigned CTL_B = 1u << 20;
constexpr unsigned SLOT_B = 2u * OP_B;
constexpr unsigned UNIT_B = 7 * 1024;
constexpr unsigned OUTZ = CTL_B + 2u * SLOT_B;
constexpr size_t WS_NEED = (size_t)OUTZ + (size_t)NR * 32 * 32 * UNIT_B;
constexpr int W_TMO = 0;
constexpr int W_FLAG = 32;
constexpr int W_UNIT = W_FLAG + 256 * 32;
constexpr int W_DQ = W_UNIT + 256 * 32;
constexpr int W_TICK = W_DQ + 8 * 32;
constexpr int W_CALL = W_TICK + 8 * 32;
constexpr int W_STAMP = 131072;
static_assert(W_CALL + NWG * 32 <= W_STAMP, "control layout");
constexpr u64 DEADLINE = 40000000ull;

constexpr int LDS_CTL = 8 * HALF_B;
constexpr int LDS_BYTES = LDS_CTL + 256;
constexpr int L_CNT = 8, L_BAD = 40, L_TURN = 41;

struct Params {
    const float* x; const float* w;
    float* out[NR]; char* ws[NR];
    int rank; int pad;
};

#define WAIT_V(n) asm volatile("s_waitcnt vmcnt(" #n ")" ::: "memory")
#define WAIT_L(n) asm volatile("s_waitcnt lgkmcnt(" #n ")" ::: "memory")
#define BAR __builtin_amdgcn_s_barrier()
#define SCHED __builtin_amdgcn_sched_barrier(0)
#define CFENCE asm volatile("" ::: "memory")
template <int N> __device__ __forceinline__ void wait_v() { asm volatile("s_waitcnt vmcnt(%0)" :: "i"(N) : "memory"); }

__device__ __forceinline__ void st4(unsigned voff, u4 v, const void* sbase) {
    asm volatile("global_store_dwordx4 %0, %1, %2 sc0 sc1\n\ts_nop 1" :: "v"(voff), "v"(v), "s"(sbase) : "memory");
}
__device__ __forceinline__ void st1(unsigned voff, unsigned v, const void* sbase) {
    asm volatile("global_store_dword %0, %1, %2 sc0 sc1" :: "v"(voff), "v"(v), "s"(sbase) : "memory");
}
typedef short s2v __attribute__((ext_vector_type(2)));
typedef unsigned short us2v __attribute__((ext_vector_type(2)));
__device__ __forceinline__ unsigned pmax_i16(unsigned a, unsigned b) { return __builtin_bit_cast(unsigned, __builtin_elementwise_max(__builtin_bit_cast(s2v, a), __builtin_bit_cast(s2v, b))); }
__device__ __forceinline__ unsigned pmax_u16(unsigned a, unsigned b) { return __builtin_bit_cast(unsigned, __builtin_elementwise_max(__builtin_bit_cast(us2v, a), __builtin_bit_cast(us2v, b))); }
__device__ __forceinline__ unsigned bytemask(unsigned s) { return (s - (s >> 7)) | s; }
#define PERM(hi, lo, sel) __builtin_amdgcn_perm((hi), (lo), (sel))
__device__ __forceinline__ unsigned pk(float a, float b) {
    h2 r; r.x = (_Float16)a; r.y = (_Float16)b; return __builtin_bit_cast(unsigned, r);
}
__device__ __forceinline__ float gelu(float y) {
    const float k1 = -2.0f * 0.7978845608028654f * 1.4426950408889634f, k2 = k1 * 0.044715f;
    const float e = __builtin_amdgcn_exp2f(y * (k1 + k2 * y * y));
    return y * __builtin_amdgcn_rcpf(1.0f + e);
}

#define STAMP(i) do {} while (0)
#define STAMPW(i) do {} while (0)

__device__ __forceinline__ bool wave_wait(gu32* pa, gu32* pb, u64 need, u64& ra, u64& rb, unsigned target, gu32* tmo, unsigned call, int lane) {
    const u64 t0 = __builtin_amdgcn_s_memrealtime();
    for (;;) {
        const unsigned va = __hip_atomic_load(pa, __ATOMIC_RELAXED, __HIP_MEMORY_SCOPE_SYSTEM);
        const unsigned vb = __hip_atomic_load(pb, __ATOMIC_RELAXED, __HIP_MEMORY_SCOPE_SYSTEM);
        const unsigned vt = __hip_atomic_load(tmo, __ATOMIC_RELAXED, __HIP_MEMORY_SCOPE_AGENT);
        if ((unsigned)__builtin_amdgcn_readfirstlane(vt) == call) return true;
        ra = __ballot((int)(va - target) >= 0); rb = __ballot((int)(vb - target) >= 0);
        if ((ra & rb & need) == need) return false;
        if (__builtin_amdgcn_s_memrealtime() - t0 > DEADLINE) {
            if (lane == 0) __hip_atomic_store(tmo, call, __ATOMIC_RELAXED, __HIP_MEMORY_SCOPE_AGENT);
            return true;
        }
        __builtin_amdgcn_s_sleep(POLL_SLEEP);
    }
}

__device__ __forceinline__ void image_of(int i, bool& isA, int& panel, int& j) {
    if (i < NIMG1) { j = i / PER1; const int g = i - j * PER1; isA = g < 16; panel = isA ? g : g - 16; }
    else { const int q = i - NIMG1; j = q / (16 - NB1); isA = false; panel = NB1 + q - j * (16 - NB1); }
}
__device__ __forceinline__ void job_loads(f4 (&L)[8], const char* x, const char* w, int i, int dest, int e, unsigned vxe, unsigned vl) {
    bool isA; int panel, j; image_of(i, isA, panel, j);
    const char* b = isA ? x + ((size_t)(panel * 256 + e * 32) * KS + j * 64) * 4 : w + ((size_t)(j * 64 + e * 8) * NG + dest * NS + panel * 256) * 4;
    const unsigned ve = isA ? vxe : vl, vo = isA ? (vxe ^ 128u) : vl;
    const unsigned s2 = isA ? 8 * KS * 4 : 2 * NG * 4, s1 = isA ? 16 : NG * 4;
#pragma unroll
    for (int r = 0; r < 8; ++r) L[r] = LOADF4((const gf4*)(b + ((r >> 1) * s2 + (r & 1) * s1) + ((r & 2) ? vo : ve)));
}
__device__ __forceinline__ void job_pack(const f4 (&L)[8], u4 (&Q)[4], bool isA) {
    if (isA) {
#pragma unroll
        for (int u = 0; u < 4; ++u) { Q[u].x = pk(L[2 * u].x, L[2 * u].y); Q[u].y = pk(L[2 * u].z, L[2 * u].w); Q[u].z = pk(L[2 * u + 1].x, L[2 * u + 1].y); Q[u].w = pk(L[2 * u + 1].z, L[2 * u + 1].w); }
    } else {
#pragma unroll
        for (int t = 0; t < 4; ++t) { Q[t].x = pk(L[0][t], L[1][t]); Q[t].y = pk(L[2][t], L[3][t]); Q[t].z = pk(L[4][t], L[5][t]); Q[t].w = pk(L[6][t], L[7][t]); }
    }
}

__global__ __launch_bounds__(NTHR, 2) void gemm_rs_kernel(const Params P) {
    __shared__ __attribute__((aligned(16))) unsigned char smem[LDS_BYTES];
    LAS unsigned char* lds = (LAS unsigned char*)smem;
    LAS unsigned* lctl = (LAS unsigned*)(lds + LDS_CTL);
    const int tid = threadIdx.x, lane = tid & 63, wid = __builtin_amdgcn_readfirstlane(tid >> 6);
    const int bid = blockIdx.x, rank = P.rank;
    gu32* ctl = (gu32*)P.ws[rank];
    STAMP(0);

    const bool sender = bid < NSEND;
    const int sq = bid >> 3, sdest = sender ? (rank + 1 + sq) & 7 : rank;
    const char* xb = (const char*)P.x; const char* wb = (const char*)P.w;
    const unsigned vl = (unsigned)lane * 16u;
    const unsigned vxe = (unsigned)((lane >> 3) * (KS * 4) + (((lane & 7) ^ (lane >> 4)) * 32));
    const int ifirst = sender ? (bid & 7) : bid - NSEND;
    f4 L0[8], L1[8];
    job_loads(L0, xb, wb, ifirst, sdest, wid, vxe, vl);

    gu32* callp = ctl + W_CALL + bid * 32;
    const unsigned call = __builtin_amdgcn_readfirstlane(__hip_atomic_load(callp, __ATOMIC_RELAXED, __HIP_MEMORY_SCOPE_AGENT)) + 1u;
    if (tid < 64) lctl[tid] = tid < 3 ? (unsigned)(ifirst + 8 * tid) : 0u;
    const unsigned slot = CTL_B + (call & 1u) * SLOT_B;
    gu32* tmo = ctl + W_TMO;
    char* dws = P.ws[(unsigned)sdest & 7u];
#define FLAG_OF(isA, panel, j) ((gu32*)dws + (W_FLAG + ((((isA) ? 0 : 16) + (panel)) * 8 + rank) * 32 + (j)))
    if (!sender) {
        WAIT_L(0); BAR; CFENCE;
        if (tid == 0) __hip_atomic_store(callp, call, __ATOMIC_RELAXED, __HIP_MEMORY_SCOPE_AGENT);
        for (int i = ifirst; i < NIMG; i += NGEM) {
            bool isA; int panel, j; image_of(i, isA, panel, j);
            if (i != ifirst) job_loads(L0, xb, wb, i, rank, wid, vxe, vl);
            u4 Q[4]; job_pack(L0, Q, isA);
            const char* db = dws + (slot + (isA ? 0u : OP_B) + (unsigned)((panel * NKT + j) * TILE_B + wid * (isA ? 4096 : 2048)));
            const int hi = isA ? 2048 : HALF_B;
            st4(vl, Q[0], db); st4(vl, Q[1], db + 1024); st4(vl, Q[2], db + hi); st4(vl, Q[3], db + hi + 1024);
            WAIT_V(0); BAR; CFENCE;
            if (tid == 0) __hip_atomic_store(FLAG_OF(isA, panel, j), call, __ATOMIC_RELAXED, __HIP_MEMORY_SCOPE_SYSTEM);
        }
    } else {
        {
            bool isA; int panel, j; image_of(ifirst, isA, panel, j);
            u4 Q[4]; job_pack(L0, Q, isA);
            const int o = wid * (isA ? 4096 : 2048) + lane * 16, hi = isA ? 2048 : HALF_B;
            *(LAS u4*)(lds + o) = Q[0]; *(LAS u4*)(lds + o + 1024) = Q[1]; *(LAS u4*)(lds + o + hi) = Q[2]; *(LAS u4*)(lds + o + hi + 1024) = Q[3];
        }
        if (wid >= 4) { job_loads(L0, xb, wb, ifirst + 8, sdest, 2 * (wid - 4), vxe, vl); job_loads(L1, xb, wb, ifirst + 8, sdest, 2 * (wid - 4) + 1, vxe, vl); }
        WAIT_L(0); BAR; CFENCE;
        if (tid == 0) __hip_atomic_store(callp, call, __ATOMIC_RELAXED, __HIP_MEMORY_SCOPE_AGENT);
        const int pbase = 8 + ((rank - sdest - 1) & 7);
        if (wid < 4) {
            gu32* prev_flag = nullptr;
            for (int n = 0;; ++n) {
                const int i = __builtin_amdgcn_readfirstlane(lctl[n & 3]);
                if (i >= NIMG) break;
                u4 Q[8];
#pragma unroll
                for (int r = 0; r < 8; ++r) Q[r] = *(const LAS u4*)(lds + (n & 1) * TILE_B + wid * 8192 + r * 1024 + lane * 16);
                bool isA; int panel, j; image_of(i, isA, panel, j);
                const char* db = dws + (slot + (isA ? 0u : OP_B) + (unsigned)((panel * NKT + pbase + j * 7) * TILE_B + wid * 8192));
#pragma unroll
                for (int r = 0; r < 8; ++r) {
                    wait_v<SF - 1>();
                    if (r == SF - 1 && n > 0) {
                        unsigned old = 0;
                        if (lane == 0) old = __hip_atomic_fetch_add(lctl + L_CNT + ((n - 1) & 3), 1u, __ATOMIC_RELAXED, __HIP_MEMORY_SCOPE_WORKGROUP);
                        if ((__builtin_amdgcn_readfirstlane(old) & 3u) == 3u && lane == 0) __hip_atomic_store(prev_flag, call, __ATOMIC_RELAXED, __HIP_MEMORY_SCOPE_SYSTEM);
                    }
                    st4(vl, Q[r], db + r * 1024);
                }
                prev_flag = FLAG_OF(isA, panel, j);
                if (wid == 0 && (n & 3) == 3) STAMPW(21 + ((n >> 2) & 7));
                BAR; CFENCE;
            }
            WAIT_V(0); BAR; CFENCE;
            if (tid == 0) __hip_atomic_store(prev_flag, call, __ATOMIC_RELAXED, __HIP_MEMORY_SCOPE_SYSTEM);
        } else {
            gu32* tick = ctl + W_TICK + sq * 32;
            const unsigned tbase = (call - 1u) * (NIMG - 24 + 8) - 24u;
            bool more = true;
            for (int n = 0;; ++n) {
                const int i = __builtin_amdgcn_readfirstlane(lctl[n & 3]);
                if (i >= NIMG) break;
                const int i1 = __builtin_amdgcn_readfirstlane(lctl[(n + 1) & 3]), i2 = __builtin_amdgcn_readfirstlane(lctl[(n + 2) & 3]);
                unsigned t = NIMG;
                if (wid == 4 && more && lane == 0) t = __hip_atomic_fetch_add(tick, 1u, __ATOMIC_RELAXED, __HIP_MEMORY_SCOPE_AGENT) - tbase;
                if (i1 < NIMG) {
                    bool isA; int panel, j; image_of(i1, isA, panel, j);
                    u4 Q0[4], Q1[4]; job_pack(L0, Q0, isA); job_pack(L1, Q1, isA);
                    const int js = isA ? 4096 : 2048, o = ((n + 1) & 1) * TILE_B + 2 * (wid - 4) * js + lane * 16, hi = isA ? 2048 : HALF_B;
                    *(LAS u4*)(lds + o) = Q0[0]; *(LAS u4*)(lds + o + 1024) = Q0[1]; *(LAS u4*)(lds + o + hi) = Q0[2]; *(LAS u4*)(lds + o + hi + 1024) = Q0[3];
                    *(LAS u4*)(lds + o + js) = Q1[0]; *(LAS u4*)(lds + o + js + 1024) = Q1[1]; *(LAS u4*)(lds + o + js + hi) = Q1[2]; *(LAS u4*)(lds + o + js + hi + 1024) = Q1[3];
                }
                CFENCE;
                const int il = i2 < NIMG ? i2 : i;
                job_loads(L0, xb, wb, il, sdest, 2 * (wid - 4), vxe, vl); job_loads(L1, xb, wb, il, sdest, 2 * (wid - 4) + 1, vxe, vl);
                if (wid == 4) {
                    t = __builtin_amdgcn_readfirstlane(t);
                    more = t < (unsigned)NIMG;
                    if (lane == 0) lctl[(n + 3) & 3] = more ? t : (unsigned)NIMG;
                }
                WAIT_L(0); BAR; CFENCE;
            }
            WAIT_V(0); BAR; CFENCE;
        }
    }
    STAMP(16);
    int pm, pn;
    {
        const int g = sender ? bid : bid - NSEND, xcd = g & 7, idx = g >> 3;
        if (sender) { const int d = (xcd + idx + 1) & 7; pm = 2 * d + (idx >= 3); pn = idx >= 3 ? 9 + idx : 13 + idx; }
        else if (idx < 24) { pm = 2 * (idx & 7) + (xcd & 1); pn = 3 * (xcd >> 1) + (idx >> 3); }
        else { pm = 2 * xcd; pn = 12; }
    }

    const int wr = wid >> 2, wc = wid & 3, fr = lane & 15, fq = lane >> 4;
    gu32* fa = ctl + W_FLAG + (pm * 8 + (lane & 7)) * 32 + (lane >> 3);
    gu32* fb = ctl + W_FLAG + ((16 + pn) * 8 + (lane & 7)) * 32 + (lane >> 3);
    const u64 own = 0x0101010101010101ull << rank;
    u64 ra = 0, rb = 0;
    bool bad = false;
#define STEP_WAIT(need, last) do { if (!bad && ((last) || (ra & rb & (need)) != (need))) bad = wave_wait(fa, fb, (need), ra, rb, call, tmo, call, lane); } while (0)
    if (wid == 0) { STEP_WAIT(own, false); WAIT_V(0); STAMPW(1); }
    CFENCE; BAR; CFENCE;

    const char* cA = P.ws[rank] + (slot + (unsigned)(pm * NKT) * TILE_B);
    const char* cB = P.ws[rank] + (slot + OP_B + (unsigned)(pn * NKT) * TILE_B);
    const unsigned vst = (unsigned)tid * 16u;
    const unsigned ldsw = (unsigned)wid * 1024u;
    const int a0 = (wr * 64 + fr) * 128 + ((fq ^ (fr >> 1)) * 16), a1 = a0 ^ 64;
    const int b0 = fq * 2048 + (wc * 16 + fr) * 16;
#define SA(b, h) (((b) * 2 + (h)) * HALF_B)
#define SB(b, h) ((4 + (b) * 2 + (h)) * HALF_B)
#define STAGE(bufoff, gbase) do { _Pragma("unroll") for (int _i = 0; _i < 2; ++_i) \
        __builtin_amdgcn_global_load_lds((const unsigned*)((gbase) + vst + _i * 8192), (LAS unsigned*)(lds + (bufoff) + ldsw + _i * 8192), 16, 0, GLDS_AUX); } while (0)
#define LDA(dst, b, h) do { _Pragma("unroll") for (int m = 0; m < 4; ++m) { dst[m][0] = *(const LAS h8*)(lds + SA(b, h) + a0 + m * 2048); dst[m][1] = *(const LAS h8*)(lds + SA(b, h) + a1 + m * 2048); } } while (0)
#define LDB(dst, b, h) do { _Pragma("unroll") for (int n = 0; n < 2; ++n) _Pragma("unroll") for (int k = 0; k < 2; ++k) dst[n][k] = *(const LAS h8*)(lds + SB(b, h) + b0 + k * 8192 + n * 1024); } while (0)
#define MMA(ai, bj, At, Bt) do { __builtin_amdgcn_s_setprio(1); _Pragma("unroll") for (int m = 0; m < 4; ++m) _Pragma("unroll") for (int n = 0; n < 2; ++n) _Pragma("unroll") for (int k = 0; k < 2; ++k) \
        acc[ai][bj][m][n] = __builtin_amdgcn_mfma_f32_16x16x32_f16(At[m][k], Bt[n][k], acc[ai][bj][m][n], 0, 0, 0); __builtin_amdgcn_s_setprio(0); } while (0)
    f4 acc[2][2][4][2];
#pragma unroll
    for (int a = 0; a < 2; ++a)
#pragma unroll
        for (int b = 0; b < 2; ++b)
#pragma unroll
            for (int m = 0; m < 4; ++m)
#pragma unroll
                for (int n = 0; n < 2; ++n) acc[a][b][m][n] = (f4){0.f, 0.f, 0.f, 0.f};
    h8 At[4][2], B0[2][2], B1[2][2];
    STAGE(SB(0, 0), cB); STAGE(SA(0, 0), cA); STAGE(SB(0, 1), cB + HALF_B); STAGE(SA(0, 1), cA + HALF_B);
    if (wr == 1) BAR;
    WAIT_V(4); BAR;
    STAGE(SB(1, 0), cB + TILE_B); STAGE(SA(1, 0), cA + TILE_B); STAGE(SB(1, 1), cB + TILE_B + HALF_B);
    WAIT_V(6); BAR;
    int jn = 0, bnd = 8;
    for (int t = 0; t < NKT; t += 2) {
        const int t2 = (t + 2) & (NKT - 1);
        const char* a1p = cA + (size_t)(t + 1) * TILE_B;
        const char* a2 = cA + (size_t)t2 * TILE_B; const char* b2 = cB + (size_t)t2 * TILE_B;
        const char* a3 = a2 + TILE_B; const char* b3 = b2 + TILE_B;
        if (t + 3 >= bnd && jn < 8) {
            if (wid == 0) {
                STEP_WAIT((0xFFull << (8 * jn)) & ~own, jn == 7);
                if (jn == 7 && lane == 0) lctl[L_BAD] = bad ? 1u : 0u;
                STAMPW(2 + jn);
            }
            CFENCE; BAR; CFENCE;
            ++jn; bnd += 7;
        }
        LDB(B0, 0, 0); SCHED; LDA(At, 0, 0); STAGE(SA(1, 1), a1p + HALF_B);
        WAIT_L(8); BAR; WAIT_L(0); MMA(0, 0, At, B0); BAR; SCHED;
        LDB(B1, 0, 1); STAGE(SB(0, 0), b2);
        BAR; WAIT_L(0); MMA(0, 1, At, B1); BAR;
        LDA(At, 0, 1); STAGE(SA(0, 0), a2);
        BAR; WAIT_L(0); MMA(1, 0, At, B0); BAR; SCHED;
        STAGE(SB(0, 1), b2 + HALF_B);
        WAIT_V(6); BAR; MMA(1, 1, At, B1); BAR;
        LDB(B0, 1, 0); SCHED; LDA(At, 1, 0); STAGE(SA(0, 1), a2 + HALF_B);
        WAIT_L(8); BAR; WAIT_L(0); MMA(0, 0, At, B0); BAR; SCHED;
        LDB(B1, 1, 1); STAGE(SB(1, 0), b3);
        BAR; WAIT_L(0); MMA(0, 1, At, B1); BAR;
        LDA(At, 1, 1); STAGE(SA(1, 0), a3);
        BAR; WAIT_L(0); MMA(1, 0, At, B0); BAR; SCHED;
        STAGE(SB(1, 1), b3 + HALF_B);
        WAIT_V(6); BAR; MMA(1, 1, At, B1); BAR;
    }
    WAIT_V(0);
    if (wr == 0) BAR;
    CFENCE; BAR; CFENCE;
    STAMP(17);

    const unsigned dest = ((unsigned)pm >> 1) & 7u;
    int tz = threadIdx.x; asm volatile("" : "+v"(tz));
    const int l2 = tz & 63;
    const unsigned vo = (unsigned)((l2 >> 4) * 4 * (NG * 4) + (l2 & 15) * 16);
    {
        const bool poison = lctl[L_BAD] != 0u;
        const float qnan = __builtin_nanf("");
#define GV(a) gelu(a)
#define ROW4(ai, m, r) (poison ? (f4){qnan, qnan, qnan, qnan} : (f4){GV(acc[ai][0][m][0][r]), GV(acc[ai][0][m][1][r]), GV(acc[ai][1][m][0][r]), GV(acc[ai][1][m][1][r])})
        const char* ob = (const char*)P.out[dest] + ((size_t)((pm & 1) * 256 + wr * 64) * NG + rank * NS + pn * 256 + wc * 64) * 4;
        if (dest == (unsigned)rank) {
#pragma unroll
            for (int ai = 0; ai < 2; ++ai)
#pragma unroll
                for (int m = 0; m < 4; ++m)
#pragma unroll
                    for (int r = 0; r < 4; ++r) { const f4 v = ROW4(ai, m, r); wait_v<7>(); st4(vo, __builtin_bit_cast(u4, v), ob + (size_t)(ai * 128 + m * 16 + r) * (NG * 4)); }
        } else {
            const int tile = (pm & 1) * 16 + pn;
            const char* zb = P.ws[dest] + (OUTZ + (unsigned)((rank * 32 + tile) * 32 + wid) * UNIT_B);
            gu32* uf = (gu32*)P.ws[dest] + W_UNIT + (rank * 32 + tile) * 32 + wid;
            const unsigned act = pn >= NB1 ? ACT2 : ACT1;
#pragma unroll
            for (int g = 0; g < 4; ++g) {
                const int ai = g >> 1;
                unsigned lo[16], mid[8], top[8], mp = 0u, mn = 0u;
#pragma unroll
                for (int q = 0; q < 8; ++q) {
                    const int m = (g & 1) * 2 + (q >> 2), r = q & 3;
                    const u4 u = __builtin_bit_cast(u4, ROW4(ai, m, r));
                    lo[2 * q] = PERM(u.y, u.x, 0x05040100u); lo[2 * q + 1] = PERM(u.w, u.z, 0x05040100u);
                    const unsigned h01 = PERM(u.y, u.x, 0x07060302u), h23 = PERM(u.w, u.z, 0x07060302u);
                    mid[q] = PERM(h23, h01, 0x06040200u); top[q] = PERM(h23, h01, 0x07050301u);
                    mp = pmax_i16(mp, pmax_i16(h01, h23)); mn = pmax_u16(mn, pmax_u16(h01, h23));
                }
                const unsigned bp1 = max((mp >> 8) & 0x7fu, mp >> 24), n0 = mn & 0xffffu, n1 = mn >> 16, nm = max(n0, n1);
                unsigned bb = (bp1 << 16) | ((nm & 0x8000u) ? (nm >> 8) & 0x7fu : 0u);
#pragma unroll
                for (int o = 32; o >= 1; o >>= 1) bb = pmax_u16(bb, (unsigned)__shfl_xor((int)bb, o));
                bb = __builtin_amdgcn_readfirstlane(bb);
                const unsigned bp = bb >> 16, bn = bb & 0x7fu, bp4 = bp * 0x01010101u, bn4 = bn * 0x01010101u;
                unsigned code[8], worst = 0u;
#pragma unroll
                for (int q = 0; q < 8; ++q) {
                    const unsigned sg = top[q] & 0x80808080u, mk = bytemask(sg);
                    const unsigned d = ((bn4 & mk) | (bp4 & ~mk)) - (top[q] & 0x7f7f7f7fu);
                    code[q] = d | (sg >> 4); worst |= d + (sg >> 7);
                }
                if (__any((worst & 0xf8f8f8f8u) != 0u)) {
#pragma unroll
                    for (int q = 0; q < 8; ++q) {
                        const unsigned sg = top[q] & 0x80808080u, d = code[q] & ~(sg >> 4) , dd = (((bn4 & bytemask(sg)) | (bp4 & ~bytemask(sg))) - (top[q] & 0x7f7f7f7fu));
                        (void)d;
                        const unsigned z = (dd + (sg >> 7)) & 0xf8f8f8f8u, nz = (((z & 0x7f7f7f7fu) + 0x7f7f7f7fu) | z) & 0x80808080u;
                        if (__any(nz != 0u)) {
                            const char* rb_ = ob + (size_t)(ai * 128 + ((g & 1) * 2 + (q >> 2)) * 16 + (q & 3)) * (NG * 4);
                            const unsigned h01 = PERM(top[q], mid[q], 0x05010400u), h23 = PERM(top[q], mid[q], 0x07030602u);
                            if (nz & 0x80u) st1(vo, PERM(h01, lo[2 * q], 0x05040100u), rb_);
                            if (nz & 0x8000u) st1(vo + 4u, PERM(h01, lo[2 * q], 0x07060302u), rb_);
                            if (nz & 0x800000u) st1(vo + 8u, PERM(h23, lo[2 * q + 1], 0x05040100u), rb_);
                            if (nz & 0x80000000u) st1(vo + 12u, PERM(h23, lo[2 * q + 1], 0x07060302u), rb_);
                            const unsigned mk = bytemask(nz);
                            code[q] = (code[q] & ~mk) | (0x0f0f0f0fu & mk);
                        }
                    }
                }
                u4 Z[7];
#pragma unroll
                for (int i = 0; i < 4; ++i) Z[i] = (u4){lo[4 * i], lo[4 * i + 1], lo[4 * i + 2], lo[4 * i + 3]};
                Z[4] = (u4){mid[0], mid[1], mid[2], mid[3]}; Z[5] = (u4){mid[4], mid[5], mid[6], mid[7]};
                Z[6] = (u4){code[0] | (code[1] << 4), code[2] | (code[3] << 4), code[4] | (code[5] << 4), code[6] | (code[7] << 4)};
                asm volatile("" : "+v"(Z[0]), "+v"(Z[1]), "+v"(Z[2]), "+v"(Z[3]), "+v"(Z[4]), "+v"(Z[5]), "+v"(Z[6]));
                while ((unsigned)__builtin_amdgcn_readfirstlane(__hip_atomic_load(lctl + L_TURN, __ATOMIC_RELAXED, __HIP_MEMORY_SCOPE_WORKGROUP)) + act <= (unsigned)(g * 8 + wid))
                    __builtin_amdgcn_s_sleep(4);
#pragma unroll
                for (int i = 0; i < 7; ++i) { wait_v<EF - 1>(); st4((unsigned)l2 * 16u, Z[i], zb + (g * 8 * UNIT_B + i * 1024)); }
                if (l2 == 0) __hip_atomic_fetch_add(lctl + L_TURN, 1u, __ATOMIC_RELAXED, __HIP_MEMORY_SCOPE_WORKGROUP);
                WAIT_V(0);
                if (l2 == 0) __hip_atomic_store(uf + g * 8, (call << 14) | (bp << 7) | bn, __ATOMIC_RELAXED, __HIP_MEMORY_SCOPE_SYSTEM);
            }
        }
    }
    if (wid == 0) STAMPW(18);
    if (wid == 7) STAMPW(19);

    {
        const int qd = bid & 7;
        gu32* dq = ctl + W_DQ + qd * 32;
        constexpr unsigned U1 = 32 * 21, UQ = U1 + 32 * 7;
        const unsigned tbase = (call - 1u) * (UQ + NWG);
        for (;;) {
            unsigned n = 0;
            if (l2 == 0) n = __hip_atomic_fetch_add(dq, 1u, __ATOMIC_RELAXED, __HIP_MEMORY_SCOPE_AGENT) - tbase;
            n = __builtin_amdgcn_readfirstlane(n);
            if (n >= UQ) break;
            int k, sg, tile;
            if (n < U1) { k = n / 21; const int T = (n - k * 21) * 8 + qd; sg = T / 24; const int ta = T - sg * 24; tile = (ta & 1) * 16 + (ta >> 1); }
            else { const int n2 = n - U1; k = n2 / 7; sg = n2 - k * 7; tile = (qd & 1) * 16 + NB1 + (qd >> 1); }
            const int src = (rank + 1 + sg) & 7;
            gu32* uf = ctl + W_UNIT + (src * 32 + tile) * 32 + k;
            unsigned hd; bool dead = false;
            {
                const u64 t0 = __builtin_amdgcn_s_memrealtime();
                for (;;) {
                    const unsigned v = __hip_atomic_load(l2 == 0 ? uf : tmo, __ATOMIC_RELAXED, __HIP_MEMORY_SCOPE_SYSTEM);
                    hd = __builtin_amdgcn_readfirstlane(v);
                    if ((unsigned)__builtin_amdgcn_readlane(v, 1) == call) { dead = true; break; }
                    if ((hd >> 14) == (call & 0x3ffffu)) break;
                    if (__builtin_amdgcn_s_memrealtime() - t0 > DEADLINE) {
                        if (l2 == 0) __hip_atomic_store(tmo, call, __ATOMIC_RELAXED, __HIP_MEMORY_SCOPE_AGENT);
                        dead = true; break;
                    }
                    __builtin_amdgcn_s_sleep(POLL_SLEEP);
                }
            }
            const int g = k >> 3, w = k & 7;
            const char* ob = (const char*)P.out[rank] + ((size_t)((tile >> 4) * 256 + (g >> 1) * 128 + (w >> 2) * 64 + (g & 1) * 32) * NG + src * NS + (tile & 15) * 256 + (w & 3) * 64) * 4;
            if (dead) {
                const unsigned qn = 0x7fc00000u;
#pragma unroll
                for (int q = 0; q < 8; ++q) st4(vo, (u4){qn, qn, qn, qn}, ob + (size_t)((q >> 2) * 16 + (q & 3)) * (NG * 4));
                continue;
            }
            const char* zb = P.ws[rank] + (OUTZ + (unsigned)((src * 32 + tile) * 32 + k) * UNIT_B);
            u4 Z[7];
            asm volatile("global_load_dwordx4 %0, %7, %8 sc0 sc1\n\tglobal_load_dwordx4 %1, %7, %8 offset:1024 sc0 sc1\n\t"
                         "global_load_dwordx4 %2, %7, %8 offset:2048 sc0 sc1\n\tglobal_load_dwordx4 %3, %7, %8 offset:3072 sc0 sc1\n\t"
                         "global_load_dwordx4 %4, %7, %9 sc0 sc1\n\tglobal_load_dwordx4 %5, %7, %9 offset:1024 sc0 sc1\n\t"
                         "global_load_dwordx4 %6, %7, %9 offset:2048 sc0 sc1\n\ts_waitcnt vmcnt(0)"
                         : "=&v"(Z[0]), "=&v"(Z[1]), "=&v"(Z[2]), "=&v"(Z[3]), "=&v"(Z[4]), "=&v"(Z[5]), "=&v"(Z[6])
                         : "v"((unsigned)l2 * 16u), "s"(zb), "s"(zb + 4096) : "memory");
            const unsigned bp4 = (((hd >> 7) & 0x7fu) | 0x80u) * 0x01010101u, bn4 = ((hd & 0x7fu) | 0x80u) * 0x01010101u;
#pragma unroll
            for (int q = 0; q < 8; ++q) {
                const unsigned c = (Z[6][q >> 1] >> (4 * (q & 1))) & 0x0f0f0f0fu, sgn = (c & 0x08080808u) << 4, mk = bytemask(sgn);
                const unsigned tp = ((((bn4 & mk) | (bp4 & ~mk)) - (c & 0x07070707u)) & 0x7f7f7f7fu) | sgn, md = Z[4 + (q >> 2)][q & 3];
                const unsigned h01 = PERM(tp, md, 0x05010400u), h23 = PERM(tp, md, 0x07030602u), l01 = Z[q >> 1][2 * (q & 1)], l23 = Z[q >> 1][2 * (q & 1) + 1];
                const u4 u = {PERM(h01, l01, 0x05040100u), PERM(h01, l01, 0x07060302u), PERM(h23, l23, 0x05040100u), PERM(h23, l23, 0x07060302u)};
                const char* rb_ = ob + (size_t)((q >> 2) * 16 + (q & 3)) * (NG * 4);
                const unsigned raw = (c + 0x01010101u) & 0x10101010u;
                if (__any(raw != 0u)) {
                    if (!(raw & 0x10u)) st1(vo, u.x, rb_);
                    if (!(raw & 0x1000u)) st1(vo + 4u, u.y, rb_);
                    if (!(raw & 0x100000u)) st1(vo + 8u, u.z, rb_);
                    if (!(raw & 0x10000000u)) st1(vo + 12u, u.w, rb_);
                } else st4(vo, u, rb_);
            }
        }
    }
    if (wid == 0) STAMPW(20);
}

}

extern "C" void kernel_launch(int rank, int nranks, ncclComm_t comm, hipStream_t stream,
                              void* const* d_in, const int* in_sizes, int n_in,
                              void* d_out, int out_size, const int* in_dtypes, int out_dtype,
                              void* const* d_out_all, void* const* d_ws_all, size_t ws_bytes) {
    (void)comm; (void)in_dtypes; (void)out_dtype; (void)d_out;
    if (nranks != NR || (unsigned)rank >= (unsigned)NR || n_in < 2 || ws_bytes < WS_NEED || in_sizes[0] != MG * KS || in_sizes[1] != KS * NG || out_size != MS * NG) {
        fprintf(stderr, "kernel_launch: unexpected problem shape\n");
        return;
    }
    Params P;
    P.x = (const float*)d_in[0]; P.w = (const float*)d_in[1];
    for (int p = 0; p < NR; ++p) { P.out[p] = (float*)d_out_all[p]; P.ws[p] = (char*)d_ws_all[p]; }
    P.rank = rank; P.pad = 0;
    hipLaunchKernelGGL(gemm_rs_kernel, dim3(NWG), dim3(NTHR), 0, stream, P);
    HIP_CHECK(hipGetLastError());
}
```

```cpp
#include <hip/hip_runtime.h>
#include <rccl/rccl.h>
#include <cstdio>

#define HIP_CHECK(expr) do { hipError_t _e = (expr); if (_e != hipSuccess) { \
    fprintf(stderr, "HIP error %s at %s:%d\n", hipGetErrorString(_e), __FILE__, __LINE__); \
    return; } } while (0)

#define CW 4
#define PF 2
#define POLL_SLEEP 16

namespace {

typedef _Float16 h8 __attribute__((ext_vector_type(8)));
typedef _Float16 h2 __attribute__((ext_vector_type(2)));
typedef float f4 __attribute__((ext_vector_type(4)));
typedef unsigned u4 __attribute__((ext_vector_type(4)));
typedef short s2v __attribute__((ext_vector_type(2)));
typedef unsigned short us2v __attribute__((ext_vector_type(2)));
typedef unsigned long long u64;
typedef __attribute__((address_space(1))) unsigned gu32;
typedef __attribute__((address_space(1))) f4 gf4;
#define LAS __attribute__((address_space(3)))

constexpr int NR = 8;
constexpr int MG = 4096, KG = 4096, NG = 32768;
constexpr int KS = KG / NR;
constexpr int NS = NG / NR;
constexpr int MS = MG / NR;
constexpr int NWG = 256, NTHR = 512;
constexpr int NCOM = 56, NGEM = NWG - NCOM;
constexpr int TILE_B = 256 * 64 * 2;
constexpr int HALF_B = TILE_B / 2;
constexpr int NKT = KG / 64;
constexpr int NCONV = 128 + 8 * 128;
constexpr unsigned NI_IN = 1024, NI = 2048;
constexpr unsigned OP_B = 16u * NKT * TILE_B;
constexpr unsigned UNIT_B = 7 * 1024;
constexpr unsigned CTL_B = 1u << 20;
constexpr unsigned O_A = 0, O_B = OP_B, O_XB = 2 * OP_B, O_OUT = 3 * OP_B, SLOT_B = O_OUT + NR * 32 * 32 * UNIT_B;
constexpr size_t WS_NEED = (size_t)CTL_B + 2 * (size_t)SLOT_B;
constexpr int W_TMO = 0;
constexpr int W_LAND = 32;
constexpr int W_AVAIL = W_LAND + 256 * 32;
constexpr int W_UNIT = W_AVAIL + 256 * 32;
constexpr int W_TICK = W_UNIT + 256 * 32;
constexpr int W_CALL = W_TICK + 8 * 32;
constexpr int W_STAMP = 131072;
static_assert(W_CALL + NWG * 32 <= W_STAMP, "control layout");
constexpr u64 DEADLINE = 40000000ull;

constexpr int LDS_CTL = 8 * HALF_B;
constexpr int LDS_BYTES = LDS_CTL + 256;
constexpr int L_BAD = 0;

struct Params {
    const float* x; const float* w;
    float* out[NR]; char* ws[NR];
    int rank; int pad;
};

#define WAIT_V(n) asm volatile("s_waitcnt vmcnt(" #n ")" ::: "memory")
#define WAIT_L(n) asm volatile("s_waitcnt lgkmcnt(" #n ")" ::: "memory")
#define BAR __builtin_amdgcn_s_barrier()
#define SCHED __builtin_amdgcn_sched_barrier(0)
#define CFENCE asm volatile("" ::: "memory")
#define RFL(x) __builtin_amdgcn_readfirstlane(x)
#define PERM(hi, lo, sel) __builtin_amdgcn_perm((hi), (lo), (sel))
template <int N> __device__ __forceinline__ void wait_v() { asm volatile("s_waitcnt vmcnt(%0)" :: "i"(N) : "memory"); }

__device__ __forceinline__ void st4(unsigned voff, u4 v, const void* sbase) {
    asm volatile("global_store_dwordx4 %0, %1, %2 sc0 sc1\n\ts_nop 1" :: "v"(voff), "v"(v), "s"(sbase) : "memory");
}
__device__ __forceinline__ void st1(unsigned voff, unsigned v, const void* sbase) {
    asm volatile("global_store_dword %0, %1, %2 sc0 sc1" :: "v"(voff), "v"(v), "s"(sbase) : "memory");
}
__device__ __forceinline__ void dma(unsigned voff, const void* sbase, unsigned ldsoff) {
    unsigned keep;
    asm volatile("s_mov_b32 %0, m0\n\ts_mov_b32 m0, %3\n\ts_nop 0\n\tglobal_load_lds_dwordx4 %1, %2 sc0 sc1\n\ts_mov_b32 m0, %0"
                 : "=&s"(keep) : "v"(voff), "s"(sbase), "s"(ldsoff) : "memory");
}
__device__ __forceinline__ unsigned pmax_i16(unsigned a, unsigned b) { return __builtin_bit_cast(unsigned, __builtin_elementwise_max(__builtin_bit_cast(s2v, a), __builtin_bit_cast(s2v, b))); }
__device__ __forceinline__ unsigned pmax_u16(unsigned a, unsigned b) { return __builtin_bit_cast(unsigned, __builtin_elementwise_max(__builtin_bit_cast(us2v, a), __builtin_bit_cast(us2v, b))); }
__device__ __forceinline__ unsigned bytemask(unsigned s) { return (s - (s >> 7)) | s; }
__device__ __forceinline__ unsigned pk(float a, float b) {
    h2 r; r.x = (_Float16)a; r.y = (_Float16)b; return __builtin_bit_cast(unsigned, r);
}
__device__ __forceinline__ float gelu(float y) {
    const float k1 = -2.0f * 0.7978845608028654f * 1.4426950408889634f, k2 = k1 * 0.044715f;
    const float e = __builtin_amdgcn_exp2f(y * (k1 + k2 * y * y));
    return y * __builtin_amdgcn_rcpf(1.0f + e);
}

#define STAMPW(i) do {} while (0)
#define STAMP(i) do { if (wid == 0) STAMPW(i); } while (0)

__device__ __forceinline__ bool wave_wait(gu32* pa, gu32* pb, u64 need, u64& ra, u64& rb, unsigned target, gu32* tmo, unsigned call, int lane) {
    const u64 t0 = __builtin_amdgcn_s_memrealtime();
    for (;;) {
        const unsigned va = __hip_atomic_load(pa, __ATOMIC_RELAXED, __HIP_MEMORY_SCOPE_SYSTEM);
        const unsigned vb = __hip_atomic_load(pb, __ATOMIC_RELAXED, __HIP_MEMORY_SCOPE_SYSTEM);
        const unsigned vt = __hip_atomic_load(tmo, __ATOMIC_RELAXED, __HIP_MEMORY_SCOPE_AGENT);
        if ((unsigned)RFL(vt) == call) return true;
        ra = __ballot((int)(va - target) >= 0); rb = __ballot((int)(vb - target) >= 0);
        if ((ra & rb & need) == need) return false;
        if (__builtin_amdgcn_s_memrealtime() - t0 > DEADLINE) {
            if (lane == 0) __hip_atomic_store(tmo, call, __ATOMIC_RELAXED, __HIP_MEMORY_SCOPE_AGENT);
            return true;
        }
        __builtin_amdgcn_s_sleep(POLL_SLEEP);
    }
}
__device__ __forceinline__ bool word_wait(gu32* ctl, int word, int shift, unsigned want, unsigned& seen, unsigned call, int lane) {
    const u64 t0 = __builtin_amdgcn_s_memrealtime();
    for (;;) {
        const unsigned v = __hip_atomic_load(ctl + (lane == 0 ? word : W_TMO), __ATOMIC_RELAXED, __HIP_MEMORY_SCOPE_SYSTEM);
        seen = RFL(v);
        if ((unsigned)__builtin_amdgcn_readlane(v, 1) == call) return true;
        if ((seen >> shift) == want) return false;
        if (__builtin_amdgcn_s_memrealtime() - t0 > DEADLINE) {
            if (lane == 0) __hip_atomic_store(ctl + W_TMO, call, __ATOMIC_RELAXED, __HIP_MEMORY_SCOPE_AGENT);
            return true;
        }
        __builtin_amdgcn_s_sleep(POLL_SLEEP / 2);
    }
}

__device__ __forceinline__ void conv_of(int c, bool& isA, int& dest, int& panel, int& j) {
    if (c < 1024) { j = c >> 7; const int g = c & 127; isA = g < 16; panel = isA ? g : (g - 16) >> 3; dest = g & 7; }
    else { const int q = c - 1024; j = q >> 4; isA = false; panel = 14 + ((q >> 3) & 1); dest = q & 7; }
}
__device__ __forceinline__ void image_of(int i, int me, bool& isA, int& panel, int& j) {
    if (i < 224) { j = i / 28; const int g = i - j * 28; isA = g < 14; panel = isA ? (2 * me + 2 + g) & 15 : g - 14; }
    else { isA = i >= 240; j = (i >> 1) & 7; panel = (isA ? 2 * me : 14) + (i & 1); }
}
__device__ __forceinline__ void unit_of(int u, int& tile, int& k) {
    if (u < 28 * 32) { k = u / 28; const int t = u - k * 28; tile = (t & 1) * 16 + (t >> 1); }
    else { const int q = u - 28 * 32; k = q >> 2; tile = (q & 1) * 16 + 14 + ((q >> 1) & 1); }
}
__device__ __forceinline__ void job_loads(f4 (&L)[8], const char* x, const char* w, int c, int e, unsigned vxe, unsigned vl) {
    bool isA; int dest, panel, j; conv_of(c, isA, dest, panel, j);
    const char* b = isA ? x + ((size_t)(panel * 256 + e * 32) * KS + j * 64) * 4 : w + ((size_t)(j * 64 + e * 8) * NG + dest * NS + panel * 256) * 4;
    const unsigned ve = isA ? vxe : vl, vo = isA ? (vxe ^ 128u) : vl;
    const unsigned s2 = isA ? 8 * KS * 4 : 2 * NG * 4, s1 = isA ? 16 : NG * 4;
#pragma unroll
    for (int r = 0; r < 8; ++r) L[r] = *(const gf4*)(b + ((r >> 1) * s2 + (r & 1) * s1) + ((r & 2) ? vo : ve));
}
__device__ __forceinline__ void job_pack(const f4 (&L)[8], u4 (&Q)[4], bool isA) {
    if (isA) {
#pragma unroll
        for (int u = 0; u < 4; ++u) { Q[u].x = pk(L[2 * u].x, L[2 * u].y); Q[u].y = pk(L[2 * u].z, L[2 * u].w); Q[u].z = pk(L[2 * u + 1].x, L[2 * u + 1].y); Q[u].w = pk(L[2 * u + 1].z, L[2 * u + 1].w); }
    } else {
#pragma unroll
        for (int t = 0; t < 4; ++t) { Q[t].x = pk(L[0][t], L[1][t]); Q[t].y = pk(L[2][t], L[3][t]); Q[t].z = pk(L[4][t], L[5][t]); Q[t].w = pk(L[6][t], L[7][t]); }
    }
}

struct Com {
    gu32* ctl; char* me_ws; const char* src_ws; char* out; LAS unsigned char* ring; unsigned ring_off;
    unsigned slot, call, vl, vo; int rank, src, lane;
};
__device__ __forceinline__ int item_word(const Com& C, unsigned n) {
    if (n < NI_IN) { bool isA; int panel, j; image_of(n >> 2, C.rank, isA, panel, j); return W_AVAIL + ((C.src * 2 + (isA ? 0 : 1)) * 16 + panel) * 32 + j; }
    if (n < NI) { int tile, k; unit_of(n - NI_IN, tile, k); return W_UNIT + (C.src * 32 + tile) * 32 + k; }
    return W_TMO;
}
__device__ __forceinline__ void put_quarter(const Com& C, unsigned n, int half) {
    bool isA; int panel, j; image_of(n >> 2, C.rank, isA, panel, j);
    const char* db = C.me_ws + (C.slot + (isA ? O_A : O_B) + (unsigned)((panel * NKT + 8 + ((C.src - C.rank - 1) & 7) + j * 7) * TILE_B) + (n & 3) * 8192);
    u4 Q[8];
#pragma unroll
    for (int r = 0; r < 8; ++r) Q[r] = *(const LAS u4*)(C.ring + half * 8192 + r * 1024 + C.lane * 16);
#pragma unroll
    for (int r = 0; r < 8; ++r) st4(C.vl, Q[r], db + r * 1024);
}
__device__ __forceinline__ void count_quarter(const Com& C, unsigned n) {
    bool isA; int panel, j; image_of(n >> 2, C.rank, isA, panel, j);
    if (C.lane == 0) __hip_atomic_fetch_add(C.ctl + W_LAND + (((isA ? 0 : 16) + panel) * 8 + C.src) * 32 + j, 1u, __ATOMIC_RELAXED, __HIP_MEMORY_SCOPE_AGENT);
}
__device__ __forceinline__ const char* unit_rows(const Com& C, unsigned n) {
    int tile, k; unit_of(n - NI_IN, tile, k);
    const int g = k >> 3, w = k & 7;
    return C.out + ((size_t)((tile >> 4) * 256 + (g >> 1) * 128 + (w >> 2) * 64 + (g & 1) * 32) * NG + C.src * NS + (tile & 15) * 256 + (w & 3) * 64) * 4;
}
__device__ __forceinline__ void put_unit(const Com& C, unsigned n, unsigned hd, int half) {
    const char* ob = unit_rows(C, n);
    u4 Z[7];
#pragma unroll
    for (int r = 0; r < 7; ++r) Z[r] = *(const LAS u4*)(C.ring + half * 8192 + r * 1024 + C.lane * 16);
    const unsigned bp4 = (((hd >> 7) & 0x7fu) | 0x80u) * 0x01010101u, bn4 = ((hd & 0x7fu) | 0x80u) * 0x01010101u;
#pragma unroll
    for (int q = 0; q < 8; ++q) {
        const unsigned c = (Z[6][q >> 1] >> (4 * (q & 1))) & 0x0f0f0f0fu, sgn = (c & 0x08080808u) << 4, mk = bytemask(sgn);
        const unsigned tp = ((((bn4 & mk) | (bp4 & ~mk)) - (c & 0x07070707u)) & 0x7f7f7f7fu) | sgn, md = Z[4 + (q >> 2)][q & 3];
        const unsigned h01 = PERM(tp, md, 0x05010400u), h23 = PERM(tp, md, 0x07030602u), l01 = Z[q >> 1][2 * (q & 1)], l23 = Z[q >> 1][2 * (q & 1) + 1];
        const u4 u = {PERM(h01, l01, 0x05040100u), PERM(h01, l01, 0x07060302u), PERM(h23, l23, 0x05040100u), PERM(h23, l23, 0x07060302u)};
        const char* rb = ob + (size_t)((q >> 2) * 16 + (q & 3)) * (NG * 4);
        const unsigned raw = (c + 0x01010101u) & 0x10101010u;
        if (__any(raw != 0u)) {
            if (!(raw & 0x10u)) st1(C.vo, u.x, rb);
            if (!(raw & 0x1000u)) st1(C.vo + 4u, u.y, rb);
            if (!(raw & 0x100000u)) st1(C.vo + 8u, u.z, rb);
            if (!(raw & 0x10000000u)) st1(C.vo + 12u, u.w, rb);
        } else st4(C.vo, u, rb);
    }
}
template <bool IN> __device__ __forceinline__ void pull_items(const Com& C, const Params& P, gu32* tick, unsigned tbase, unsigned& cur, unsigned& nxt, unsigned& seen, bool& dead) {
    constexpr int CNT = IN ? 8 : 7;
    constexpr unsigned END = IN ? NI_IN : NI; constexpr int SH = IN ? 0 : 15;
    const unsigned want = IN ? C.call : (C.call & 0x1ffffu);
    unsigned pend = ~0u, pend_hd = 0; int half = 0;
    while (cur < END) {
        if (dead || (seen >> SH) != want || (!IN && (seen & 0x4000u))) {
            if (pend != ~0u) {
                WAIT_V(0);
                if (IN) { put_quarter(C, pend, half ^ 1); WAIT_V(0); count_quarter(C, pend); } else put_unit(C, pend, pend_hd, half ^ 1);
                pend = ~0u;
            }
            if (!dead) dead = word_wait(C.ctl, item_word(C, cur), SH, want, seen, C.call, C.lane);
            if (dead || (!IN && (seen & 0x4000u))) {
                if (dead && !IN) {
                    const char* ob = unit_rows(C, cur); const unsigned qn = 0x7fc00000u;
#pragma unroll
                    for (int q = 0; q < 8; ++q) st4(C.vo, (u4){qn, qn, qn, qn}, ob + (size_t)((q >> 2) * 16 + (q & 3)) * (NG * 4));
                }
                unsigned t = NI;
                if (nxt < NI && C.lane == 0) t = __hip_atomic_fetch_add(tick, 1u, __ATOMIC_RELAXED, __HIP_MEMORY_SCOPE_AGENT) - tbase;
                cur = nxt; nxt = RFL(t); seen = 0u;
                continue;
            }
        }
        const char* sb;
        if (IN) { bool isA; int panel, j; image_of(cur >> 2, C.rank, isA, panel, j);
                  sb = C.src_ws + (C.slot + (isA ? O_A + (unsigned)((panel * NKT + j) * TILE_B) : O_XB + (unsigned)(((C.rank * 16 + panel) * 8 + j) * TILE_B)) + (cur & 3) * 8192); }
        else { int tile, k; unit_of(cur - NI_IN, tile, k); sb = C.src_ws + (C.slot + O_OUT + (unsigned)((C.rank * 32 + tile) * 32 + k) * UNIT_B); }
        unsigned tv = 0, fv; u64 ex;
        asm volatile("s_mov_b64 %1, exec\n\ts_mov_b64 exec, 1\n\tglobal_atomic_add %0, %2, %3, off sc0\n\ts_mov_b64 exec, %1"
                     : "+v"(tv), "=&s"(ex) : "v"(tick), "v"(nxt < NI ? 1u : 0u) : "memory");
        asm volatile("global_load_dword %0, %1, %2 sc0 sc1" : "=v"(fv) : "v"((unsigned)item_word(C, nxt) * 4u), "s"(C.ctl) : "memory");
        unsigned nn = NI, sn = 0;
#pragma unroll
        for (int r = 0; r < CNT; ++r) {
            if (r < PF) wait_v<PF + 1>(); else if (r > PF && r <= 2 * PF) wait_v<PF + 7>(); else wait_v<PF - 1>();
            if (IN && r == 2 * PF + 1 && pend != ~0u) count_quarter(C, pend);
            dma(C.vl, sb + r * 1024, C.ring_off + half * 8192 + r * 1024);
            if (r == PF) {
                asm volatile("" : "+v"(tv), "+v"(fv));
                if (nxt < NI) nn = (unsigned)RFL(tv) - tbase;
                sn = RFL(fv);
                if (pend != ~0u) { if (IN) put_quarter(C, pend, half ^ 1); else put_unit(C, pend, pend_hd, half ^ 1); }
            }
        }
        if (IN && (cur & 127u) >= 124u) STAMPW(21 + (cur >> 7));
        pend = cur; pend_hd = seen; half ^= 1;
        cur = nxt; nxt = nn; seen = sn;
    }
    if (pend != ~0u) {
        WAIT_V(0);
        if (IN) { put_quarter(C, pend, half ^ 1); WAIT_V(0); count_quarter(C, pend); } else put_unit(C, pend, pend_hd, half ^ 1);
    }
}

__global__ __launch_bounds__(NTHR, 2) void gemm_rs_kernel(const Params P) {
    __shared__ __attribute__((aligned(16))) unsigned char smem[LDS_BYTES];
    LAS unsigned char* lds = (LAS unsigned char*)smem;
    LAS unsigned* lctl = (LAS unsigned*)(lds + LDS_CTL);
    const int tid = threadIdx.x, lane = tid & 63, wid = RFL(tid >> 6);
    const int bid = blockIdx.x, rank = P.rank;
    gu32* ctl = (gu32*)P.ws[rank];
    STAMP(0);
    const unsigned vl = (unsigned)lane * 16u;
    const int gid = bid - NCOM;
    const char* xb = (const char*)P.x; const char* wb = (const char*)P.w;
    const unsigned vxe = (unsigned)((lane >> 3) * (KS * 4) + (((lane & 7) ^ (lane >> 4)) * 32));
    f4 L[8];
    if (gid >= 0) job_loads(L, xb, wb, gid, wid, vxe, vl);

    gu32* callp = ctl + W_CALL + bid * 32;
    const unsigned call = RFL(__hip_atomic_load(callp, __ATOMIC_RELAXED, __HIP_MEMORY_SCOPE_AGENT)) + 1u;
    asm volatile("" :: "s"(call) : "memory");
    BAR; CFENCE;
    if (tid == 0) __hip_atomic_store(callp, call, __ATOMIC_RELAXED, __HIP_MEMORY_SCOPE_AGENT);
    const unsigned slot = CTL_B + (call & 1u) * SLOT_B;
    gu32* tmo = ctl + W_TMO;

    if (gid < 0) {
        if (wid >= CW) return;
        static_assert(PF >= 1 && 2 * PF + 1 < 8, "window");
        Com C;
        C.ctl = ctl; C.me_ws = P.ws[rank]; C.rank = rank; C.src = (rank + 1 + (bid >> 3)) & 7; C.src_ws = P.ws[(unsigned)C.src & 7u]; C.out = (char*)P.out[rank];
        C.ring = lds + wid * 16384; C.ring_off = (unsigned)(size_t)C.ring; C.slot = slot; C.call = call; C.vl = vl; C.lane = lane;
        C.vo = (unsigned)((lane >> 4) * 4 * (NG * 4) + (lane & 15) * 16);
        constexpr unsigned NWV = 8 * CW;
        const unsigned tbase = (call - 1u) * (NI - 2 * NWV + NWV) - 2 * NWV;
        unsigned cur = (unsigned)((bid & 7) * CW + wid), nxt = cur + NWV, seen = 0u; bool dead = false;
        gu32* tick = ctl + W_TICK + (bid >> 3) * 32;
        pull_items<true>(C, P, tick, tbase, cur, nxt, seen, dead);
        STAMP(16);
        pull_items<false>(C, P, tick, tbase, cur, nxt, seen, dead);
        STAMP(20);
        return;
    }

    {
        gu32* wsl = (gu32*)P.ws[lane & 7];
        for (int c = gid; c < NCONV; c += NGEM) {
            bool isA; int dest, panel, j; conv_of(c, isA, dest, panel, j);
            u4 Q[4]; job_pack(L, Q, isA);
            asm volatile("" : "+v"(Q[0]), "+v"(Q[1]), "+v"(Q[2]), "+v"(Q[3]));
            const unsigned io = isA ? O_A + (unsigned)((panel * NKT + j) * TILE_B) : dest == rank ? O_B + (unsigned)((panel * NKT + j) * TILE_B) : O_XB + (unsigned)(((dest * 16 + panel) * 8 + j) * TILE_B);
            const char* db = P.ws[rank] + (slot + io + (unsigned)(wid * (isA ? 4096 : 2048)));
            const int hi = isA ? 2048 : HALF_B;
            st4(vl, Q[0], db); st4(vl, Q[1], db + 1024); st4(vl, Q[2], db + hi); st4(vl, Q[3], db + hi + 1024);
            if (c + NGEM < NCONV) { job_loads(L, xb, wb, c + NGEM, wid, vxe, vl); wait_v<8>(); }
            else WAIT_V(0);
            BAR; CFENCE;
            if (wid == 0 && lane < NR && (isA || lane == dest)) {
                const bool mine = lane == rank;
                __hip_atomic_store(wsl + (mine ? W_LAND + (((isA ? 0 : 16) + panel) * 8 + rank) * 32 : W_AVAIL + ((rank * 2 + (isA ? 0 : 1)) * 16 + panel) * 32) + j,
                                   mine ? call * 4u : call, __ATOMIC_RELAXED, __HIP_MEMORY_SCOPE_SYSTEM);
            }
        }
        WAIT_V(0);
        STAMP(16);
    }

    const int wr = wid >> 2, wc = wid & 3;
    const int xcd = gid & 7, idx = gid >> 3, xh = xcd & 1, xq = xcd >> 1;
    const int n1 = 25 - (xq & 1);
    for (int round = 0; round < 2; ++round) {
        int pm, pn;
        if (round == 0 && idx < n1) { const int u = (xq * 49 + 1) / 2 + idx; pn = u / 7; pm = 2 * ((rank + 1 + u - pn * 7) & 7) + xh; }
        else if (round == 0 || idx < 3) { const int a = round == 0 ? 6 : idx + 3 * (xq & 1); pn = 14 + (xq >> 1); pm = 2 * ((rank + 1 + a) & 7) + xh; }
        else if (idx < 7) { pm = 2 * rank + xh; pn = 4 * xq + idx - 3; }
        else break;
        pm = RFL(pm); pn = RFL(pn);
        int tz = threadIdx.x; asm volatile("" : "+v"(tz));
        const int ln = tz & 63, fr = ln & 15, fq = ln >> 4;
        gu32* fa = ctl + W_LAND + (pm * 8 + (ln & 7)) * 32 + (ln >> 3);
        gu32* fb = ctl + W_LAND + ((16 + pn) * 8 + (ln & 7)) * 32 + (ln >> 3);
        const u64 own = 0x0101010101010101ull << rank;
        u64 ra = 0, rb = 0;
        bool bad = false;
#define STEP_WAIT(need, last) do { if (!bad && ((last) || (ra & rb & (need)) != (need))) bad = wave_wait(fa, fb, (need), ra, rb, call * 4u, tmo, call, ln); } while (0)
        if (wid == 0) { STEP_WAIT(own, false); WAIT_V(0); if (round == 0) STAMPW(1); }
        CFENCE; BAR; CFENCE;

        const char* cA = P.ws[rank] + (slot + O_A + (unsigned)(pm * NKT) * TILE_B);
        const char* cB = P.ws[rank] + (slot + O_B + (unsigned)(pn * NKT) * TILE_B);
        const unsigned vst = (unsigned)tz * 16u;
        const unsigned ldsw = (unsigned)wid * 1024u;
        const int a0 = (wr * 64 + fr) * 128 + ((fq ^ (fr >> 1)) * 16), a1 = a0 ^ 64;
        const int b0 = fq * 2048 + (wc * 16 + fr) * 16;
#define SA(b, h) (((b) * 2 + (h)) * HALF_B)
#define SB(b, h) ((4 + (b) * 2 + (h)) * HALF_B)
#define STAGE(bufoff, gbase) do { _Pragma("unroll") for (int _i = 0; _i < 2; ++_i) \
        __builtin_amdgcn_global_load_lds((const unsigned*)((gbase) + vst + _i * 8192), (LAS unsigned*)(lds + (bufoff) + ldsw + _i * 8192), 16, 0, 17); } while (0)
#define LDA(dst, b, h) do { _Pragma("unroll") for (int m = 0; m < 4; ++m) { dst[m][0] = *(const LAS h8*)(lds + SA(b, h) + a0 + m * 2048); dst[m][1] = *(const LAS h8*)(lds + SA(b, h) + a1 + m * 2048); } } while (0)
#define LDB(dst, b, h) do { _Pragma("unroll") for (int n = 0; n < 2; ++n) _Pragma("unroll") for (int k = 0; k < 2; ++k) dst[n][k] = *(const LAS h8*)(lds + SB(b, h) + b0 + k * 8192 + n * 1024); } while (0)
#define MMA(ai, bj, At, Bt) do { __builtin_amdgcn_s_setprio(1); _Pragma("unroll") for (int m = 0; m < 4; ++m) _Pragma("unroll") for (int n = 0; n < 2; ++n) _Pragma("unroll") for (int k = 0; k < 2; ++k) \
        acc[ai][bj][m][n] = __builtin_amdgcn_mfma_f32_16x16x32_f16(At[m][k], Bt[n][k], acc[ai][bj][m][n], 0, 0, 0); __builtin_amdgcn_s_setprio(0); } while (0)
        f4 acc[2][2][4][2];
#pragma unroll
        for (int a = 0; a < 2; ++a)
#pragma unroll
            for (int b = 0; b < 2; ++b)
#pragma unroll
                for (int m = 0; m < 4; ++m)
#pragma unroll
                    for (int n = 0; n < 2; ++n) acc[a][b][m][n] = (f4){0.f, 0.f, 0.f, 0.f};
        h8 At[4][2], B0[2][2], B1[2][2];
        STAGE(SB(0, 0), cB); STAGE(SA(0, 0), cA); STAGE(SB(0, 1), cB + HALF_B); STAGE(SA(0, 1), cA + HALF_B);
        if (wr == 1) BAR;
        WAIT_V(4); BAR;
        STAGE(SB(1, 0), cB + TILE_B); STAGE(SA(1, 0), cA + TILE_B); STAGE(SB(1, 1), cB + TILE_B + HALF_B);
        WAIT_V(6); BAR;
        int jn = 0, bnd = 8;
        for (int t = 0; t < NKT; t += 2) {
            const int t2 = (t + 2) & (NKT - 1);
            const char* a1p = cA + (size_t)(t + 1) * TILE_B;
            const char* a2 = cA + (size_t)t2 * TILE_B; const char* b2 = cB + (size_t)t2 * TILE_B;
            const char* a3 = a2 + TILE_B; const char* b3 = b2 + TILE_B;
            if (t + 3 >= bnd && jn < 8) {
                if (wid == 0) {
                    STEP_WAIT((0xFFull << (8 * jn)) & ~own, jn == 7);
                    if (jn == 7 && ln == 0) lctl[L_BAD] = bad ? 1u : 0u;
                    if (round == 0) STAMPW(2 + jn);
                }
                CFENCE; BAR; CFENCE;
                ++jn; bnd += 7;
            }
            LDB(B0, 0, 0); SCHED; LDA(At, 0, 0); STAGE(SA(1, 1), a1p + HALF_B);
            WAIT_L(8); BAR; WAIT_L(0); MMA(0, 0, At, B0); BAR; SCHED;
            LDB(B1, 0, 1); STAGE(SB(0, 0), b2);
            BAR; WAIT_L(0); MMA(0, 1, At, B1); BAR;
            LDA(At, 0, 1); STAGE(SA(0, 0), a2);
            BAR; WAIT_L(0); MMA(1, 0, At, B0); BAR; SCHED;
            STAGE(SB(0, 1), b2 + HALF_B);
            WAIT_V(6); BAR; MMA(1, 1, At, B1); BAR;
            LDB(B0, 1, 0); SCHED; LDA(At, 1, 0); STAGE(SA(0, 1), a2 + HALF_B);
            WAIT_L(8); BAR; WAIT_L(0); MMA(0, 0, At, B0); BAR; SCHED;
            LDB(B1, 1, 1); STAGE(SB(1, 0), b3);
            BAR; WAIT_L(0); MMA(0, 1, At, B1); BAR;
            LDA(At, 1, 1); STAGE(SA(1, 0), a3);
            BAR; WAIT_L(0); MMA(1, 0, At, B0); BAR; SCHED;
            STAGE(SB(1, 1), b3 + HALF_B);
            WAIT_V(6); BAR; MMA(1, 1, At, B1); BAR;
        }
        WAIT_V(0);
        if (wr == 0) BAR;
        CFENCE;
        STAMP(round == 0 ? 17 : 11);

        const unsigned dest = ((unsigned)pm >> 1) & 7u;
        int ty = threadIdx.x; asm volatile("" : "+v"(ty));
        const int l2 = ty & 63;
        const unsigned vo = (unsigned)((l2 >> 4) * 4 * (NG * 4) + (l2 & 15) * 16);
        const bool poison = lctl[L_BAD] != 0u;
        const float qnan = __builtin_nanf("");
#define ROW4(ai, m, r) (poison ? (f4){qnan, qnan, qnan, qnan} : (f4){gelu(acc[ai][0][m][0][r]), gelu(acc[ai][0][m][1][r]), gelu(acc[ai][1][m][0][r]), gelu(acc[ai][1][m][1][r])})
        const char* ob = (const char*)P.out[dest] + ((size_t)((pm & 1) * 256 + wr * 64) * NG + rank * NS + pn * 256 + wc * 64) * 4;
        if (dest == (unsigned)rank) {
#pragma unroll
            for (int ai = 0; ai < 2; ++ai)
#pragma unroll
                for (int m = 0; m < 4; ++m)
#pragma unroll
                    for (int r = 0; r < 4; ++r) { const f4 v = ROW4(ai, m, r); st4(vo, __builtin_bit_cast(u4, v), ob + (size_t)(ai * 128 + m * 16 + r) * (NG * 4)); }
        } else {
            const int tile = (pm & 1) * 16 + pn;
            const char* zb = P.ws[rank] + (slot + O_OUT + (unsigned)((dest * 32 + tile) * 32 + wid) * UNIT_B);
            gu32* uf = (gu32*)P.ws[dest] + W_UNIT + (rank * 32 + tile) * 32 + wid;
#pragma unroll
            for (int g = 0; g < 4; ++g) {
                const int ai = g >> 1;
                unsigned lo[16], mid[8], top[8], mp = 0u, mn = 0u;
#pragma unroll
                for (int q = 0; q < 8; ++q) {
                    const int m = (g & 1) * 2 + (q >> 2), r = q & 3;
                    const u4 u = __builtin_bit_cast(u4, ROW4(ai, m, r));
                    lo[2 * q] = PERM(u.y, u.x, 0x05040100u); lo[2 * q + 1] = PERM(u.w, u.z, 0x05040100u);
                    const unsigned h01 = PERM(u.y, u.x, 0x07060302u), h23 = PERM(u.w, u.z, 0x07060302u);
                    mid[q] = PERM(h23, h01, 0x06040200u); top[q] = PERM(h23, h01, 0x07050301u);
                    mp = pmax_i16(mp, pmax_i16(h01, h23)); mn = pmax_u16(mn, pmax_u16(h01, h23));
                }
                const unsigned nm = max(mn & 0xffffu, mn >> 16);
                unsigned bb = (max((mp >> 8) & 0x7fu, mp >> 24) << 16) | ((nm & 0x8000u) ? (nm >> 8) & 0x7fu : 0u);
#pragma unroll
                for (int o = 32; o >= 1; o >>= 1) bb = pmax_u16(bb, (unsigned)__shfl_xor((int)bb, o));
                bb = RFL(bb);
                const unsigned bp = bb >> 16, bn = bb & 0x7fu, bp4 = bp * 0x01010101u, bn4 = bn * 0x01010101u;
                unsigned code[8], worst = 0u, hd = (call << 15) | (bp << 7) | bn;
#pragma unroll
                for (int q = 0; q < 8; ++q) {
                    const unsigned sg = top[q] & 0x80808080u, mk = bytemask(sg);
                    const unsigned d = ((bn4 & mk) | (bp4 & ~mk)) - (top[q] & 0x7f7f7f7fu);
                    code[q] = d | (sg >> 4); worst |= d + (sg >> 7);
                }
                if (__any((worst & 0xf8f8f8f8u) != 0u)) {
                    unsigned nz[8], cnt = 0u;
#pragma unroll
                    for (int q = 0; q < 8; ++q) {
                        const unsigned sg = top[q] & 0x80808080u, mk = bytemask(sg);
                        const unsigned z = (((bn4 & mk) | (bp4 & ~mk)) - (top[q] & 0x7f7f7f7fu) + (sg >> 7)) & 0xf8f8f8f8u;
                        nz[q] = (((z & 0x7f7f7f7fu) + 0x7f7f7f7fu) | z) & 0x80808080u; cnt += __popc(nz[q]);
                    }
#pragma unroll
                    for (int o = 32; o >= 1; o >>= 1) cnt += (unsigned)__shfl_xor((int)cnt, o);
                    const bool whole = (unsigned)RFL(cnt) > 12u;
                    if (whole) hd |= 0x4000u;
#pragma unroll
                    for (int q = 0; q < 8; ++q) {
                        if (whole || __any(nz[q] != 0u)) {
                            const char* rp = ob + (size_t)(ai * 128 + ((g & 1) * 2 + (q >> 2)) * 16 + (q & 3)) * (NG * 4);
                            const unsigned h01 = PERM(top[q], mid[q], 0x05010400u), h23 = PERM(top[q], mid[q], 0x07030602u);
                            const u4 u = {PERM(h01, lo[2 * q], 0x05040100u), PERM(h01, lo[2 * q], 0x07060302u), PERM(h23, lo[2 * q + 1], 0x05040100u), PERM(h23, lo[2 * q + 1], 0x07060302u)};
                            if (whole) st4(vo, u, rp);
                            else {
                                if (nz[q] & 0x80u) st1(vo, u.x, rp);
                                if (nz[q] & 0x8000u) st1(vo + 4u, u.y, rp);
                                if (nz[q] & 0x800000u) st1(vo + 8u, u.z, rp);
                                if (nz[q] & 0x80000000u) st1(vo + 12u, u.w, rp);
                                const unsigned mk = bytemask(nz[q]);
                                code[q] = (code[q] & ~mk) | (0x0f0f0f0fu & mk);
                            }
                        }
                    }
                }
                const char* zp = zb + g * 8 * UNIT_B;
#pragma unroll
                for (int i = 0; i < 4; ++i) st4((unsigned)l2 * 16u, (u4){lo[4 * i], lo[4 * i + 1], lo[4 * i + 2], lo[4 * i + 3]}, zp + i * 1024);
                st4((unsigned)l2 * 16u, (u4){mid[0], mid[1], mid[2], mid[3]}, zp + 4096); st4((unsigned)l2 * 16u, (u4){mid[4], mid[5], mid[6], mid[7]}, zp + 5120);
                st4((unsigned)l2 * 16u, (u4){code[0] | (code[1] << 4), code[2] | (code[3] << 4), code[4] | (code[5] << 4), code[6] | (code[7] << 4)}, zp + 6144);
                WAIT_V(0);
                if (l2 == 0) __hip_atomic_store(uf + g * 8, hd, __ATOMIC_RELAXED, __HIP_MEMORY_SCOPE_SYSTEM);
            }
        }
        STAMP(round == 0 ? 18 : 12);
        WAIT_V(0); CFENCE; BAR; CFENCE;
    }
    STAMP(20);
}

}

extern "C" void kernel_launch(int rank, int nranks, ncclComm_t comm, hipStream_t stream,
                              void* const* d_in, const int* in_sizes, int n_in,
                              void* d_out, int out_size, const int* in_dtypes, int out_dtype,
                              void* const* d_out_all, void* const* d_ws_all, size_t ws_bytes) {
    (void)comm; (void)in_dtypes; (void)out_dtype; (void)d_out;
    if (nranks != NR || (unsigned)rank >= (unsigned)NR || n_in < 2 || ws_bytes < WS_NEED || in_sizes[0] != MG * KS || in_sizes[1] != KS * NG || out_size != MS * NG) {
        fprintf(stderr, "kernel_launch: unexpected problem shape\n");
        return;
    }
    Params P;
    P.x = (const float*)d_in[0]; P.w = (const float*)d_in[1];
    for (int p = 0; p < NR; ++p) { P.out[p] = (float*)d_out_all[p]; P.ws[p] = (char*)d_ws_all[p]; }
    P.rank = rank; P.pad = 0;
    hipLaunchKernelGGL(gemm_rs_kernel, dim3(NWG), dim3(NTHR), 0, stream, P);
    HIP_CHECK(hipGetLastError());
}
```

```cpp
#include <hip/hip_runtime.h>
#include <rccl/rccl.h>
#include <cstdio>

#define HIP_CHECK(expr) do { hipError_t _e = (expr); if (_e != hipSuccess) { \
    fprintf(stderr, "HIP error %s at %s:%d\n", hipGetErrorString(_e), __FILE__, __LINE__); \
    return; } } while (0)

#define STAGGER 2
#define POLL_SLEEP 16

namespace {

typedef _Float16 h8 __attribute__((ext_vector_type(8)));
typedef _Float16 h2 __attribute__((ext_vector_type(2)));
typedef float f4 __attribute__((ext_vector_type(4)));
typedef unsigned u4 __attribute__((ext_vector_type(4)));
typedef short s2v __attribute__((ext_vector_type(2)));
typedef unsigned short us2v __attribute__((ext_vector_type(2)));
typedef unsigned long long u64;
typedef __attribute__((address_space(1))) unsigned gu32;
typedef __attribute__((address_space(1))) f4 gf4;
#define LAS __attribute__((address_space(3)))

constexpr int NR = 8;
constexpr int MG = 4096, KG = 4096, NG = 32768;
constexpr int KS = KG / NR;
constexpr int NS = NG / NR;
constexpr int MS = MG / NR;
constexpr int NWG = 256, NTHR = 512;
constexpr int NCOM = 56, NGEM = NWG - NCOM;
constexpr int TILE_B = 256 * 64 * 2;
constexpr int HALF_B = TILE_B / 2;
constexpr int NKT = KG / 64;
constexpr unsigned NI_IN = 1024, NI = 2048;
constexpr unsigned OP_B = 16u * NKT * TILE_B;
constexpr unsigned UNIT_B = 8192;
constexpr unsigned CTL_B = 1u << 20;
constexpr unsigned O_A = 0, O_B = OP_B, O_X = 2 * OP_B, O_OUT = 3 * OP_B, SLOT_B = O_OUT + NR * 32 * 32 * UNIT_B;
constexpr size_t WS_NEED = (size_t)CTL_B + 2 * (size_t)SLOT_B;
constexpr int W_TMO = 0;
constexpr int W_LAND = 32;
constexpr int W_AVAIL = W_LAND + 256 * 32;
constexpr int W_UNIT = W_AVAIL + 512 * 32;
constexpr unsigned H_RAW = 1u << 18;
constexpr int W_TICK = W_UNIT + 256 * 32;
constexpr int W_CALL = W_TICK + 8 * 32;
constexpr int W_STAMP = 131072;
static_assert(W_CALL + NWG * 32 <= W_STAMP, "control layout");
constexpr u64 DEADLINE = 40000000ull;

constexpr int LDS_CTL = 8 * HALF_B;
constexpr int LDS_BYTES = LDS_CTL + 1024;
constexpr int L_BAD = 0, L_CNT = 2, L_HD = 64;

struct Params {
    const float* x; const float* w;
    float* out[NR]; char* ws[NR];
    int rank; int pad;
};

#define WAIT_V(n) asm volatile("s_waitcnt vmcnt(" #n ")" ::: "memory")
#define WAIT_L(n) asm volatile("s_waitcnt lgkmcnt(" #n ")" ::: "memory")
#define BAR __builtin_amdgcn_s_barrier()
#define SCHED __builtin_amdgcn_sched_barrier(0)
#define CFENCE asm volatile("" ::: "memory")
#define RFL(x) __builtin_amdgcn_readfirstlane(x)
#define PERM(hi, lo, sel) __builtin_amdgcn_perm((hi), (lo), (sel))
template <int N> __device__ __forceinline__ void wait_v() { asm volatile("s_waitcnt vmcnt(%0)" :: "i"(N) : "memory"); }

__device__ __forceinline__ void st4(unsigned voff, u4 v, const void* sbase) {
    asm volatile("global_store_dwordx4 %0, %1, %2 sc0 sc1\n\ts_nop 1" :: "v"(voff), "v"(v), "s"(sbase) : "memory");
}
__device__ __forceinline__ void st1(unsigned voff, unsigned v, const void* sbase) {
    asm volatile("global_store_dword %0, %1, %2 sc0 sc1" :: "v"(voff), "v"(v), "s"(sbase) : "memory");
}
__device__ __forceinline__ void dma(unsigned voff, const void* sbase, unsigned ldsoff) {
    unsigned keep;
    asm volatile("s_mov_b32 %0, m0\n\ts_mov_b32 m0, %3\n\ts_nop 0\n\tglobal_load_lds_dwordx4 %1, %2 sc0 sc1\n\ts_mov_b32 m0, %0"
                 : "=&s"(keep) : "v"(voff), "s"(sbase), "s"(ldsoff) : "memory");
}
__device__ __forceinline__ void dma_part(unsigned voff, const void* sbase, unsigned ldsoff, unsigned lanes) {
    unsigned keep; u64 ex;
    asm volatile("s_mov_b32 %0, m0\n\ts_mov_b32 m0, %4\n\ts_mov_b64 %1, exec\n\ts_mov_b64 exec, %5\n\tglobal_load_lds_dwordx4 %2, %3 sc0 sc1\n\ts_mov_b64 exec, %1\n\ts_mov_b32 m0, %0"
                 : "=&s"(keep), "=&s"(ex) : "v"(voff), "s"(sbase), "s"(ldsoff), "s"(~0ull >> (64u - lanes)) : "memory");
}
__device__ __forceinline__ unsigned pmax_i16(unsigned a, unsigned b) { return __builtin_bit_cast(unsigned, __builtin_elementwise_max(__builtin_bit_cast(s2v, a), __builtin_bit_cast(s2v, b))); }
__device__ __forceinline__ unsigned pmax_u16(unsigned a, unsigned b) { return __builtin_bit_cast(unsigned, __builtin_elementwise_max(__builtin_bit_cast(us2v, a), __builtin_bit_cast(us2v, b))); }
__device__ __forceinline__ unsigned bytemask(unsigned s) { return (s - (s >> 7)) | s; }
__device__ __forceinline__ unsigned pk(float a, float b) {
    h2 r; r.x = (_Float16)a; r.y = (_Float16)b; return __builtin_bit_cast(unsigned, r);
}
__device__ __forceinline__ float gelu(float y) {
    const float k1 = -2.0f * 0.7978845608028654f * 1.4426950408889634f, k2 = k1 * 0.044715f;
    const float e = __builtin_amdgcn_exp2f(y * (k1 + k2 * y * y));
    return y * __builtin_amdgcn_rcpf(1.0f + e);
}

#define STAMPW(i) do {} while (0)
#define STAMP(i) do { if (wid == 0) STAMPW(i); } while (0)

__device__ __forceinline__ bool wave_wait(gu32* pa, gu32* pb, u64 need, u64& ra, u64& rb, unsigned target, gu32* tmo, unsigned call, int lane) {
    const u64 t0 = __builtin_amdgcn_s_memrealtime();
    for (;;) {
        const unsigned va = __hip_atomic_load(pa, __ATOMIC_RELAXED, __HIP_MEMORY_SCOPE_SYSTEM);
        const unsigned vb = __hip_atomic_load(pb, __ATOMIC_RELAXED, __HIP_MEMORY_SCOPE_SYSTEM);
        const unsigned vt = __hip_atomic_load(tmo, __ATOMIC_RELAXED, __HIP_MEMORY_SCOPE_AGENT);
        if ((unsigned)RFL(vt) == call) return true;
        ra = __ballot((int)(va - target) >= 0); rb = __ballot((int)(vb - target) >= 0);
        if ((ra & rb & need) == need) return false;
        if (__builtin_amdgcn_s_memrealtime() - t0 > DEADLINE) {
            if (lane == 0) __hip_atomic_store(tmo, call, __ATOMIC_RELAXED, __HIP_MEMORY_SCOPE_AGENT);
            return true;
        }
        __builtin_amdgcn_s_sleep(POLL_SLEEP);
    }
}
__device__ __forceinline__ bool word_wait(gu32* ctl, int word, int shift, unsigned want, unsigned& seen, unsigned call, int lane) {
    const u64 t0 = __builtin_amdgcn_s_memrealtime();
    for (;;) {
        const unsigned v = __hip_atomic_load(ctl + (lane == 0 ? word : W_TMO), __ATOMIC_RELAXED, __HIP_MEMORY_SCOPE_SYSTEM);
        seen = RFL(v);
        if ((unsigned)__builtin_amdgcn_readlane(v, 1) == call) return true;
        if ((seen >> shift) == want) return false;
        if (__builtin_amdgcn_s_memrealtime() - t0 > DEADLINE) {
            if (lane == 0) __hip_atomic_store(ctl + W_TMO, call, __ATOMIC_RELAXED, __HIP_MEMORY_SCOPE_AGENT);
            return true;
        }
        __builtin_amdgcn_s_sleep(POLL_SLEEP / 2);
    }
}

__device__ __forceinline__ void conv_of(int c, int me, bool& isA, int& dest, int& panel, int& j) {
    int b = -1;
    isA = false; dest = me;
    if (c < 200) { j = c >= 100; const int g = c - j * 100; isA = g < 16; panel = g; b = g - 16; }
    else if (c < 296) { isA = true; j = 2 + ((c - 200) >> 4); panel = (c - 200) & 15; }
    else if (c < 400) { j = (c - 296) / 12; panel = c - 296 - j * 12; }
    else if (c < 904) { j = (c - 400) / 84; b = c - 400 - j * 84; j += 2; }
    else { const int q = c - 904; j = q >> 5; panel = 12 + ((q >> 3) & 3); dest = q & 7; }
    if (b >= 0) { panel = b / 7; dest = (me + 1 + b - panel * 7) & 7; }
}
__device__ __forceinline__ void image_of(int i, int me, bool& isA, int& panel, int& j) {
    if (i < 208) { j = i / 26; const int g = i - j * 26; isA = g < 14; panel = isA ? (2 * me + 2 + g) & 15 : g - 14; }
    else if (i < 240) { isA = false; j = (i - 208) >> 2; panel = 12 + (i & 3); }
    else { isA = true; j = (i - 240) >> 1; panel = 2 * me + (i & 1); }
}
__device__ __forceinline__ int late_peer(int x, int k) { return (k + 3 * x) % 7; }
__device__ __forceinline__ unsigned late_order(int a) {
    unsigned o = 0u; int n = 0;
    for (int x = 0; x < 8; ++x) if ((a + 7 * 4 - 3 * x) % 7 < 4) o |= (unsigned)x << (4 * n++);
    for (int x = 0; x < 8; ++x) if ((a + 7 * 4 - 3 * x) % 7 >= 4) o |= (unsigned)x << (4 * n++);
    return o;
}
__device__ __forceinline__ void unit_of(int u, unsigned late, int& tile, int& k) {
    if (u < 768) { const int g = u / 192, t = (u - g * 192) >> 3; k = g * 8 + (u & 7); tile = (t & 1) * 16 + (t >> 1); }
    else { const int x = (late >> ((u - 768) >> 5) * 4) & 7; k = u & 31; tile = (x & 1) * 16 + 12 + (x >> 1); }
}
__device__ __forceinline__ void job_loads(f4 (&L)[8], const char* x, const char* w, bool isA, int dest, int panel, int j, int e, unsigned vxe, unsigned vl) {
    const char* b = isA ? x + ((size_t)(panel * 256 + e * 32) * KS + j * 64) * 4 : w + ((size_t)(j * 64 + e * 8) * NG + dest * NS + panel * 256) * 4;
    const unsigned ve = isA ? vxe : vl, vo = isA ? (vxe ^ 128u) : vl;
    const unsigned s2 = isA ? 8 * KS * 4 : 2 * NG * 4, s1 = isA ? 16 : NG * 4;
#pragma unroll
    for (int r = 0; r < 8; ++r) L[r] = *(const gf4*)(b + ((r >> 1) * s2 + (r & 1) * s1) + ((r & 2) ? vo : ve));
}
__device__ __forceinline__ void job_pack(const f4 (&L)[8], u4 (&Q)[4], bool isA) {
    if (isA) {
#pragma unroll
        for (int u = 0; u < 4; ++u) { Q[u].x = pk(L[2 * u].x, L[2 * u].y); Q[u].y = pk(L[2 * u].z, L[2 * u].w); Q[u].z = pk(L[2 * u + 1].x, L[2 * u + 1].y); Q[u].w = pk(L[2 * u + 1].z, L[2 * u + 1].w); }
    } else {
#pragma unroll
        for (int t = 0; t < 4; ++t) { Q[t].x = pk(L[0][t], L[1][t]); Q[t].y = pk(L[2][t], L[3][t]); Q[t].z = pk(L[4][t], L[5][t]); Q[t].w = pk(L[6][t], L[7][t]); }
    }
}

struct Com {
    gu32* ctl; char* me_ws; const char* src_ws; char* out; LAS unsigned char* ring; LAS unsigned* word;
    unsigned slot, call, vl, vo, late; int rank, src, lane;
};
__device__ __forceinline__ int item_word(const Com& C, unsigned n) {
    if (n < NI_IN) { bool isA; int panel, j; image_of(n >> 2, C.rank, isA, panel, j); return W_AVAIL + ((((C.call & 1) * 8 + C.src) * 2 + (isA ? 0 : 1)) * 16 + panel) * 32 + j * 4 + (n & 3); }
    if (n < NI) { int tile, k; unit_of(n - NI_IN, C.late, tile, k); return W_UNIT + (C.src * 32 + tile) * 32 + k; }
    return W_TMO;
}
__device__ __forceinline__ int h_byte(int idx, int lane) {
    return ((idx >> 5) * 2 + ((idx >> 2) & 1)) * 1024 + lane * 16 + (idx & 3) * 4 + ((idx >> 3) & 3);
}
__device__ __forceinline__ unsigned piece_off(bool isA, int q, int p) {
    return isA ? (unsigned)(q * 8192 + p * 1024) : (unsigned)((2 * q + (p >> 2)) * 2048 + ((p >> 1) & 1) * HALF_B + (p & 1) * 1024);
}
__device__ __forceinline__ void put_quarter(const Com& C, unsigned n, unsigned hd, LAS unsigned char* z) {
    bool isA; int panel, j; image_of(n >> 2, C.rank, isA, panel, j);
    const char* db = C.me_ws + (C.slot + (isA ? O_A : O_B) + (unsigned)((panel * NKT + 8 + ((C.src - C.rank - 1) & 7) + j * 7) * TILE_B));
    LAS unsigned char* zl = z + C.lane * 16;
    if (hd & H_RAW) {
#pragma unroll
        for (int r = 0; r < 8; ++r) st4(C.vl, *(const LAS u4*)(zl + r * 1024), db + piece_off(isA, n & 3, r));
    } else {
        u4 lo[4], pz[3], h[4];
#pragma unroll
        for (int r = 0; r < 4; ++r) lo[r] = *(const LAS u4*)(zl + r * 1024);
#pragma unroll
        for (int r = 0; r < 3; ++r) pz[r] = *(const LAS u4*)(zl + 4096 + r * 1024);
        const unsigned bq = ((((hd >> 8) & 31u) << 2) | 0x80u) * 0x01010101u;
        unsigned m0 = 0u, m1 = 0u;
#pragma unroll
        for (int i = 0; i < 16; ++i) {
            const int g = i >> 2, k = i & 3;
            const unsigned f = g < 3 ? pz[g][k] : ((pz[0][k] >> 6) & 0x03030303u) | ((pz[1][k] >> 4) & 0x0c0c0c0cu) | ((pz[2][k] >> 2) & 0x30303030u);
            const unsigned c = f & 0x1c1c1c1cu, es = ((c + 0x04040404u) << 2) & 0x80808080u;
            h[g][k] = ((f << 2) & 0x80808080u) | ((bq - c) & 0x7c7c7c7cu & ~bytemask(es)) | (f & 0x03030303u);
            if (i < 8) m0 |= es >> (7 - i); else m1 |= es >> (15 - i);
        }
        asm volatile("" : "+v"(lo[0]), "+v"(lo[1]), "+v"(lo[2]), "+v"(lo[3]) :: "memory");
#pragma unroll
        for (int r = 0; r < 4; ++r) *(LAS u4*)(zl + r * 1024) = h[r];
        CFENCE;
        u64 m = ((u64)m1 << 32) | m0; unsigned base = 0u;
        for (;;) {
            const u64 act = __ballot(m != 0ull);
            if (act == 0ull) break;
            if (m != 0ull) {
                const unsigned pos = base + __builtin_amdgcn_mbcnt_hi((unsigned)(act >> 32), __builtin_amdgcn_mbcnt_lo((unsigned)act, 0u));
                const int o = h_byte(__builtin_ctzll(m), C.lane);
                const unsigned e = z[7168 + (pos & 255u)];
                __hip_atomic_fetch_or((LAS unsigned*)(z + (o & ~3)), e << (2 + 8 * (o & 3)), __ATOMIC_RELAXED, __HIP_MEMORY_SCOPE_WORKGROUP);
                m &= m - 1ull;
            }
            base += (unsigned)__popcll(act);
        }
        CFENCE;
#pragma unroll
        for (int r = 0; r < 4; ++r) h[r] = *(const LAS u4*)(zl + r * 1024);
#pragma unroll
        for (int p = 0; p < 8; ++p) {
            const unsigned ha = h[p >> 1][2 * (p & 1)], hb = h[p >> 1][2 * (p & 1) + 1], la = lo[p >> 1][2 * (p & 1)], lb = lo[p >> 1][2 * (p & 1) + 1];
            st4(C.vl, (u4){PERM(ha, la, 0x05010400u), PERM(ha, la, 0x07030602u), PERM(hb, lb, 0x05010400u), PERM(hb, lb, 0x07030602u)}, db + piece_off(isA, n & 3, p));
        }
    }
    WAIT_V(0);
    if (C.lane == 0) __hip_atomic_fetch_add(C.ctl + W_LAND + (((isA ? 0 : 16) + panel) * 8 + C.src) * 32 + j, 1u, __ATOMIC_RELAXED, __HIP_MEMORY_SCOPE_AGENT);
}
__device__ __forceinline__ const char* unit_rows(const Com& C, unsigned n) {
    int tile, k; unit_of(n - NI_IN, C.late, tile, k);
    const int g = k >> 3, w = k & 7;
    return C.out + ((size_t)((tile >> 4) * 256 + (g >> 1) * 128 + (w >> 2) * 64 + (g & 1) * 32) * NG + C.src * NS + (tile & 15) * 256 + (w & 3) * 64) * 4;
}
__device__ __forceinline__ void put_unit(const Com& C, unsigned n, unsigned hd, LAS unsigned char* z) {
    const char* ob = unit_rows(C, n);
    LAS unsigned char* zl = z + C.lane * 16;
    if (hd & H_RAW) {
#pragma unroll
        for (int q = 0; q < 8; ++q) st4(C.vo, *(const LAS u4*)(zl + q * 1024), ob + (size_t)((q >> 2) * 16 + (q & 3)) * (NG * 4));
        return;
    }
    u4 Z[6], E[2]; unsigned pc[3], c[8], m = 0u;
#pragma unroll
    for (int r = 0; r < 6; ++r) Z[r] = *(const LAS u4*)(zl + r * 1024);
#pragma unroll
    for (int r = 0; r < 3; ++r) pc[r] = *(const LAS unsigned*)(z + 6144 + r * 256 + C.lane * 4);
    c[0] = pc[0]; c[1] = pc[0] >> 3; c[2] = ((pc[0] >> 6) & 0x03030303u) | (pc[1] << 2 & 0x04040404u); c[3] = pc[1] >> 1; c[4] = pc[1] >> 4;
    c[5] = ((pc[1] >> 7) & 0x01010101u) | (pc[2] << 1 & 0x06060606u); c[6] = pc[2] >> 2; c[7] = pc[2] >> 5;
    const unsigned bp4 = ((hd >> 8) & 0xffu) * 0x01010101u, bn4 = (hd & 0xffu) * 0x01010101u;
#pragma unroll
    for (int q = 0; q < 8; ++q) {
        const unsigned cc = c[q] & 0x07070707u, es = ((cc + 0x01010101u) << 4) & 0x80808080u, keep = ~bytemask(es), mk = bytemask(Z[4 + (q >> 2)][q & 3] & 0x80808080u);
        E[q >> 2][q & 3] = (((bn4 & mk) | (bp4 & ~mk)) - (cc & keep)) & keep;
        m |= es >> (7 - q);
    }
    asm volatile("" : "+v"(Z[0]), "+v"(Z[1]) :: "memory");
    *(LAS u4*)zl = E[0]; *(LAS u4*)(zl + 1024) = E[1];
    CFENCE;
    for (unsigned base = 0u;;) {
        const u64 act = __ballot(m != 0u);
        if (act == 0ull) break;
        if (m != 0u) {
            const unsigned pos = base + __builtin_amdgcn_mbcnt_hi((unsigned)(act >> 32), __builtin_amdgcn_mbcnt_lo((unsigned)act, 0u));
            const int o = h_byte(__builtin_ctz(m), C.lane);
            const unsigned e = z[6912 + (pos & 255u)];
            __hip_atomic_fetch_or((LAS unsigned*)(z + (o & ~3)), e << (8 * (o & 3)), __ATOMIC_RELAXED, __HIP_MEMORY_SCOPE_WORKGROUP);
            m &= m - 1u;
        }
        base += (unsigned)__popcll(act);
    }
    CFENCE;
    E[0] = *(const LAS u4*)zl; E[1] = *(const LAS u4*)(zl + 1024);
#pragma unroll
    for (int q = 0; q < 8; ++q) {
        const unsigned e = E[q >> 2][q & 3], sm = Z[4 + (q >> 2)][q & 3];
        const unsigned tp = (sm & 0x80808080u) | ((e >> 1) & 0x7f7f7f7fu), md = (sm & 0x7f7f7f7fu) | ((e << 7) & 0x80808080u);
        const unsigned h01 = PERM(tp, md, 0x05010400u), h23 = PERM(tp, md, 0x07030602u), l01 = Z[q >> 1][2 * (q & 1)], l23 = Z[q >> 1][2 * (q & 1) + 1];
        st4(C.vo, (u4){PERM(h01, l01, 0x05040100u), PERM(h01, l01, 0x07060302u), PERM(h23, l23, 0x05040100u), PERM(h23, l23, 0x07060302u)},
            ob + (size_t)((q >> 2) * 16 + (q & 3)) * (NG * 4));
    }
}
#define LDS_LOAD(p) __hip_atomic_load((p), __ATOMIC_RELAXED, __HIP_MEMORY_SCOPE_WORKGROUP)
#define LDS_STORE(p, v) __hip_atomic_store((p), (v), __ATOMIC_RELAXED, __HIP_MEMORY_SCOPE_WORKGROUP)
constexpr unsigned IT_DEAD = 0x80000000u;
constexpr int PF = 2;
__device__ __forceinline__ void load_items(const Com& C, const Params& P, gu32* tick, unsigned tbase, unsigned cur, unsigned nxt) {
    const unsigned want = C.call & 0x1fffu, ring = (unsigned)(size_t)C.ring;
    unsigned k = 0u, seen = 0u; bool pend = false, dead = false;
    int nit = 0; (void)nit;
    while (cur < NI) {
        const bool in = cur < NI_IN;
        const char* sb;
        if (in) { bool isA; int panel, j; image_of(cur >> 2, C.rank, isA, panel, j);
                  sb = C.src_ws + (C.slot + O_X + (unsigned)((((isA ? C.src : C.rank) * 16 + panel) * 8 + j) * TILE_B) + (cur & 3) * 8192); }
        else { int tile, u; unit_of(cur - NI_IN, C.late, tile, u); sb = C.src_ws + (C.slot + O_OUT + (unsigned)((C.rank * 32 + tile) * 32 + u) * UNIT_B); }
        if (!dead && (seen >> 19) != want) {
            if (pend) { WAIT_V(0); if (C.lane == 0) LDS_STORE(C.word, k); pend = false; }
            dead = word_wait(C.ctl, item_word(C, cur), 19, want, seen, C.call, C.lane);
        }
        const bool skip = dead && in;
        while (k >= (unsigned)RFL(LDS_LOAD(C.word + 1)) + 4u) __builtin_amdgcn_s_sleep(1);
        if (!skip && C.lane == 0) { C.word[4 + 2 * (k & 3)] = dead ? cur | IT_DEAD : cur; C.word[5 + 2 * (k & 3)] = seen; }
        unsigned tv = 0, fv = 0; u64 ex;
        asm volatile("s_mov_b64 %1, exec\n\ts_mov_b64 exec, 1\n\tglobal_atomic_add %0, %2, %3, off sc0\n\ts_mov_b64 exec, %1"
                     : "+v"(tv), "=&s"(ex) : "v"(tick), "v"(nxt < NI ? 1u : 0u) : "memory");
        if (!dead) {
            const unsigned nw = (unsigned)item_word(C, nxt) * 4u;
            const unsigned lo = ring + (k & 3) * 8192, list = ((seen >> 16) & 3u) * 4u + 4u;
            const bool raw = seen & H_RAW;
            static_assert(PF <= 2, "the header load must be older than load 6 - PF");
#pragma unroll
            for (int r = 0; r < 6; ++r) {
                if (r < PF || (r > 3 && r <= 3 + PF)) wait_v<PF>(); else wait_v<PF - 1>();
                if (r == PF && pend && C.lane == 0) LDS_STORE(C.word, k);
                dma(C.vl, sb + r * 1024, lo + r * 1024);
                if (r == 3) asm volatile("global_load_dword %0, %1, %2 sc0 sc1" : "+v"(fv) : "v"(nw), "s"(C.ctl) : "memory");
            }
            wait_v<PF - 1>();
            dma_part(C.vl, sb + 6144, lo + 6144, in || raw ? 64u : 48u + list);
            if (in || raw) { wait_v<PF - 1>(); dma_part(C.vl, sb + 7168, lo + 7168, raw ? 64u : list); }
            pend = true; ++k;
        } else {
            WAIT_V(0);
            if (!skip) ++k;
            if (C.lane == 0) LDS_STORE(C.word, k);
            pend = false;
        }
        asm volatile("" : "+v"(tv), "+v"(fv));
        const unsigned nn = nxt < NI ? (unsigned)RFL(tv) - tbase : NI;
        cur = nxt; nxt = nn; seen = RFL(fv);
    }
    WAIT_V(0);
    while (k >= (unsigned)RFL(LDS_LOAD(C.word + 1)) + 4u) __builtin_amdgcn_s_sleep(1);
    if (C.lane == 0) { C.word[4 + 2 * (k & 3)] = ~0u; CFENCE; LDS_STORE(C.word, k + 1u); }
}
__device__ __forceinline__ void unpack_items(const Com& C) {
    for (unsigned k = 0u;; ++k) {
        while ((unsigned)RFL(LDS_LOAD(C.word)) <= k) __builtin_amdgcn_s_sleep(1);
        CFENCE;
        const unsigned n = RFL(C.word[4 + 2 * (k & 3)]), hd = RFL(C.word[5 + 2 * (k & 3)]);
        if (n == ~0u) break;
        LAS unsigned char* z = C.ring + (k & 3) * 8192;
        if (n & IT_DEAD) {
            const char* ob = unit_rows(C, n & ~IT_DEAD); const unsigned qn = 0x7fc00000u;
#pragma unroll
            for (int q = 0; q < 8; ++q) st4(C.vo, (u4){qn, qn, qn, qn}, ob + (size_t)((q >> 2) * 16 + (q & 3)) * (NG * 4));
        } else if (n < NI_IN) put_quarter(C, n, hd, z);
        else put_unit(C, n, hd, z);
        WAIT_L(0);
        if (C.lane == 0) LDS_STORE(C.word + 1, k + 1u);
    }
}

__global__ __launch_bounds__(NTHR, 2) void gemm_rs_kernel(const Params P) {
    __shared__ __attribute__((aligned(16))) unsigned char smem[LDS_BYTES];
    LAS unsigned char* lds = (LAS unsigned char*)smem;
    LAS unsigned* lctl = (LAS unsigned*)(lds + LDS_CTL);
    const int tid = threadIdx.x, lane = tid & 63, wid = RFL(tid >> 6);
    const int bid = blockIdx.x, rank = P.rank;
    char* myws = P.ws[rank];
    asm volatile("" : "+s"(myws));
    gu32* ctl = (gu32*)myws;
    STAMP(0);
    const unsigned vl = (unsigned)lane * 16u;
    const int gid = bid - NCOM;
    const char* xb = (const char*)P.x; const char* wb = (const char*)P.w;
    const unsigned vxe = (unsigned)((lane >> 3) * (KS * 4) + (((lane & 7) ^ (lane >> 4)) * 32));
    gu32* callp = ctl + W_CALL + bid * 32;
    const unsigned callv = __hip_atomic_load(callp, __ATOMIC_RELAXED, __HIP_MEMORY_SCOPE_AGENT);
#define TAKE_CALL() const unsigned call = RFL(callv) + 1u; \
    if (tid < 64) lctl[tid] = 0u; \
    asm volatile("s_waitcnt lgkmcnt(0)" :: "s"(call) : "memory");        \
    BAR; CFENCE; \
    if (tid == 0) __hip_atomic_store(callp, call, __ATOMIC_RELAXED, __HIP_MEMORY_SCOPE_AGENT); \
    const unsigned slot = CTL_B + (call & 1u) * SLOT_B

    if (gid < 0) {
        const int peer = (rank + 1 + (bid >> 3)) & 7, q0 = (bid & 7) * 4 + (wid & 3), panel0 = (2 * peer + 2 + (bid & 7)) & 15;
        f4 L0[8], L1[8];
        if (wid >= 4) { job_loads(L0, xb, wb, true, 0, panel0, 0, 2 * (wid & 3), vxe, vl); job_loads(L1, xb, wb, true, 0, panel0, 0, 2 * (wid & 3) + 1, vxe, vl); }
        TAKE_CALL();
        Com C;
        C.ctl = ctl; C.me_ws = myws; C.rank = rank; C.src = peer; C.src_ws = P.ws[(unsigned)peer & 7u]; C.out = (char*)P.out[rank];
        C.ring = lds + (wid & 3) * 32768; C.word = lctl + (wid & 3) * 16; C.slot = slot; C.call = call; C.vl = vl; C.lane = lane;
        C.vo = (unsigned)((lane >> 4) * 4 * (NG * 4) + (lane & 15) * 16);
        if (wid >= 4) {
            u4 D[8];
            job_pack(L0, *(u4 (*)[4])&D[0], true); job_pack(L1, *(u4 (*)[4])&D[4], true);
            const char* db = C.src_ws + (slot + O_A + (unsigned)((panel0 * NKT + 8 + ((rank - peer - 1) & 7)) * TILE_B + (wid & 3) * 8192));
#pragma unroll
            for (int p = 0; p < 8; ++p) { WAIT_V(1); st4(vl, D[p], db + p * 1024); }
            WAIT_V(0);
            if (lane == 0) __hip_atomic_fetch_add((gu32*)C.src_ws + W_LAND + (panel0 * 8 + rank) * 32, 1u, __ATOMIC_RELAXED, __HIP_MEMORY_SCOPE_SYSTEM);
            if (wid == 4) STAMPW(1);
        }
        C.late = late_order((rank - peer - 1) & 7);
        if (wid >= 4) {
            unpack_items(C);
            if (wid == 4) STAMPW(20);
        } else {
            const unsigned tbase = (call - 1u) * (NI - 96 + 32) - 96;
            load_items(C, P, ctl + W_TICK + (bid >> 3) * 32, tbase, 32u + q0, 64u + q0);
        }
        return;
    }

    f4 L0[8], L1[8];
    const int qlate = 1600 + (gid - 168) * 8 + wid;
    const int q0 = gid < 196 ? gid * 8 + wid : qlate;
    {
        bool isA; int dest, panel, j; conv_of(q0 >> 2, rank, isA, dest, panel, j);
        if (STAGGER) for (int i = gid >> 3; i > 0; --i) __builtin_amdgcn_s_sleep(STAGGER * 4);
        CFENCE;
        job_loads(L0, xb, wb, isA, dest, panel, j, 2 * (q0 & 3), vxe, vl); job_loads(L1, xb, wb, isA, dest, panel, j, 2 * (q0 & 3) + 1, vxe, vl);
    }
    TAKE_CALL();
    gu32* tmo = ctl + W_TMO;
    {
        gu32* wsl = (gu32*)P.ws[lane & 7];
        asm volatile("" : "+v"(wsl));
        LAS unsigned char* z = lds + wid * 16384;
        for (int qn = q0, it = 0; qn < 4640; ++it) {
            bool isA; int dest, panel, j; conv_of(qn >> 2, rank, isA, dest, panel, j);
            const int qx = gid < 168 ? 4640 : qn < 1600 ? qlate : qn + 256;
            const int q = qn & 3;
            u4 D[8];
            job_pack(L0, *(u4 (*)[4])&D[0], isA); job_pack(L1, *(u4 (*)[4])&D[4], isA);
            const bool own = isA || dest == rank, peers = isA || dest != rank;
            const char* db = myws + (slot + (isA ? O_A : O_B) + (unsigned)((panel * NKT + j) * TILE_B));
            if (own) {
#pragma unroll
                for (int p = 0; p < 8; ++p) st4(vl, D[p], db + piece_off(isA, q, p));
            }
            unsigned hd = call << 19;
            if (peers) {
                const char* xp = myws + (slot + O_X + (unsigned)((((isA ? rank : dest) * 16 + panel) * 8 + j) * TILE_B) + q * 8192);
                u4 lo[4], h[4]; unsigned se = 0u;
#pragma unroll
                for (int i = 0; i < 16; ++i) {
                    const unsigned d0 = D[i >> 1][2 * (i & 1)], d1 = D[i >> 1][2 * (i & 1) + 1];
                    lo[i >> 2][i & 3] = PERM(d1, d0, 0x06040200u); h[i >> 2][i & 3] = PERM(d1, d0, 0x07050301u);
                    se = __builtin_amdgcn_sad_u8(h[i >> 2][i & 3] & 0x7c7c7c7cu, 0u, se);
                }
#pragma unroll
                for (int r = 0; r < 4; ++r) *(LAS u4*)(z + r * 1024 + lane * 16) = h[r];
#pragma unroll
                for (int o = 32; o >= 1; o >>= 1) se += (unsigned)__shfl_xor((int)se, o);
                const unsigned base = min(max(((unsigned)RFL(se) + 47500u) >> 14, 6u), 31u);
                const unsigned bq = ((base << 2) | 0x80u) * 0x01010101u;
                u4 f[4]; unsigned m0 = 0u, m1 = 0u;
#pragma unroll
                for (int i = 0; i < 16; ++i) {
                    const unsigned hh = h[i >> 2][i & 3], x = (bq - (hh & 0x7c7c7c7cu)) ^ 0x80808080u;
                    const unsigned es = (((x & 0x7f7f7f7fu) + 0x64646464u) | x) & 0x80808080u;
                    f[i >> 2][i & 3] = ((hh >> 2) & 0x20202020u) | ((x | bytemask(es)) & 0x1c1c1c1cu) | (hh & 0x03030303u);
                    if (i < 8) m0 |= es >> (7 - i); else m1 |= es >> (15 - i);
                }
                CFENCE;
                u64 m = ((u64)m1 << 32) | m0; unsigned cnt = 0u;
                for (;;) {
                    const u64 act = __ballot(m != 0ull);
                    if (act == 0ull || cnt > 256u) break;
                    if (m != 0ull) {
                        const unsigned pos = cnt + __builtin_amdgcn_mbcnt_hi((unsigned)(act >> 32), __builtin_amdgcn_mbcnt_lo((unsigned)act, 0u));
                        z[4096 + pos] = (unsigned char)((z[h_byte(__builtin_ctzll(m), lane)] >> 2) & 31u);
                        m &= m - 1ull;
                    }
                    cnt += (unsigned)__popcll(act);
                }
                CFENCE;
                if (cnt <= 256u) {
                    hd |= (((cnt - (cnt != 0u)) >> 6) << 16) | (base << 8);
#pragma unroll
                    for (int r = 0; r < 4; ++r) st4(vl, lo[r], xp + r * 1024);
#pragma unroll
                    for (int r = 0; r < 3; ++r) st4(vl, f[r] | ((f[3] << (6 - 2 * r)) & 0xc0c0c0c0u), xp + 4096 + r * 1024);
                    st1((unsigned)lane * 4u, *(const LAS unsigned*)(z + 4096 + lane * 4), xp + 7168);
                } else {
                    hd |= H_RAW;
#pragma unroll
                    for (int p = 0; p < 8; ++p) st4(vl, D[p], xp + p * 1024);
                }
            }
            if (qx < 4640) {
                bool nA; int nd, np, nj; conv_of(qx >> 2, rank, nA, nd, np, nj);
                job_loads(L0, xb, wb, nA, nd, np, nj, 2 * (qx & 3), vxe, vl); job_loads(L1, xb, wb, nA, nd, np, nj, 2 * (qx & 3) + 1, vxe, vl);
                wait_v<16>();
            } else WAIT_V(0);
            const int im = it * 2 + (wid >> 2);
            unsigned old = 0u;
            if (lane == 0) { lctl[L_HD + im * 4 + q] = hd; old = __hip_atomic_fetch_add(lctl + L_CNT + im, 1u, __ATOMIC_RELAXED, __HIP_MEMORY_SCOPE_WORKGROUP); }
            if ((unsigned)RFL(old) == 3u && lane < NR && (isA || lane == dest)) {
                CFENCE;
                if (lane == rank) __hip_atomic_fetch_add(ctl + W_LAND + (((isA ? 0 : 16) + panel) * 8 + rank) * 32 + j, 4u, __ATOMIC_RELAXED, __HIP_MEMORY_SCOPE_AGENT);
                else { const u4 h4 = *(const LAS u4*)(lctl + L_HD + im * 4);
                       asm volatile("global_store_dwordx4 %0, %1, off sc0 sc1\n\ts_nop 1" :: "v"(wsl + W_AVAIL + ((((call & 1) * 8 + rank) * 2 + (isA ? 0 : 1)) * 16 + panel) * 32 + j * 4), "v"(h4) : "memory"); }
            }
            if (qn == q0) STAMP(10);
            qn = qx;
        }
        WAIT_V(0); WAIT_L(0);
        STAMP(16);
    }

    const int wr = wid >> 2, wc = wid & 3;
    const int xcd = gid & 7, idx = gid >> 3, xh = xcd & 1, xq = xcd >> 1;
    for (int round = 0; round < 2; ++round) {
        int pm, pn;
        if (round == 0 && idx < 21) { const int b = idx / 7; pn = 3 * xq + b; pm = 2 * ((rank + 1 + idx - b * 7) & 7) + xh; }
        else if (round == 0 || idx < 3) { pn = 12 + xq; pm = 2 * ((rank + 1 + late_peer(xcd, round == 0 ? idx - 21 : idx + 4)) & 7) + xh; }
        else if (idx < 7) { pm = 2 * rank + xh; pn = 4 * xq + idx - 3; }
        else break;
        pm = RFL(pm); pn = RFL(pn);
        int tz = threadIdx.x; asm volatile("" : "+v"(tz));
        const int ln = tz & 63, fr = ln & 15, fq = ln >> 4;
        gu32* fa = ctl + W_LAND + (pm * 8 + (ln & 7)) * 32 + (ln >> 3);
        gu32* fb = ctl + W_LAND + ((16 + pn) * 8 + (ln & 7)) * 32 + (ln >> 3);
        const u64 own = 0x0101010101010101ull << rank;
        u64 ra = 0, rb = 0;
        bool bad = false;
#define STEP_WAIT(need, last) do { if (!bad && ((last) || (ra & rb & (need)) != (need))) bad = wave_wait(fa, fb, (need), ra, rb, call * 4u, tmo, call, ln); } while (0)
        if (wid == 0) { STEP_WAIT(own, false); WAIT_V(0); if (round == 0) STAMPW(1); }
        CFENCE; BAR; CFENCE;

        const char* cA = myws + (slot + O_A + (unsigned)(pm * NKT) * TILE_B);
        const char* cB = myws + (slot + O_B + (unsigned)(pn * NKT) * TILE_B);
        const unsigned vst = (unsigned)tz * 16u;
        const unsigned ldsw = (unsigned)wid * 1024u;
        const int a0 = (wr * 64 + fr) * 128 + ((fq ^ (fr >> 1)) * 16), a1 = a0 ^ 64;
        const int b0 = fq * 2048 + (wc * 16 + fr) * 16;
#define SA(b, h) (((b) * 2 + (h)) * HALF_B)
#define SB(b, h) ((4 + (b) * 2 + (h)) * HALF_B)
#define STAGE(bufoff, gbase) do { _Pragma("unroll") for (int _i = 0; _i < 2; ++_i) \
        __builtin_amdgcn_global_load_lds((const unsigned*)((gbase) + vst + _i * 8192), (LAS unsigned*)(lds + (bufoff) + ldsw + _i * 8192), 16, 0, 17); } while (0)
#define LDA(dst, b, h) do { _Pragma("unroll") for (int m = 0; m < 4; ++m) { dst[m][0] = *(const LAS h8*)(lds + SA(b, h) + a0 + m * 2048); dst[m][1] = *(const LAS h8*)(lds + SA(b, h) + a1 + m * 2048); } } while (0)
#define LDB(dst, b, h) do { _Pragma("unroll") for (int n = 0; n < 2; ++n) _Pragma("unroll") for (int k = 0; k < 2; ++k) dst[n][k] = *(const LAS h8*)(lds + SB(b, h) + b0 + k * 8192 + n * 1024); } while (0)
#define MMA(ai, bj, At, Bt) do { __builtin_amdgcn_s_setprio(1); _Pragma("unroll") for (int m = 0; m < 4; ++m) _Pragma("unroll") for (int n = 0; n < 2; ++n) _Pragma("unroll") for (int k = 0; k < 2; ++k) \
        acc[ai][bj][m][n] = __builtin_amdgcn_mfma_f32_16x16x32_f16(At[m][k], Bt[n][k], acc[ai][bj][m][n], 0, 0, 0); __builtin_amdgcn_s_setprio(0); } while (0)
        f4 acc[2][2][4][2];
#pragma unroll
        for (int a = 0; a < 2; ++a)
#pragma unroll
            for (int b = 0; b < 2; ++b)
#pragma unroll
                for (int m = 0; m < 4; ++m)
#pragma unroll
                    for (int n = 0; n < 2; ++n) acc[a][b][m][n] = (f4){0.f, 0.f, 0.f, 0.f};
        h8 At[4][2], B0[2][2], B1[2][2];
        STAGE(SB(0, 0), cB); STAGE(SA(0, 0), cA); STAGE(SB(0, 1), cB + HALF_B); STAGE(SA(0, 1), cA + HALF_B);
        if (wr == 1) BAR;
        WAIT_V(4); BAR;
        STAGE(SB(1, 0), cB + TILE_B); STAGE(SA(1, 0), cA + TILE_B); STAGE(SB(1, 1), cB + TILE_B + HALF_B);
        WAIT_V(6); BAR;
        int jn = 0, bnd = 8;
        for (int t = 0; t < NKT; t += 2) {
            const int t2 = (t + 2) & (NKT - 1);
            const char* a1p = cA + (size_t)(t + 1) * TILE_B;
            const char* a2 = cA + (size_t)t2 * TILE_B; const char* b2 = cB + (size_t)t2 * TILE_B;
            const char* a3 = a2 + TILE_B; const char* b3 = b2 + TILE_B;
            if (t + 3 >= bnd && jn < 8) {
                if (wid == 0) {
                    STEP_WAIT((0xFFull << (8 * jn)) & ~own, jn == 7);
                    if (jn == 7 && ln == 0) lctl[L_BAD] = bad ? 1u : 0u;
                    if (round == 0) STAMPW(2 + jn);
                }
                CFENCE; BAR; CFENCE;
                ++jn; bnd += 7;
            }
            LDB(B0, 0, 0); SCHED; LDA(At, 0, 0); STAGE(SA(1, 1), a1p + HALF_B);
            WAIT_L(8); BAR; WAIT_L(0); MMA(0, 0, At, B0); BAR; SCHED;
            LDB(B1, 0, 1); STAGE(SB(0, 0), b2);
            BAR; WAIT_L(0); MMA(0, 1, At, B1); BAR;
            LDA(At, 0, 1); STAGE(SA(0, 0), a2);
            BAR; WAIT_L(0); MMA(1, 0, At, B0); BAR; SCHED;
            STAGE(SB(0, 1), b2 + HALF_B);
            WAIT_V(6); BAR; MMA(1, 1, At, B1); BAR;
            LDB(B0, 1, 0); SCHED; LDA(At, 1, 0); STAGE(SA(0, 1), a2 + HALF_B);
            WAIT_L(8); BAR; WAIT_L(0); MMA(0, 0, At, B0); BAR; SCHED;
            LDB(B1, 1, 1); STAGE(SB(1, 0), b3);
            BAR; WAIT_L(0); MMA(0, 1, At, B1); BAR;
            LDA(At, 1, 1); STAGE(SA(1, 0), a3);
            BAR; WAIT_L(0); MMA(1, 0, At, B0); BAR; SCHED;
            STAGE(SB(1, 1), b3 + HALF_B);
            WAIT_V(6); BAR; MMA(1, 1, At, B1); BAR;
        }
        WAIT_V(0);
        if (wr == 0) BAR;
        CFENCE; BAR; CFENCE;
        STAMP(round == 0 ? 17 : 11);

        const unsigned dest = ((unsigned)pm >> 1) & 7u;
        int ty = threadIdx.x; asm volatile("" : "+v"(ty));
        const int l2 = ty & 63;
        const unsigned vo = (unsigned)((l2 >> 4) * 4 * (NG * 4) + (l2 & 15) * 16);
        const bool poison = lctl[L_BAD] != 0u;
        const float qnan = __builtin_nanf("");
#define ROW4(ai, m, r) (poison ? (f4){qnan, qnan, qnan, qnan} : (f4){gelu(acc[ai][0][m][0][r]), gelu(acc[ai][0][m][1][r]), gelu(acc[ai][1][m][0][r]), gelu(acc[ai][1][m][1][r])})
        if (dest == (unsigned)rank) {
            const char* ob = (const char*)P.out[rank] + ((size_t)((pm & 1) * 256 + wr * 64) * NG + rank * NS + pn * 256 + wc * 64) * 4;
#pragma unroll
            for (int ai = 0; ai < 2; ++ai)
#pragma unroll
                for (int m = 0; m < 4; ++m)
#pragma unroll
                    for (int r = 0; r < 4; ++r) { const f4 v = ROW4(ai, m, r); st4(vo, __builtin_bit_cast(u4, v), ob + (size_t)(ai * 128 + m * 16 + r) * (NG * 4)); }
        } else {
            const int tile = (pm & 1) * 16 + pn;
            const char* zb = myws + (slot + O_OUT + (unsigned)((dest * 32 + tile) * 32 + wid) * UNIT_B);
            gu32* uf = (gu32*)P.ws[dest] + W_UNIT + (rank * 32 + tile) * 32;
            LAS unsigned char* z = lds + wid * 16384;
#pragma unroll
            for (int g = 0; g < 4; ++g) {
                const int ai = g >> 1;
                unsigned lo[16], sm[8], mp = 0u, mn = 0u; u4 E[2];
#pragma unroll
                for (int q = 0; q < 8; ++q) {
                    const int m = (g & 1) * 2 + (q >> 2), r = q & 3;
                    const u4 u = __builtin_bit_cast(u4, ROW4(ai, m, r));
                    lo[2 * q] = PERM(u.y, u.x, 0x05040100u); lo[2 * q + 1] = PERM(u.w, u.z, 0x05040100u);
                    const unsigned h01 = PERM(u.y, u.x, 0x07060302u), h23 = PERM(u.w, u.z, 0x07060302u);
                    sm[q] = (PERM(h23, h01, 0x06040200u) & 0x7f7f7f7fu) | (PERM(h23, h01, 0x07050301u) & 0x80808080u);
                    E[q >> 2][q & 3] = PERM(h23 << 1, h01 << 1, 0x07050301u);
                    mp = pmax_i16(mp, pmax_i16(h01, h23)); mn = pmax_u16(mn, pmax_u16(h01, h23));
                }
                *(LAS u4*)(z + l2 * 16) = E[0]; *(LAS u4*)(z + 1024 + l2 * 16) = E[1];
                const unsigned nm = max(mn & 0xffffu, mn >> 16);
                unsigned bb = (max((mp >> 7) & 0xffu, mp >> 23) << 16) | ((nm & 0x8000u) ? (nm >> 7) & 0xffu : 0u);
#pragma unroll
                for (int o = 32; o >= 1; o >>= 1) bb = pmax_u16(bb, (unsigned)__shfl_xor((int)bb, o));
                bb = RFL(bb);
                const unsigned bp = bb >> 16, bn = bb & 0xffu, bp4 = bp * 0x01010101u, bn4 = bn * 0x01010101u;
                unsigned c[8], m = 0u, cnt = 0u, hd = call << 19;
#pragma unroll
                for (int q = 0; q < 8; ++q) {
                    const unsigned mk = bytemask(sm[q] & 0x80808080u), d = ((bn4 & mk) | (bp4 & ~mk)) - E[q >> 2][q & 3];
                    const unsigned es = (((d & 0x7f7f7f7fu) + 0x79797979u) | d) & 0x80808080u;
                    c[q] = (d | bytemask(es)) & 0x07070707u;
                    m |= es >> (7 - q);
                }
                CFENCE;
                for (;;) {
                    const u64 act = __ballot(m != 0u);
                    if (act == 0ull || cnt > 256u) break;
                    if (m != 0u) {
                        const unsigned pos = cnt + __builtin_amdgcn_mbcnt_hi((unsigned)(act >> 32), __builtin_amdgcn_mbcnt_lo((unsigned)act, 0u));
                        z[2048 + pos] = z[h_byte(__builtin_ctz(m), l2)];
                        m &= m - 1u;
                    }
                    cnt += (unsigned)__popcll(act);
                }
                CFENCE;
                const char* zp = zb + g * 8 * UNIT_B;
                if (cnt <= 256u) {
                    hd |= (((cnt - (cnt != 0u)) >> 6) << 16) | (bp << 8) | bn;
#pragma unroll
                    for (int i = 0; i < 4; ++i) st4((unsigned)l2 * 16u, (u4){lo[4 * i], lo[4 * i + 1], lo[4 * i + 2], lo[4 * i + 3]}, zp + i * 1024);
                    st4((unsigned)l2 * 16u, (u4){sm[0], sm[1], sm[2], sm[3]}, zp + 4096); st4((unsigned)l2 * 16u, (u4){sm[4], sm[5], sm[6], sm[7]}, zp + 5120);
                    st1((unsigned)l2 * 4u, c[0] | (c[1] << 3) | (c[2] << 6 & 0xc0c0c0c0u), zp + 6144);
                    st1((unsigned)l2 * 4u, (c[2] >> 2 & 0x01010101u) | (c[3] << 1) | (c[4] << 4) | (c[5] << 7 & 0x80808080u), zp + 6400);
                    st1((unsigned)l2 * 4u, (c[5] >> 1 & 0x03030303u) | (c[6] << 2) | (c[7] << 5), zp + 6656);
                    st1((unsigned)l2 * 4u, *(const LAS unsigned*)(z + 2048 + l2 * 4), zp + 6912);
                } else {
                    hd |= H_RAW;
#pragma unroll
                    for (int q = 0; q < 8; ++q) {
                        const unsigned e = E[q >> 2][q & 3], tp = (sm[q] & 0x80808080u) | ((e >> 1) & 0x7f7f7f7fu), md = (sm[q] & 0x7f7f7f7fu) | ((e << 7) & 0x80808080u);
                        const unsigned h01 = PERM(tp, md, 0x05010400u), h23 = PERM(tp, md, 0x07030602u);
                        st4((unsigned)l2 * 16u, (u4){PERM(h01, lo[2 * q], 0x05040100u), PERM(h01, lo[2 * q], 0x07060302u), PERM(h23, lo[2 * q + 1], 0x05040100u), PERM(h23, lo[2 * q + 1], 0x07060302u)}, zp + q * 1024);
                    }
                }
                WAIT_V(0);
                const int ig = round * 4 + g;
                unsigned old = 0u;
                if (l2 == 0) { lctl[L_HD + 104 + ig * 8 + wid] = hd; old = __hip_atomic_fetch_add(lctl + L_CNT + 26 + ig, 1u, __ATOMIC_RELAXED, __HIP_MEMORY_SCOPE_WORKGROUP); }
                if ((unsigned)RFL(old) == 7u && l2 < 8) { CFENCE; __hip_atomic_store(uf + g * 8 + l2, lctl[L_HD + 104 + ig * 8 + l2], __ATOMIC_RELAXED, __HIP_MEMORY_SCOPE_SYSTEM); }
            }
        }
        STAMP(round == 0 ? 18 : 12);
        WAIT_V(0); WAIT_L(0); BAR; CFENCE;
    }
    STAMP(20);
}

}

extern "C" void kernel_launch(int rank, int nranks, ncclComm_t comm, hipStream_t stream,
                              void* const* d_in, const int* in_sizes, int n_in,
                              void* d_out, int out_size, const int* in_dtypes, int out_dtype,
                              void* const* d_out_all, void* const* d_ws_all, size_t ws_bytes) {
    (void)comm; (void)in_dtypes; (void)out_dtype; (void)d_out;
    if (nranks != NR || (unsigned)rank >= (unsigned)NR || n_in < 2 || ws_bytes < WS_NEED || in_sizes[0] != MG * KS || in_sizes[1] != KS * NG || out_size != MS * NG) {
        fprintf(stderr, "kernel_launch: unexpected problem shape\n");
        return;
    }
    Params P;
    P.x = (const float*)d_in[0]; P.w = (const float*)d_in[1];
    for (int p = 0; p < NR; ++p) { P.out[p] = (float*)d_out_all[p]; P.ws[p] = (char*)d_ws_all[p]; }
    P.rank = rank; P.pad = 0;
    hipLaunchKernelGGL(gemm_rs_kernel, dim3(NWG), dim3(NTHR), 0, stream, P);
    HIP_CHECK(hipGetLastError());
}
```

```cpp
#include <hip/hip_runtime.h>
#include <rccl/rccl.h>
#include <cstdio>

#define HIP_CHECK(expr) do { hipError_t _e = (expr); if (_e != hipSuccess) { \
    fprintf(stderr, "HIP error %s at %s:%d\n", hipGetErrorString(_e), __FILE__, __LINE__); \
    return; } } while (0)

namespace {

typedef _Float16 h8 __attribute__((ext_vector_type(8)));
typedef _Float16 h2 __attribute__((ext_vector_type(2)));
typedef float f4 __attribute__((ext_vector_type(4)));
typedef unsigned u4 __attribute__((ext_vector_type(4)));
typedef short s2v __attribute__((ext_vector_type(2)));
typedef unsigned short us2v __attribute__((ext_vector_type(2)));
typedef unsigned long long u64;
typedef __attribute__((address_space(1))) unsigned gu32;
typedef __attribute__((address_space(1))) f4 gf4;
#define LAS __attribute__((address_space(3)))

constexpr int NR = 8;
constexpr int MG = 4096, KG = 4096, NG = 32768;
constexpr int KS = KG / NR;
constexpr int NS = NG / NR;
constexpr int MS = MG / NR;
constexpr int NWG = 256, NTHR = 512;
constexpr int NCOM = 56;
constexpr int NX = 2, PUSHX = 1, POLL_SLEEP = 16;
constexpr int TILE_B = 256 * 64 * 2;
constexpr int HALF_B = TILE_B / 2;
constexpr int NKT = KG / 64;
constexpr unsigned NI_IN = 1024, NI = 2048;
constexpr unsigned OP_B = 16u * NKT * TILE_B;
constexpr unsigned UNIT_B = 8192;
constexpr unsigned CTL_B = 1u << 20;
constexpr unsigned O_A = 0, O_B = OP_B, O_X = 2 * OP_B, O_OUT = 3 * OP_B, O_LIN = O_OUT + NR * 32 * 32 * UNIT_B,
                   SLOT_B = O_LIN + (1u << 20);
constexpr size_t WS_NEED = (size_t)CTL_B + 2 * (size_t)SLOT_B;
constexpr int W_TMO = 0;
constexpr int W_LAND = 32;
constexpr int W_AVAIL = W_LAND + 256 * 32;
constexpr int W_UNIT = W_AVAIL + 512 * 32;
constexpr unsigned H_RAW = 1u << 18;
constexpr int W_TICK = W_UNIT + 256 * 32;
constexpr int W_CALL = W_TICK + 8 * 32;
constexpr int W_STAMP = 131072;
static_assert(W_CALL + NWG * 32 <= W_STAMP, "control layout");
constexpr u64 DEADLINE = 40000000ull;

constexpr int LDS_CTL = 8 * HALF_B;
constexpr int LDS_LIN = LDS_CTL + 1024, LDS_BYTES = LDS_LIN + 2048;
constexpr int L_BAD = 0, L_CNT = 2, L_HD = 64;

struct Params {
    const float* x; const float* w; char* myws; float* out;
    int rank; int pad;
    char* ws[NR];
    char* peer_ws[NR];
};

#define WAIT_V(n) asm volatile("s_waitcnt vmcnt(" #n ")" ::: "memory")
#define WAIT_L(n) asm volatile("s_waitcnt lgkmcnt(" #n ")" ::: "memory")
#define BAR __builtin_amdgcn_s_barrier()
#define SCHED __builtin_amdgcn_sched_barrier(0)
#define CFENCE asm volatile("" ::: "memory")
#define RFL(x) __builtin_amdgcn_readfirstlane(x)
#define PERM(hi, lo, sel) __builtin_amdgcn_perm((hi), (lo), (sel))
template <int N> __device__ __forceinline__ void wait_v() { asm volatile("s_waitcnt vmcnt(%0)" :: "i"(N) : "memory"); }

__device__ __forceinline__ void st4(unsigned voff, u4 v, const void* sbase) {
    asm volatile("global_store_dwordx4 %0, %1, %2 sc0 sc1\n\ts_nop 1" :: "v"(voff), "v"(v), "s"(sbase) : "memory");
}
__device__ __forceinline__ void st1(unsigned voff, unsigned v, const void* sbase) {
    asm volatile("global_store_dword %0, %1, %2 sc0 sc1" :: "v"(voff), "v"(v), "s"(sbase) : "memory");
}
__device__ __forceinline__ void dma(unsigned voff, const void* sbase, unsigned ldsoff) {
    unsigned keep;
    asm volatile("s_mov_b32 %0, m0\n\ts_mov_b32 m0, %3\n\ts_nop 0\n\tglobal_load_lds_dwordx4 %1, %2 sc0 sc1\n\ts_mov_b32 m0, %0"
                 : "=&s"(keep) : "v"(voff), "s"(sbase), "s"(ldsoff) : "memory");
}
__device__ __forceinline__ void dma_part(unsigned voff, const void* sbase, unsigned ldsoff, unsigned lanes) {
    unsigned keep; u64 ex;
    asm volatile("s_mov_b32 %0, m0\n\ts_mov_b32 m0, %4\n\ts_mov_b64 %1, exec\n\ts_mov_b64 exec, %5\n\tglobal_load_lds_dwordx4 %2, %3 sc0 sc1\n\ts_mov_b64 exec, %1\n\ts_mov_b32 m0, %0"
                 : "=&s"(keep), "=&s"(ex) : "v"(voff), "s"(sbase), "s"(ldsoff), "s"(~0ull >> (64u - lanes)) : "memory");
}
__device__ __forceinline__ unsigned pmax_i16(unsigned a, unsigned b) { return __builtin_bit_cast(unsigned, __builtin_elementwise_max(__builtin_bit_cast(s2v, a), __builtin_bit_cast(s2v, b))); }
__device__ __forceinline__ unsigned pmax_u16(unsigned a, unsigned b) { return __builtin_bit_cast(unsigned, __builtin_elementwise_max(__builtin_bit_cast(us2v, a), __builtin_bit_cast(us2v, b))); }
__device__ __forceinline__ unsigned bytemask(unsigned s) { return (s - (s >> 7)) | s; }
__device__ __forceinline__ unsigned pk(float a, float b) {
    h2 r; r.x = (_Float16)a; r.y = (_Float16)b; return __builtin_bit_cast(unsigned, r);
}
__device__ __forceinline__ float gelu(float y) {
    const float k1 = -2.0f * 0.7978845608028654f * 1.4426950408889634f, k2 = k1 * 0.044715f;
    const float e = __builtin_amdgcn_exp2f(y * (k1 + k2 * y * y));
    return y * __builtin_amdgcn_rcpf(1.0f + e);
}

#define STAMPW(i) do {} while (0)
#define STAMP(i) do { if (wid == 0) STAMPW(i); } while (0)

__device__ __forceinline__ bool wave_wait(gu32* pa, gu32* pb, u64 need, u64& ra, u64& rb, unsigned target, gu32* tmo, unsigned call, int lane) {
    const u64 t0 = __builtin_amdgcn_s_memrealtime();
    for (;;) {
        const unsigned va = __hip_atomic_load(pa, __ATOMIC_RELAXED, __HIP_MEMORY_SCOPE_SYSTEM);
        const unsigned vb = __hip_atomic_load(pb, __ATOMIC_RELAXED, __HIP_MEMORY_SCOPE_SYSTEM);
        const unsigned vt = __hip_atomic_load(tmo, __ATOMIC_RELAXED, __HIP_MEMORY_SCOPE_AGENT);
        if ((unsigned)RFL(vt) == call) return true;
        ra = __ballot((int)(va - target) >= 0); rb = __ballot((int)(vb - target) >= 0);
        if ((ra & rb & need) == need) return false;
        if (__builtin_amdgcn_s_memrealtime() - t0 > DEADLINE) {
            if (lane == 0) __hip_atomic_store(tmo, call, __ATOMIC_RELAXED, __HIP_MEMORY_SCOPE_AGENT);
            return true;
        }
        __builtin_amdgcn_s_sleep(POLL_SLEEP);
    }
}
__device__ __forceinline__ bool header_wait(gu32* ctl, int word, unsigned& h0, unsigned& h1, unsigned call, int lane) {
    const u64 t0 = __builtin_amdgcn_s_memrealtime();
    for (;;) {
        const unsigned v = __hip_atomic_load(ctl + (lane < 2 ? word + lane : W_TMO), __ATOMIC_RELAXED, __HIP_MEMORY_SCOPE_SYSTEM);
        h0 = RFL(v); h1 = __builtin_amdgcn_readlane(v, 1);
        if ((unsigned)__builtin_amdgcn_readlane(v, 2) == call) return true;
        if ((h0 >> 19) == (call & 0x1fffu) && (h1 >> 19) == (call & 0x1fffu)) return false;
        if (__builtin_amdgcn_s_memrealtime() - t0 > DEADLINE) {
            if (lane == 0) __hip_atomic_store(ctl + W_TMO, call, __ATOMIC_RELAXED, __HIP_MEMORY_SCOPE_AGENT);
            return true;
        }
        __builtin_amdgcn_s_sleep(POLL_SLEEP / 2);
    }
}

__device__ __forceinline__ void conv_of(int c, int me, bool& isA, int& dest, int& panel, int& j) {
    int b = -1;
    isA = false; dest = me;
    if (c < 200) { j = c >= 100; const int g = c - j * 100; isA = g < 16; panel = g; b = g - 16; }
    else if (c < 296) { isA = true; j = 2 + ((c - 200) >> 4); panel = (c - 200) & 15; }
    else if (c < 400) { j = (c - 296) / 12; panel = c - 296 - j * 12; }
    else if (c < 904) { j = (c - 400) / 84; b = c - 400 - j * 84; j += 2; }
    else { const int q = c - 904; j = q >> 5; panel = 12 + ((q >> 3) & 3); dest = q & 7; }
    if (b >= 0) { panel = b / 7; dest = (me + 1 + b - panel * 7) & 7; }
}
__device__ __forceinline__ void image_of(int i, int me, bool& isA, int& panel, int& j) {
    constexpr int J0 = 8 - NX, I0 = 26 * J0, I1 = I0 + 18 * NX;
    int g = 14;
    if (i < I0) { j = i / 26; g = i - j * 26; panel = g - 14; }
    else if (i < I1) { j = (i - I0) / 18; g = i - I0 - j * 18; j += J0; panel = 3 * (g - 14); }
    else if (i < 208) { j = J0 + ((i - I1) >> 3); panel = (((i - I1) >> 1) & 3) * 3 + 1 + ((i - I1) & 1); }
    else if (i < 240) { j = (i - 208) >> 2; panel = 12 + (i & 3); }
    else { j = (i - 240) >> 1; g = (i & 1) - 2; }
    isA = g < 14;
    if (isA) panel = (2 * me + 2 + g) & 15;
}
__device__ __forceinline__ int late_peer(int x, int k) { return (k + 3 * x) % 7; }
__device__ __forceinline__ unsigned late_order(int a) {
    unsigned o = 0u; int n = 0;
    for (int x = 0; x < 8; ++x) if ((a + 7 * 4 - 3 * x) % 7 < 4) o |= (unsigned)x << (4 * n++);
    for (int x = 0; x < 8; ++x) if ((a + 7 * 4 - 3 * x) % 7 >= 4) o |= (unsigned)x << (4 * n++);
    return o;
}
__device__ __forceinline__ void unit_of(int u, unsigned late, int& tile, int& k) {
    const int h = (u >> 3) & 1;
    if (u < 256) { k = (u >> 6) * 8 + (u & 7); tile = h * 16 + 3 * ((u >> 4) & 3); }
    else if (u < 768) { k = ((u - 256) >> 7) * 8 + (u & 7); tile = h * 16 + ((u >> 5) & 3) * 3 + 1 + ((u >> 4) & 1); }
    else { const int x = (late >> ((u - 768) >> 5) * 4) & 7; k = u & 31; tile = (x & 1) * 16 + 12 + (x >> 1); }
}
__device__ __forceinline__ void job_loads(f4 (&L)[8], const char* x, const char* w, bool isA, int dest, int panel, int j, int e, unsigned vxe, unsigned vl) {
    const char* b = isA ? x + ((size_t)(panel * 256 + e * 32) * KS + j * 64) * 4 : w + ((size_t)(j * 64 + e * 8) * NG + dest * NS + panel * 256) * 4;
    const unsigned ve = isA ? vxe : vl, vo = isA ? (vxe ^ 128u) : vl;
    const unsigned s2 = isA ? 8 * KS * 4 : 2 * NG * 4, s1 = isA ? 16 : NG * 4;
#pragma unroll
    for (int r = 0; r < 8; ++r) L[r] = *(const gf4*)(b + ((r >> 1) * s2 + (r & 1) * s1) + ((r & 2) ? vo : ve));
}
__device__ __forceinline__ void job_pack(const f4 (&L)[8], u4 (&Q)[4], bool isA) {
    if (isA) {
#pragma unroll
        for (int u = 0; u < 4; ++u) { Q[u].x = pk(L[2 * u].x, L[2 * u].y); Q[u].y = pk(L[2 * u].z, L[2 * u].w); Q[u].z = pk(L[2 * u + 1].x, L[2 * u + 1].y); Q[u].w = pk(L[2 * u + 1].z, L[2 * u + 1].w); }
    } else {
#pragma unroll
        for (int t = 0; t < 4; ++t) { Q[t].x = pk(L[0][t], L[1][t]); Q[t].y = pk(L[2][t], L[3][t]); Q[t].z = pk(L[4][t], L[5][t]); Q[t].w = pk(L[6][t], L[7][t]); }
    }
}

struct Com {
    gu32* ctl; char* me_ws; const char* src_ws; char* out; LAS unsigned char* ring; LAS unsigned char* lines; LAS unsigned* word;
    unsigned slot, call, vl, vo, late; int rank, src, lane;
};
__device__ __forceinline__ int item_word(const Com& C, unsigned n) {
    if (n < NI_IN) { bool isA; int panel, j; image_of(n >> 2, C.rank, isA, panel, j); return W_AVAIL + ((((C.call & 1) * 8 + C.src) * 2 + (isA ? 0 : 1)) * 16 + panel) * 32 + j * 4 + (n & 3); }
    if (n < NI) { int tile, k; unit_of(n - NI_IN, C.late, tile, k); return W_UNIT + (C.src * 32 + tile) * 32 + k; }
    return W_TMO;
}
__device__ __forceinline__ int h_byte(int idx, int lane) {
    return ((idx >> 5) * 2 + ((idx >> 2) & 1)) * 1024 + lane * 16 + (idx & 3) * 4 + ((idx >> 3) & 3);
}
__device__ __forceinline__ unsigned list_entry(const LAS unsigned char* head, const LAS unsigned char* more, unsigned pos) {
    const unsigned b = pos < 128u ? head[pos >> 1] : more[((pos - 128u) >> 1) & 255u];
    return (b >> ((pos & 1u) * 4u)) & 15u;
}
__device__ __forceinline__ unsigned piece_off(bool isA, int q, int p) {
    return isA ? (unsigned)(q * 8192 + p * 1024) : (unsigned)((2 * q + (p >> 2)) * 2048 + ((p >> 1) & 1) * HALF_B + (p & 1) * 1024);
}
__device__ __forceinline__ void put_quarter(const Com& C, unsigned n, unsigned hd, LAS unsigned char* z, const LAS unsigned char* head) {
    bool isA; int panel, j; image_of(n >> 2, C.rank, isA, panel, j);
    const char* db = C.me_ws + (C.slot + (isA ? O_A : O_B) + (unsigned)((panel * NKT + 8 + ((C.src - C.rank - 1) & 7) + j * 7) * TILE_B));
    LAS unsigned char* zl = z + C.lane * 16;
    if (hd & H_RAW) {
#pragma unroll
        for (int r = 0; r < 8; ++r) st4(C.vl, *(const LAS u4*)(zl + r * 1024), db + piece_off(isA, n & 3, r));
    } else {
        u4 lo[4], pz[3], h[4];
#pragma unroll
        for (int r = 0; r < 4; ++r) lo[r] = *(const LAS u4*)(zl + r * 1024);
#pragma unroll
        for (int r = 0; r < 3; ++r) pz[r] = *(const LAS u4*)(zl + 4096 + r * 1024);
        const unsigned bq = ((((hd >> 8) & 31u) << 2) | 0x80u) * 0x01010101u;
        unsigned m0 = 0u, m1 = 0u;
#pragma unroll
        for (int i = 0; i < 16; ++i) {
            const int g = i >> 2, k = i & 3;
            const unsigned f = g < 3 ? pz[g][k] : ((pz[0][k] >> 6) & 0x03030303u) | ((pz[1][k] >> 4) & 0x0c0c0c0cu) | ((pz[2][k] >> 2) & 0x30303030u);
            const unsigned c = f & 0x1c1c1c1cu, es = ((c + 0x04040404u) << 2) & 0x80808080u;
            h[g][k] = ((f << 2) & 0x80808080u) | ((bq - c) & 0x7c7c7c7cu & ~bytemask(es)) | (f & 0x03030303u);
            if (i < 8) m0 |= es >> (7 - i); else m1 |= es >> (15 - i);
        }
        asm volatile("" : "+v"(lo[0]), "+v"(lo[1]), "+v"(lo[2]), "+v"(lo[3]) :: "memory");
#pragma unroll
        for (int r = 0; r < 4; ++r) *(LAS u4*)(zl + r * 1024) = h[r];
        CFENCE;
        u64 m = ((u64)m1 << 32) | m0; unsigned base = 0u;
        for (;;) {
            const u64 act = __ballot(m != 0ull);
            if (act == 0ull) break;
            if (m != 0ull) {
                const unsigned pos = base + __builtin_amdgcn_mbcnt_hi((unsigned)(act >> 32), __builtin_amdgcn_mbcnt_lo((unsigned)act, 0u));
                const int o = h_byte(__builtin_ctzll(m), C.lane);
                const unsigned q4 = list_entry(head, z + 7168, pos), e = (((hd >> 8) & 31u) + 2u - (q4 < 2u ? q4 : q4 + 7u)) & 31u;
                __hip_atomic_fetch_or((LAS unsigned*)(z + (o & ~3)), e << (2 + 8 * (o & 3)), __ATOMIC_RELAXED, __HIP_MEMORY_SCOPE_WORKGROUP);
                m &= m - 1ull;
            }
            base += (unsigned)__popcll(act);
        }
        CFENCE;
#pragma unroll
        for (int r = 0; r < 4; ++r) h[r] = *(const LAS u4*)(zl + r * 1024);
#pragma unroll
        for (int p = 0; p < 8; ++p) {
            const unsigned ha = h[p >> 1][2 * (p & 1)], hb = h[p >> 1][2 * (p & 1) + 1], la = lo[p >> 1][2 * (p & 1)], lb = lo[p >> 1][2 * (p & 1) + 1];
            st4(C.vl, (u4){PERM(ha, la, 0x05010400u), PERM(ha, la, 0x07030602u), PERM(hb, lb, 0x05010400u), PERM(hb, lb, 0x07030602u)}, db + piece_off(isA, n & 3, p));
        }
    }
    WAIT_V(0);
    if (C.lane == 0) __hip_atomic_fetch_add(C.ctl + W_LAND + (((isA ? 0 : 16) + panel) * 8 + C.src) * 32 + j, 1u, __ATOMIC_RELAXED, __HIP_MEMORY_SCOPE_AGENT);
}
__device__ __forceinline__ const char* unit_rows(const Com& C, unsigned n) {
    int tile, k; unit_of(n - NI_IN, C.late, tile, k);
    const int g = k >> 3, w = k & 7;
    return C.out + ((size_t)((tile >> 4) * 256 + (g >> 1) * 128 + (w >> 2) * 64 + (g & 1) * 32) * NG + C.src * NS + (tile & 15) * 256 + (w & 3) * 64) * 4;
}
__device__ __forceinline__ void put_unit(const Com& C, unsigned n, unsigned hd, LAS unsigned char* z, const LAS unsigned char* head) {
    const char* ob = unit_rows(C, n);
    LAS unsigned char* zl = z + C.lane * 16;
    if (hd & H_RAW) {
#pragma unroll
        for (int q = 0; q < 8; ++q) st4(C.vo, *(const LAS u4*)(zl + q * 1024), ob + (size_t)((q >> 2) * 16 + (q & 3)) * (NG * 4));
        return;
    }
    u4 Z[6], E[2]; unsigned pc[3], c[8], m = 0u;
#pragma unroll
    for (int r = 0; r < 6; ++r) Z[r] = *(const LAS u4*)(zl + r * 1024);
#pragma unroll
    for (int r = 0; r < 3; ++r) pc[r] = *(const LAS unsigned*)(z + 6144 + r * 256 + C.lane * 4);
    c[0] = pc[0]; c[1] = pc[0] >> 3; c[2] = ((pc[0] >> 6) & 0x03030303u) | (pc[1] << 2 & 0x04040404u); c[3] = pc[1] >> 1; c[4] = pc[1] >> 4;
    c[5] = ((pc[1] >> 7) & 0x01010101u) | (pc[2] << 1 & 0x06060606u); c[6] = pc[2] >> 2; c[7] = pc[2] >> 5;
    const unsigned bp4 = ((hd >> 8) & 0xffu) * 0x01010101u, bn4 = (hd & 0xffu) * 0x01010101u;
#pragma unroll
    for (int q = 0; q < 8; ++q) {
        const unsigned cc = c[q] & 0x07070707u, es = ((cc + 0x01010101u) << 4) & 0x80808080u, keep = ~bytemask(es), mk = bytemask(Z[4 + (q >> 2)][q & 3] & 0x80808080u);
        E[q >> 2][q & 3] = ((bn4 & mk) | (bp4 & ~mk)) - (cc & keep);
        m |= es >> (7 - q);
    }
    asm volatile("" : "+v"(Z[0]), "+v"(Z[1]) :: "memory");
    *(LAS u4*)zl = E[0]; *(LAS u4*)(zl + 1024) = E[1];
    CFENCE;
    for (unsigned base = 0u;;) {
        const u64 act = __ballot(m != 0u);
        if (act == 0ull) break;
        if (m != 0u) {
            const unsigned pos = base + __builtin_amdgcn_mbcnt_hi((unsigned)(act >> 32), __builtin_amdgcn_mbcnt_lo((unsigned)act, 0u));
            const int o = h_byte(__builtin_ctz(m), C.lane);
            __hip_atomic_fetch_sub((LAS unsigned*)(z + (o & ~3)), (7u + list_entry(head, z + 6912, pos)) << (8 * (o & 3)), __ATOMIC_RELAXED, __HIP_MEMORY_SCOPE_WORKGROUP);
            m &= m - 1u;
        }
        base += (unsigned)__popcll(act);
    }
    CFENCE;
    E[0] = *(const LAS u4*)zl; E[1] = *(const LAS u4*)(zl + 1024);
#pragma unroll
    for (int q = 0; q < 8; ++q) {
        const unsigned e = E[q >> 2][q & 3], sm = Z[4 + (q >> 2)][q & 3];
        const unsigned tp = (sm & 0x80808080u) | ((e >> 1) & 0x7f7f7f7fu), md = (sm & 0x7f7f7f7fu) | ((e << 7) & 0x80808080u);
        const unsigned h01 = PERM(tp, md, 0x05010400u), h23 = PERM(tp, md, 0x07030602u), l01 = Z[q >> 1][2 * (q & 1)], l23 = Z[q >> 1][2 * (q & 1) + 1];
        st4(C.vo, (u4){PERM(h01, l01, 0x05040100u), PERM(h01, l01, 0x07060302u), PERM(h23, l23, 0x05040100u), PERM(h23, l23, 0x07060302u)},
            ob + (size_t)((q >> 2) * 16 + (q & 3)) * (NG * 4));
    }
}
#define LDS_LOAD(p) __hip_atomic_load((p), __ATOMIC_RELAXED, __HIP_MEMORY_SCOPE_WORKGROUP)
#define LDS_STORE(p, v) __hip_atomic_store((p), (v), __ATOMIC_RELAXED, __HIP_MEMORY_SCOPE_WORKGROUP)
constexpr unsigned IT_DEAD = 0x80000000u;
constexpr int PF = 2;
__device__ __forceinline__ void load_items(const Com& C, const Params& P, gu32* tick, unsigned tbase, unsigned cur, unsigned nxt) {
    const unsigned want = C.call & 0x1fffu, ring = (unsigned)(size_t)C.ring;
    unsigned k = 0u, h0 = (want ^ 1u) << 19, h1 = 0u; bool pend = false, dead = false;
    while (cur < NI / 2) {
        const bool in = cur < NI_IN / 2;
        unsigned so;
        if (in) { bool isA; int panel, j; image_of(cur >> 1, C.rank, isA, panel, j); so = (unsigned)((((isA ? C.src : C.rank) * 16 + panel) * 8 + j) * TILE_B) + (cur & 1) * 16384; }
        else { int tile, u; unit_of(2 * cur - NI_IN, C.late, tile, u); so = (O_OUT - O_X) + (unsigned)((C.rank * 32 + tile) * 32 + u) * UNIT_B; }
        if (!dead && ((h0 >> 19) != want || (h1 >> 19) != want)) {
            if (pend) { WAIT_V(0); if (C.lane == 0) LDS_STORE(C.word, k); pend = false; }
            dead = header_wait(C.ctl, item_word(C, 2 * cur), h0, h1, C.call, C.lane);
        }
        unsigned tv = 0; u64 fv = 0, ex;
        asm volatile("s_mov_b64 %1, exec\n\ts_mov_b64 exec, 1\n\tglobal_atomic_add %0, %2, %3, off sc0\n\ts_mov_b64 exec, %1"
                     : "+v"(tv), "=&s"(ex) : "v"(tick), "v"(nxt < NI / 2 ? 1u : 0u) : "memory");
        const unsigned nw = (unsigned)item_word(C, 2 * nxt) * 4u;
#pragma unroll
        for (int e = 0; e < 2; ++e) {
            if (dead && in) break;
            const unsigned hd = e ? h1 : h0;
            while (k >= (unsigned)RFL(LDS_LOAD(C.word + 1)) + 4u) __builtin_amdgcn_s_sleep(1);
            if (C.lane == 0) { C.word[4 + 2 * (k & 3)] = (2 * cur + e) | (dead ? IT_DEAD : 0u); C.word[5 + 2 * (k & 3)] = hd; }
            if (!dead) {
                const char* sb = C.src_ws + (C.slot + O_X + so + e * 8192);
                const unsigned lo = ring + (k & 3) * 8192, more = ((hd >> 16) & 3u) * 8u;
                const bool raw = hd & H_RAW;
                if (e == 0) dma_part(C.vl, C.src_ws + (C.slot + O_LIN + (so >> 14) * 128u), (unsigned)(size_t)C.lines + ((k >> 1) & 3u) * 128u, 8u);
                static_assert(PF <= 2, "the header load must be older than load 6 - PF");
#pragma unroll
                for (int r = 0; r < 6; ++r) {
                    if (e == 0 && r < PF) wait_v<PF + 1>(); else if (e == 1 && r > 3 && r <= 3 + PF) wait_v<PF>(); else wait_v<PF - 1>();
                    if (r == PF && pend && C.lane == 0) LDS_STORE(C.word, k);
                    dma(C.vl, sb + r * 1024, lo + r * 1024);
                    if (e == 1 && r == 3) asm volatile("global_load_dwordx2 %0, %1, %2 sc0 sc1" : "+v"(fv) : "v"(nw), "s"(C.ctl) : "memory");
                }
                wait_v<PF - 1>();
                dma_part(C.vl, sb + 6144, lo + 6144, in || raw ? 64u : 48u + more);
                if (raw || (in && more)) { wait_v<PF - 1>(); dma_part(C.vl, sb + 7168, lo + 7168, raw ? 64u : more); }
                pend = true; ++k;
            } else { ++k; CFENCE; if (C.lane == 0) LDS_STORE(C.word, k); }
        }
        if (dead) WAIT_V(0);
        asm volatile("" : "+v"(tv), "+v"(fv));
        const unsigned nn = nxt < NI / 2 ? (unsigned)RFL(tv) - tbase : NI / 2;
        cur = nxt; nxt = nn; h0 = RFL((unsigned)fv); h1 = RFL((unsigned)(fv >> 32));
    }
    WAIT_V(0);
    while (k >= (unsigned)RFL(LDS_LOAD(C.word + 1)) + 4u) __builtin_amdgcn_s_sleep(1);
    if (C.lane == 0) { C.word[4 + 2 * (k & 3)] = ~0u; CFENCE; LDS_STORE(C.word, k + 1u); }
}
__device__ __forceinline__ void unpack_items(const Com& C) {
    for (unsigned k = 0u;; ++k) {
        while ((unsigned)RFL(LDS_LOAD(C.word)) <= k) __builtin_amdgcn_s_sleep(1);
        CFENCE;
        const unsigned n = RFL(C.word[4 + 2 * (k & 3)]), hd = RFL(C.word[5 + 2 * (k & 3)]);
        if (n == ~0u) break;
        LAS unsigned char* z = C.ring + (k & 3) * 8192;
        const LAS unsigned char* head = C.lines + (k & 7) * 64;
        if (n & IT_DEAD) {
            const char* ob = unit_rows(C, n & ~IT_DEAD); const unsigned qn = 0x7fc00000u;
#pragma unroll
            for (int q = 0; q < 8; ++q) st4(C.vo, (u4){qn, qn, qn, qn}, ob + (size_t)((q >> 2) * 16 + (q & 3)) * (NG * 4));
        } else if (n < NI_IN) put_quarter(C, n, hd, z, head);
        else put_unit(C, n, hd, z, head);
        WAIT_L(0);
        if (C.lane == 0) LDS_STORE(C.word + 1, k + 1u);
    }
}

__global__ __launch_bounds__(NTHR, 2) void gemm_rs_kernel(const Params P) {
    __shared__ __attribute__((aligned(16))) unsigned char smem[LDS_BYTES];
    LAS unsigned char* lds = (LAS unsigned char*)smem;
    LAS unsigned* lctl = (LAS unsigned*)(lds + LDS_CTL);
    const int tid = threadIdx.x, lane = tid & 63, wid = RFL(tid >> 6);
    const int bid = blockIdx.x;
    int rank = P.rank; char* myws = P.myws; char* peerws = P.peer_ws[(bid >> 3) & 7]; const char* xb = (const char*)P.x; const char* wb = (const char*)P.w;
    asm volatile("" : "+s"(myws), "+s"(peerws), "+s"(xb), "+s"(wb), "+s"(rank));
    gu32* ctl = (gu32*)myws;
    STAMP(0);
    const unsigned vl = (unsigned)lane * 16u;
    const int gid = bid - NCOM;
    const unsigned vxe = (unsigned)((lane >> 3) * (KS * 4) + (((lane & 7) ^ (lane >> 4)) * 32));
    gu32* callp = ctl + W_CALL + bid * 32;
#define TAKE_CALL() CFENCE; const unsigned call = RFL(__hip_atomic_load(callp, __ATOMIC_RELAXED, __HIP_MEMORY_SCOPE_AGENT)) + 1u; \
    if (tid < 64) lctl[tid] = 0u; \
    asm volatile("s_waitcnt lgkmcnt(0)" :: "s"(call) : "memory");        \
    BAR; CFENCE; \
    if (tid == 0) __hip_atomic_store(callp, call, __ATOMIC_RELAXED, __HIP_MEMORY_SCOPE_AGENT); \
    const unsigned slot = CTL_B + (call & 1u) * SLOT_B

    if (__builtin_expect(gid < 0, 1)) {
        const int peer = (rank + 1 + (bid >> 3)) & 7, q0 = (bid & 7) * 4 + (wid & 3);
        const int qp = wid >= 4 ? q0 : 32 + (bid & 7) * PUSHX + wid, panel0 = (2 * peer + 2 + (qp >> 2)) & 15;
        const bool push = wid >= 4 || wid < PUSHX;
        f4 L0[8], L1[8];
        if (push) { job_loads(L0, xb, wb, true, 0, panel0, 0, 2 * (qp & 3), vxe, vl); job_loads(L1, xb, wb, true, 0, panel0, 0, 2 * (qp & 3) + 1, vxe, vl); }
        TAKE_CALL();
        Com C;
        C.ctl = ctl; C.me_ws = myws; C.rank = rank; C.src = peer; C.src_ws = peerws; C.out = (char*)P.out;
        C.ring = lds + (wid & 3) * 32768; C.lines = lds + LDS_LIN + (wid & 3) * 512; C.word = lctl + (wid & 3) * 16; C.slot = slot; C.call = call; C.vl = vl; C.lane = lane;
        C.vo = (unsigned)((lane >> 4) * 4 * (NG * 4) + (lane & 15) * 16);
        if (push) {
            u4 D[8];
            job_pack(L0, *(u4 (*)[4])&D[0], true); job_pack(L1, *(u4 (*)[4])&D[4], true);
            const char* db = C.src_ws + (slot + O_A + (unsigned)((panel0 * NKT + 8 + ((rank - peer - 1) & 7)) * TILE_B + (qp & 3) * 8192));
#pragma unroll
            for (int p = 0; p < 8; ++p) { if (p >= 2) WAIT_V(1); st4(vl, D[p], db + p * 1024); }
            WAIT_V(0);
            if (lane == 0) __hip_atomic_fetch_add((gu32*)C.src_ws + W_LAND + (panel0 * 8 + rank) * 32, 1u, __ATOMIC_RELAXED, __HIP_MEMORY_SCOPE_SYSTEM);
            if (wid == 4) STAMPW(1);
        }
        C.late = late_order((rank - peer - 1) & 7);
        if (wid >= 4) {
            unpack_items(C);
            if (wid == 4) STAMPW(20);
        } else {
            constexpr unsigned F0 = 16 + 4 * PUSHX;
            const unsigned tbase = (call - 1u) * (NI / 2 - F0 - 64 + 32) - (F0 + 64);
            load_items(C, P, ctl + W_TICK + (bid >> 3) * 32, tbase, F0 + q0, F0 + 32u + q0);
        }
        return;
    }

    f4 L0[8], L1[8];
    const int qlate = 1600 + (gid - 168) * 8 + wid;
    const int q0 = gid < 196 ? gid * 8 + wid : qlate;
    {
        bool isA; int dest, panel, j; conv_of(q0 >> 2, rank, isA, dest, panel, j);
        for (int i = gid >> 3; i > 0; --i) __builtin_amdgcn_s_sleep(8);
        CFENCE;
        job_loads(L0, xb, wb, isA, dest, panel, j, 2 * (q0 & 3), vxe, vl); job_loads(L1, xb, wb, isA, dest, panel, j, 2 * (q0 & 3) + 1, vxe, vl);
    }
    TAKE_CALL();
    gu32* tmo = ctl + W_TMO;
    {
        gu32* wsl = (gu32*)P.ws[lane & 7];
        asm volatile("" : "+v"(wsl));
        LAS unsigned char* z = lds + wid * 16384;
        for (int qn = q0, it = 0; qn < 4640; ++it) {
            bool isA; int dest, panel, j; conv_of(qn >> 2, rank, isA, dest, panel, j);
            const int qx = gid < 168 ? 4640 : qn < 1600 ? qlate : qn + 256;
            const int q = qn & 3;
            u4 D[8];
            job_pack(L0, *(u4 (*)[4])&D[0], isA); job_pack(L1, *(u4 (*)[4])&D[4], isA);
            const bool own = isA || dest == rank, peers = isA || dest != rank;
            const char* db = myws + (slot + (isA ? O_A : O_B) + (unsigned)((panel * NKT + j) * TILE_B));
            if (own) {
#pragma unroll
                for (int p = 0; p < 8; ++p) st4(vl, D[p], db + piece_off(isA, q, p));
            }
            unsigned hd = call << 19;
            if (peers) {
                const unsigned xo = (unsigned)((((isA ? rank : dest) * 16 + panel) * 8 + j) * TILE_B) + q * 8192;
                const char* xp = myws + (slot + O_X + xo);
                u4 lo[4], h[4]; unsigned se = 0u;
#pragma unroll
                for (int i = 0; i < 16; ++i) {
                    const unsigned d0 = D[i >> 1][2 * (i & 1)], d1 = D[i >> 1][2 * (i & 1) + 1];
                    lo[i >> 2][i & 3] = PERM(d1, d0, 0x06040200u); h[i >> 2][i & 3] = PERM(d1, d0, 0x07050301u);
                    se = __builtin_amdgcn_sad_u8(h[i >> 2][i & 3] & 0x7c7c7c7cu, 0u, se);
                }
#pragma unroll
                for (int r = 0; r < 4; ++r) *(LAS u4*)(z + r * 1024 + lane * 16) = h[r];
                *(LAS u64*)(z + 4096 + lane * 8) = 0ull;
#pragma unroll
                for (int o = 32; o >= 1; o >>= 1) se += (unsigned)__shfl_xor((int)se, o);
                const unsigned base = min(max(((unsigned)RFL(se) + 47500u) >> 14, 6u), 31u);
                const unsigned bq = ((base << 2) | 0x80u) * 0x01010101u;
                u4 f[4]; unsigned m0 = 0u, m1 = 0u;
#pragma unroll
                for (int i = 0; i < 16; ++i) {
                    const unsigned hh = h[i >> 2][i & 3], x = (bq - (hh & 0x7c7c7c7cu)) ^ 0x80808080u;
                    const unsigned es = (((x & 0x7f7f7f7fu) + 0x64646464u) | x) & 0x80808080u;
                    f[i >> 2][i & 3] = ((hh >> 2) & 0x20202020u) | ((x | bytemask(es)) & 0x1c1c1c1cu) | (hh & 0x03030303u);
                    if (i < 8) m0 |= es >> (7 - i); else m1 |= es >> (15 - i);
                }
                CFENCE;
                u64 m = ((u64)m1 << 32) | m0; unsigned cnt = 0u; bool far = false;
                for (;;) {
                    const u64 act = __ballot(m != 0ull);
                    if (act == 0ull || cnt > 640u) break;
                    if (m != 0ull) {
                        const unsigned pos = cnt + __builtin_amdgcn_mbcnt_hi((unsigned)(act >> 32), __builtin_amdgcn_mbcnt_lo((unsigned)act, 0u));
                        const unsigned d = base + 2u - ((z[h_byte(__builtin_ctzll(m), lane)] >> 2) & 31u);
                        far |= d > 22u;
                        __hip_atomic_fetch_or((LAS unsigned*)(z + 4096 + (pos >> 3) * 4), ((d < 2u ? d : d - 7u) & 15u) << ((pos & 7u) * 4u), __ATOMIC_RELAXED, __HIP_MEMORY_SCOPE_WORKGROUP);
                        m &= m - 1ull;
                    }
                    cnt += (unsigned)__popcll(act);
                }
                CFENCE;
                if (cnt <= 640u && !__any(far)) {
                    hd |= (((cnt + 127u) >> 8) << 16) | (base << 8);
#pragma unroll
                    for (int r = 0; r < 4; ++r) st4(vl, lo[r], xp + r * 1024);
#pragma unroll
                    for (int r = 0; r < 3; ++r) st4(vl, f[r] | ((f[3] << (6 - 2 * r)) & 0xc0c0c0c0u), xp + 4096 + r * 1024);
                    if (lane < 16) st1((unsigned)lane * 4u, *(const LAS unsigned*)(z + 4096 + lane * 4), myws + (slot + O_LIN + (xo >> 13) * 64u));
                    if (cnt > 128u) st1((unsigned)lane * 4u, *(const LAS unsigned*)(z + 4160 + lane * 4), xp + 7168);
                } else {
                    hd |= H_RAW;
#pragma unroll
                    for (int p = 0; p < 8; ++p) st4(vl, D[p], xp + p * 1024);
                }
            }
            if (qx < 4640) {
                bool nA; int nd, np, nj; conv_of(qx >> 2, rank, nA, nd, np, nj);
                job_loads(L0, xb, wb, nA, nd, np, nj, 2 * (qx & 3), vxe, vl); job_loads(L1, xb, wb, nA, nd, np, nj, 2 * (qx & 3) + 1, vxe, vl);
                wait_v<16>();
            } else WAIT_V(0);
            const int im = it * 2 + (wid >> 2);
            unsigned old = 0u;
            if (lane == 0) { lctl[L_HD + im * 4 + q] = hd; old = __hip_atomic_fetch_add(lctl + L_CNT + im, 1u, __ATOMIC_RELAXED, __HIP_MEMORY_SCOPE_WORKGROUP); }
            if ((unsigned)RFL(old) == 3u && lane < NR && (isA || lane == dest)) {
                CFENCE;
                if (lane == rank) __hip_atomic_fetch_add(ctl + W_LAND + (((isA ? 0 : 16) + panel) * 8 + rank) * 32 + j, 4u, __ATOMIC_RELAXED, __HIP_MEMORY_SCOPE_AGENT);
                else { const u4 h4 = *(const LAS u4*)(lctl + L_HD + im * 4);
                       asm volatile("global_store_dwordx4 %0, %1, off sc0 sc1\n\ts_nop 1" :: "v"(wsl + W_AVAIL + ((((call & 1) * 8 + rank) * 2 + (isA ? 0 : 1)) * 16 + panel) * 32 + j * 4), "v"(h4) : "memory"); }
            }
            if (qn == q0) STAMP(10);
            qn = qx;
        }
        WAIT_V(0); WAIT_L(0);
        STAMP(16);
    }

    const int wr = wid >> 2, wc = wid & 3;
    const int xcd = gid & 7, idx = gid >> 3, xh = xcd & 1, xq = xcd >> 1;
    for (int round = 0; round < 2; ++round) {
        int pm, pn;
        if (round == 0 && idx < 21) { const int b = idx / 7; pn = 3 * xq + b; pm = 2 * ((rank + 1 + idx - b * 7) & 7) + xh; }
        else if (round == 0 || idx < 3) { pn = 12 + xq; pm = 2 * ((rank + 1 + late_peer(xcd, round == 0 ? idx - 21 : idx + 4)) & 7) + xh; }
        else if (idx < 7) { pm = 2 * rank + xh; pn = 4 * xq + idx - 3; }
        else break;
        pm = RFL(pm); pn = RFL(pn);
        int tz = threadIdx.x; asm volatile("" : "+v"(tz));
        const int ln = tz & 63, fr = ln & 15, fq = ln >> 4;
        gu32* fa = ctl + W_LAND + (pm * 8 + (ln & 7)) * 32 + (ln >> 3);
        gu32* fb = ctl + W_LAND + ((16 + pn) * 8 + (ln & 7)) * 32 + (ln >> 3);
        const u64 own = 0x0101010101010101ull << rank;
        u64 ra = 0, rb = 0;
        bool bad = false;
#define STEP_WAIT(need, last) do { if (!bad && ((last) || (ra & rb & (need)) != (need))) bad = wave_wait(fa, fb, (need), ra, rb, call * 4u, tmo, call, ln); } while (0)
        if (wid == 0) { STEP_WAIT(own, false); WAIT_V(0); if (round == 0) STAMPW(1); }
        CFENCE; BAR; CFENCE;

        const char* cA = myws + (slot + O_A + (unsigned)(pm * NKT) * TILE_B);
        const char* cB = myws + (slot + O_B + (unsigned)(pn * NKT) * TILE_B);
        const unsigned vst = (unsigned)tz * 16u;
        const unsigned ldsw = (unsigned)wid * 1024u;
        const int a0 = (wr * 64 + fr) * 128 + ((fq ^ (fr >> 1)) * 16), a1 = a0 ^ 64;
        const int b0 = fq * 2048 + (wc * 16 + fr) * 16;
#define SA(b, h) (((b) * 2 + (h)) * HALF_B)
#define SB(b, h) ((4 + (b) * 2 + (h)) * HALF_B)
#define STAGE(bufoff, gbase) do { _Pragma("unroll") for (int _i = 0; _i < 2; ++_i) \
        __builtin_amdgcn_global_load_lds((const unsigned*)((gbase) + vst + _i * 8192), (LAS unsigned*)(lds + (bufoff) + ldsw + _i * 8192), 16, 0, 17); } while (0)
#define LDA(dst, b, h) do { _Pragma("unroll") for (int m = 0; m < 4; ++m) { dst[m][0] = *(const LAS h8*)(lds + SA(b, h) + a0 + m * 2048); dst[m][1] = *(const LAS h8*)(lds + SA(b, h) + a1 + m * 2048); } } while (0)
#define LDB(dst, b, h) do { _Pragma("unroll") for (int n = 0; n < 2; ++n) _Pragma("unroll") for (int k = 0; k < 2; ++k) dst[n][k] = *(const LAS h8*)(lds + SB(b, h) + b0 + k * 8192 + n * 1024); } while (0)
#define MMA(ai, bj, At, Bt) do { __builtin_amdgcn_s_setprio(1); _Pragma("unroll") for (int m = 0; m < 4; ++m) _Pragma("unroll") for (int n = 0; n < 2; ++n) _Pragma("unroll") for (int k = 0; k < 2; ++k) \
        acc[ai][bj][m][n] = __builtin_amdgcn_mfma_f32_16x16x32_f16(At[m][k], Bt[n][k], acc[ai][bj][m][n], 0, 0, 0); __builtin_amdgcn_s_setprio(0); } while (0)
        f4 acc[2][2][4][2];
#pragma unroll
        for (int a = 0; a < 2; ++a)
#pragma unroll
            for (int b = 0; b < 2; ++b)
#pragma unroll
                for (int m = 0; m < 4; ++m)
#pragma unroll
                    for (int n = 0; n < 2; ++n) acc[a][b][m][n] = (f4){0.f, 0.f, 0.f, 0.f};
        h8 At[4][2], B0[2][2], B1[2][2];
        STAGE(SB(0, 0), cB); STAGE(SA(0, 0), cA); STAGE(SB(0, 1), cB + HALF_B); STAGE(SA(0, 1), cA + HALF_B);
        if (wr == 1) BAR;
        WAIT_V(4); BAR;
        STAGE(SB(1, 0), cB + TILE_B); STAGE(SA(1, 0), cA + TILE_B); STAGE(SB(1, 1), cB + TILE_B + HALF_B);
        WAIT_V(6); BAR;
        int jn = 0, bnd = 8;
        for (int t = 0; t < NKT; t += 2) {
            const int t2 = (t + 2) & (NKT - 1);
            const char* a1p = cA + (size_t)(t + 1) * TILE_B;
            const char* a2 = cA + (size_t)t2 * TILE_B; const char* b2 = cB + (size_t)t2 * TILE_B;
            const char* a3 = a2 + TILE_B; const char* b3 = b2 + TILE_B;
            if (t + 3 >= bnd && jn < 8) {
                if (wid == 0) {
                    STEP_WAIT((0xFFull << (8 * jn)) & ~own, jn == 7);
                    if (jn == 7 && ln == 0) lctl[L_BAD] = bad ? 1u : 0u;
                    if (round == 0) STAMPW(2 + jn);
                }
                CFENCE; BAR; CFENCE;
                ++jn; bnd += 7;
            }
            LDB(B0, 0, 0); SCHED; LDA(At, 0, 0); STAGE(SA(1, 1), a1p + HALF_B);
            WAIT_L(8); BAR; WAIT_L(0); MMA(0, 0, At, B0); BAR; SCHED;
            LDB(B1, 0, 1); STAGE(SB(0, 0), b2);
            BAR; WAIT_L(0); MMA(0, 1, At, B1); BAR;
            LDA(At, 0, 1); STAGE(SA(0, 0), a2);
            BAR; WAIT_L(0); MMA(1, 0, At, B0); BAR; SCHED;
            STAGE(SB(0, 1), b2 + HALF_B);
            WAIT_V(6); BAR; MMA(1, 1, At, B1); BAR;
            LDB(B0, 1, 0); SCHED; LDA(At, 1, 0); STAGE(SA(0, 1), a2 + HALF_B);
            WAIT_L(8); BAR; WAIT_L(0); MMA(0, 0, At, B0); BAR; SCHED;
            LDB(B1, 1, 1); STAGE(SB(1, 0), b3);
            BAR; WAIT_L(0); MMA(0, 1, At, B1); BAR;
            LDA(At, 1, 1); STAGE(SA(1, 0), a3);
            BAR; WAIT_L(0); MMA(1, 0, At, B0); BAR; SCHED;
            STAGE(SB(1, 1), b3 + HALF_B);
            WAIT_V(6); BAR; MMA(1, 1, At, B1); BAR;
        }
        WAIT_V(0);
        if (wr == 0) BAR;
        CFENCE; BAR; CFENCE;
        STAMP(round == 0 ? 17 : 11);

        const unsigned dest = ((unsigned)pm >> 1) & 7u;
        int ty = threadIdx.x; asm volatile("" : "+v"(ty));
        const int l2 = ty & 63;
        const unsigned vo = (unsigned)((l2 >> 4) * 4 * (NG * 4) + (l2 & 15) * 16);
        const bool poison = lctl[L_BAD] != 0u;
        const float qnan = __builtin_nanf("");
#define ROW4(ai, m, r) (poison ? (f4){qnan, qnan, qnan, qnan} : (f4){gelu(acc[ai][0][m][0][r]), gelu(acc[ai][0][m][1][r]), gelu(acc[ai][1][m][0][r]), gelu(acc[ai][1][m][1][r])})
        if (dest == (unsigned)rank) {
            const char* ob = (const char*)P.out + ((size_t)((pm & 1) * 256 + wr * 64) * NG + rank * NS + pn * 256 + wc * 64) * 4;
#pragma unroll
            for (int ai = 0; ai < 2; ++ai)
#pragma unroll
                for (int m = 0; m < 4; ++m)
#pragma unroll
                    for (int r = 0; r < 4; ++r) { const f4 v = ROW4(ai, m, r); st4(vo, __builtin_bit_cast(u4, v), ob + (size_t)(ai * 128 + m * 16 + r) * (NG * 4)); }
        } else {
            const int tile = (pm & 1) * 16 + pn;
            const unsigned zo = (O_OUT - O_X) + (unsigned)((dest * 32 + tile) * 32 + wid) * UNIT_B;
            gu32* uf = (gu32*)P.ws[dest] + W_UNIT + (rank * 32 + tile) * 32;
            LAS unsigned char* z = lds + wid * 16384;
#pragma unroll
            for (int g = 0; g < 4; ++g) {
                const int ai = g >> 1;
                unsigned lo[16], sm[8], mp = 0u, mn = 0u; u4 E[2];
#pragma unroll
                for (int q = 0; q < 8; ++q) {
                    const int m = (g & 1) * 2 + (q >> 2), r = q & 3;
                    const u4 u = __builtin_bit_cast(u4, ROW4(ai, m, r));
                    lo[2 * q] = PERM(u.y, u.x, 0x05040100u); lo[2 * q + 1] = PERM(u.w, u.z, 0x05040100u);
                    const unsigned h01 = PERM(u.y, u.x, 0x07060302u), h23 = PERM(u.w, u.z, 0x07060302u);
                    sm[q] = (PERM(h23, h01, 0x06040200u) & 0x7f7f7f7fu) | (PERM(h23, h01, 0x07050301u) & 0x80808080u);
                    E[q >> 2][q & 3] = PERM(h23 << 1, h01 << 1, 0x07050301u);
                    mp = pmax_i16(mp, pmax_i16(h01, h23)); mn = pmax_u16(mn, pmax_u16(h01, h23));
                }
                *(LAS u64*)(z + 2048 + l2 * 8) = 0ull;
                const unsigned nm = max(mn & 0xffffu, mn >> 16);
                unsigned bb = (max((mp >> 7) & 0xffu, mp >> 23) << 16) | ((nm & 0x8000u) ? (nm >> 7) & 0xffu : 0u);
#pragma unroll
                for (int o = 32; o >= 1; o >>= 1) bb = pmax_u16(bb, (unsigned)__shfl_xor((int)bb, o));
                bb = RFL(bb);
                const unsigned bp = bb >> 16, bn = bb & 0xffu, bp4 = bp * 0x01010101u, bn4 = bn * 0x01010101u;
                unsigned c[8], m = 0u, cnt = 0u, hd = call << 19; u4 D[2]; bool far = false;
#pragma unroll
                for (int q = 0; q < 8; ++q) {
                    const unsigned mk = bytemask(sm[q] & 0x80808080u), d = ((bn4 & mk) | (bp4 & ~mk)) - E[q >> 2][q & 3];
                    const unsigned es = (((d & 0x7f7f7f7fu) + 0x79797979u) | d) & 0x80808080u;
                    c[q] = (d | bytemask(es)) & 0x07070707u; D[q >> 2][q & 3] = d;
                    m |= es >> (7 - q);
                }
                *(LAS u4*)(z + l2 * 16) = D[0]; *(LAS u4*)(z + 1024 + l2 * 16) = D[1];
                CFENCE;
                for (;;) {
                    const u64 act = __ballot(m != 0u);
                    if (act == 0ull || cnt > 640u) break;
                    if (m != 0u) {
                        const unsigned pos = cnt + __builtin_amdgcn_mbcnt_hi((unsigned)(act >> 32), __builtin_amdgcn_mbcnt_lo((unsigned)act, 0u));
                        const unsigned d = z[h_byte(__builtin_ctz(m), l2)];
                        far |= d > 22u;
                        __hip_atomic_fetch_or((LAS unsigned*)(z + 2048 + (pos >> 3) * 4), ((d - 7u) & 15u) << ((pos & 7u) * 4u), __ATOMIC_RELAXED, __HIP_MEMORY_SCOPE_WORKGROUP);
                        m &= m - 1u;
                    }
                    cnt += (unsigned)__popcll(act);
                }
                CFENCE;
                const char* zp = myws + (slot + O_X + zo + g * 8 * UNIT_B);
                if (cnt <= 640u && !__any(far)) {
                    hd |= (((cnt + 127u) >> 8) << 16) | (bp << 8) | bn;
#pragma unroll
                    for (int i = 0; i < 4; ++i) st4((unsigned)l2 * 16u, (u4){lo[4 * i], lo[4 * i + 1], lo[4 * i + 2], lo[4 * i + 3]}, zp + i * 1024);
                    st4((unsigned)l2 * 16u, (u4){sm[0], sm[1], sm[2], sm[3]}, zp + 4096); st4((unsigned)l2 * 16u, (u4){sm[4], sm[5], sm[6], sm[7]}, zp + 5120);
                    st1((unsigned)l2 * 4u, c[0] | (c[1] << 3) | (c[2] << 6 & 0xc0c0c0c0u), zp + 6144);
                    st1((unsigned)l2 * 4u, (c[2] >> 2 & 0x01010101u) | (c[3] << 1) | (c[4] << 4) | (c[5] << 7 & 0x80808080u), zp + 6400);
                    st1((unsigned)l2 * 4u, (c[5] >> 1 & 0x03030303u) | (c[6] << 2) | (c[7] << 5), zp + 6656);
                    if (l2 < 16) st1((unsigned)l2 * 4u, *(const LAS unsigned*)(z + 2048 + l2 * 4), myws + (slot + O_LIN + ((zo >> 13) + g * 8) * 64u));
                    if (cnt > 128u) st1((unsigned)l2 * 4u, *(const LAS unsigned*)(z + 2112 + l2 * 4), zp + 6912);
                } else {
                    hd |= H_RAW;
#pragma unroll
                    for (int q = 0; q < 8; ++q) {
                        const unsigned e = E[q >> 2][q & 3], tp = (sm[q] & 0x80808080u) | ((e >> 1) & 0x7f7f7f7fu), md = (sm[q] & 0x7f7f7f7fu) | ((e << 7) & 0x80808080u);
                        const unsigned h01 = PERM(tp, md, 0x05010400u), h23 = PERM(tp, md, 0x07030602u);
                        st4((unsigned)l2 * 16u, (u4){PERM(h01, lo[2 * q], 0x05040100u), PERM(h01, lo[2 * q], 0x07060302u), PERM(h23, lo[2 * q + 1], 0x05040100u), PERM(h23, lo[2 * q + 1], 0x07060302u)}, zp + q * 1024);
                    }
                }
                WAIT_V(0);
                const int ig = round * 4 + g;
                unsigned old = 0u;
                if (l2 == 0) { lctl[L_HD + 104 + ig * 8 + wid] = hd; old = __hip_atomic_fetch_add(lctl + L_CNT + 26 + ig, 1u, __ATOMIC_RELAXED, __HIP_MEMORY_SCOPE_WORKGROUP); }
                if ((unsigned)RFL(old) == 7u && l2 < 8) { CFENCE; __hip_atomic_store(uf + g * 8 + l2, lctl[L_HD + 104 + ig * 8 + l2], __ATOMIC_RELAXED, __HIP_MEMORY_SCOPE_SYSTEM); }
            }
        }
        STAMP(round == 0 ? 18 : 12);
        WAIT_V(0); WAIT_L(0); BAR; CFENCE;
    }
    STAMP(20);
}

}

extern "C" void kernel_launch(int rank, int nranks, ncclComm_t comm, hipStream_t stream,
                              void* const* d_in, const int* in_sizes, int n_in,
                              void* d_out, int out_size, const int* in_dtypes, int out_dtype,
                              void* const* d_out_all, void* const* d_ws_all, size_t ws_bytes) {
    (void)comm; (void)in_dtypes; (void)out_dtype; (void)d_out_all;
    if (nranks != NR || (unsigned)rank >= (unsigned)NR || n_in < 2 || ws_bytes < WS_NEED || in_sizes[0] != MG * KS || in_sizes[1] != KS * NG || out_size != MS * NG) {
        fprintf(stderr, "kernel_launch: unexpected problem shape\n");
        return;
    }
    Params P;
    P.x = (const float*)d_in[0]; P.w = (const float*)d_in[1];
    for (int p = 0; p < NR; ++p) P.ws[p] = (char*)d_ws_all[p];
    for (int p = 0; p < NR; ++p) P.peer_ws[p] = P.ws[(rank + 1 + p) & 7];
    P.myws = P.ws[rank]; P.out = (float*)d_out;
    P.rank = rank; P.pad = 0;
    hipLaunchKernelGGL(gemm_rs_kernel, dim3(NWG), dim3(NTHR), 0, stream, P);
    HIP_CHECK(hipGetLastError());
}
```

```cpp
#include <hip/hip_runtime.h>
#include <rccl/rccl.h>
#include <cstdio>

#define HIP_CHECK(expr) do { hipError_t _e = (expr); if (_e != hipSuccess) { \
    fprintf(stderr, "HIP error %s at %s:%d\n", hipGetErrorString(_e), __FILE__, __LINE__); \
    return; } } while (0)

namespace {

typedef _Float16 h8 __attribute__((ext_vector_type(8)));
typedef _Float16 h2 __attribute__((ext_vector_type(2)));
typedef float f4 __attribute__((ext_vector_type(4)));
typedef unsigned u4 __attribute__((ext_vector_type(4)));
typedef short s2v __attribute__((ext_vector_type(2)));
typedef unsigned short us2v __attribute__((ext_vector_type(2)));
typedef unsigned long long u64;
typedef __attribute__((address_space(1))) unsigned gu32;
typedef __attribute__((address_space(1))) f4 gf4;
#define LAS __attribute__((address_space(3)))

constexpr int NR = 8;
constexpr int MG = 4096, KG = 4096, NG = 32768;
constexpr int KS = KG / NR;
constexpr int NS = NG / NR;
constexpr int MS = MG / NR;
constexpr int NWG = 256, NTHR = 512;
constexpr int NCOM = 56;
constexpr int NX = 4;
constexpr int PUSHX = 1;
constexpr unsigned NSGL = 128;
constexpr int POLL_SLEEP = 16;
constexpr int TILE_B = 256 * 64 * 2;
constexpr int HALF_B = TILE_B / 2;
constexpr int NKT = KG / 64;
constexpr unsigned NI_IN = 1024, NI = 2048;
constexpr unsigned OP_B = 16u * NKT * TILE_B;
constexpr unsigned UNIT_B = 8192;
constexpr unsigned CTL_B = 1u << 20;
constexpr unsigned O_A = 0, O_B = OP_B, O_X = 2 * OP_B, O_OUT = 3 * OP_B, O_LIN = O_OUT + NR * 32 * 32 * UNIT_B,
                   SLOT_B = O_LIN + (1u << 20);
constexpr size_t WS_NEED = (size_t)CTL_B + 2 * (size_t)SLOT_B;
constexpr int W_TMO = 0;
constexpr int W_LAND = 32;
constexpr int W_AVAIL = W_LAND + 256 * 32;
constexpr int W_UNIT = W_AVAIL + 512 * 32;
constexpr unsigned H_RAW = 1u << 18;
constexpr int W_TICK = W_UNIT + 256 * 32;
constexpr int W_CALL = W_TICK + 8 * 32;
static_assert((W_CALL + NWG * 32) * 4 <= CTL_B, "control layout");
constexpr u64 DEADLINE = 40000000ull;

constexpr int LDS_CTL = 8 * HALF_B;
constexpr int LDS_LIN = LDS_CTL + 1024, LDS_BYTES = LDS_LIN + 2048;
constexpr int L_BAD = 0, L_CNT = 2, L_HD = 64;

struct Params {
    const float* x; const float* w; char* myws; float* out;
    int rank; int pad;
    char* ws[NR];
    char* peer_ws[NR];
};

#define WAIT_V(n) asm volatile("s_waitcnt vmcnt(" #n ")" ::: "memory")
#define WAIT_L(n) asm volatile("s_waitcnt lgkmcnt(" #n ")" ::: "memory")
#define BAR __builtin_amdgcn_s_barrier()
#define SCHED __builtin_amdgcn_sched_barrier(0)
#define CFENCE asm volatile("" ::: "memory")
#define RFL(x) __builtin_amdgcn_readfirstlane(x)
#define PERM(hi, lo, sel) __builtin_amdgcn_perm((hi), (lo), (sel))
template <int N> __device__ __forceinline__ void wait_v() { asm volatile("s_waitcnt vmcnt(%0)" :: "i"(N) : "memory"); }

__device__ __forceinline__ void st4(unsigned voff, u4 v, const void* sbase) {
    asm volatile("global_store_dwordx4 %0, %1, %2 sc0 sc1\n\ts_nop 1" :: "v"(voff), "v"(v), "s"(sbase) : "memory");
}
__device__ __forceinline__ void st1(unsigned voff, unsigned v, const void* sbase) {
    asm volatile("global_store_dword %0, %1, %2 sc0 sc1" :: "v"(voff), "v"(v), "s"(sbase) : "memory");
}
__device__ __forceinline__ void dma(unsigned voff, const void* sbase, unsigned ldsoff) {
    unsigned keep;
    asm volatile("s_mov_b32 %0, m0\n\ts_mov_b32 m0, %3\n\ts_nop 0\n\tglobal_load_lds_dwordx4 %1, %2 sc0 sc1\n\ts_mov_b32 m0, %0"
                 : "=&s"(keep) : "v"(voff), "s"(sbase), "s"(ldsoff) : "memory");
}
__device__ __forceinline__ void dma_part(unsigned voff, const void* sbase, unsigned ldsoff, unsigned lanes) {
    unsigned keep; u64 ex;
    asm volatile("s_mov_b32 %0, m0\n\ts_mov_b32 m0, %4\n\ts_mov_b64 %1, exec\n\ts_mov_b64 exec, %5\n\tglobal_load_lds_dwordx4 %2, %3 sc0 sc1\n\ts_mov_b64 exec, %1\n\ts_mov_b32 m0, %0"
                 : "=&s"(keep), "=&s"(ex) : "v"(voff), "s"(sbase), "s"(ldsoff), "s"(~0ull >> (64u - lanes)) : "memory");
}
__device__ __forceinline__ unsigned pmax_i16(unsigned a, unsigned b) { return __builtin_bit_cast(unsigned, __builtin_elementwise_max(__builtin_bit_cast(s2v, a), __builtin_bit_cast(s2v, b))); }
__device__ __forceinline__ unsigned pmax_u16(unsigned a, unsigned b) { return __builtin_bit_cast(unsigned, __builtin_elementwise_max(__builtin_bit_cast(us2v, a), __builtin_bit_cast(us2v, b))); }
__device__ __forceinline__ unsigned bytemask(unsigned s) { return (s - (s >> 7)) | s; }
__device__ __forceinline__ unsigned pk(float a, float b) {
    h2 r; r.x = (_Float16)a; r.y = (_Float16)b; return __builtin_bit_cast(unsigned, r);
}
__device__ __forceinline__ float gelu(float y) {
    const float k1 = -2.0f * 0.7978845608028654f * 1.4426950408889634f, k2 = k1 * 0.044715f;
    const float e = __builtin_amdgcn_exp2f(y * (k1 + k2 * y * y));
    return y * __builtin_amdgcn_rcpf(1.0f + e);
}

__device__ __forceinline__ bool wave_wait(gu32* pa, gu32* pb, u64 need, u64& ra, u64& rb, unsigned target, gu32* tmo, unsigned call, int lane) {
    const u64 t0 = __builtin_amdgcn_s_memrealtime();
    for (;;) {
        const unsigned va = __hip_atomic_load(pa, __ATOMIC_RELAXED, __HIP_MEMORY_SCOPE_SYSTEM);
        const unsigned vb = __hip_atomic_load(pb, __ATOMIC_RELAXED, __HIP_MEMORY_SCOPE_SYSTEM);
        const unsigned vt = __hip_atomic_load(tmo, __ATOMIC_RELAXED, __HIP_MEMORY_SCOPE_AGENT);
        if ((unsigned)RFL(vt) == call) return true;
        ra = __ballot((int)(va - target) >= 0); rb = __ballot((int)(vb - target) >= 0);
        if ((ra & rb & need) == need) return false;
        if (__builtin_amdgcn_s_memrealtime() - t0 > DEADLINE) {
            if (lane == 0) __hip_atomic_store(tmo, call, __ATOMIC_RELAXED, __HIP_MEMORY_SCOPE_AGENT);
            return true;
        }
        __builtin_amdgcn_s_sleep(POLL_SLEEP);
    }
}
__device__ __forceinline__ bool header_wait(gu32* ctl, int word, unsigned& h0, unsigned& h1, unsigned call, int lane) {
    const u64 t0 = __builtin_amdgcn_s_memrealtime();
    for (;;) {
        const unsigned v = __hip_atomic_load(ctl + (lane < 2 ? word + lane : W_TMO), __ATOMIC_RELAXED, __HIP_MEMORY_SCOPE_SYSTEM);
        h0 = RFL(v); h1 = __builtin_amdgcn_readlane(v, 1);
        if ((unsigned)__builtin_amdgcn_readlane(v, 2) == call) return true;
        if ((h0 >> 19) == (call & 0x1fffu) && (h1 >> 19) == (call & 0x1fffu)) return false;
        if (__builtin_amdgcn_s_memrealtime() - t0 > DEADLINE) {
            if (lane == 0) __hip_atomic_store(ctl + W_TMO, call, __ATOMIC_RELAXED, __HIP_MEMORY_SCOPE_AGENT);
            return true;
        }
        __builtin_amdgcn_s_sleep(POLL_SLEEP / 2);
    }
}

__device__ __forceinline__ void conv_of(int c, int me, bool& isA, int& dest, int& panel, int& j) {
    int b = -1;
    isA = false; dest = me;
    if (c < 200) { j = c >= 100; const int g = c - j * 100; isA = g < 16; panel = g; b = g - 16; }
    else if (c < 296) { isA = true; j = 2 + ((c - 200) >> 4); panel = (c - 200) & 15; }
    else if (c < 400) { j = (c - 296) / 12; panel = c - 296 - j * 12; }
    else if (c < 904) { j = (c - 400) / 84; b = c - 400 - j * 84; j += 2; }
    else { const int q = c - 904; j = q >> 5; panel = 12 + ((q >> 3) & 3); dest = q & 7; }
    if (b >= 0) { panel = b / 7; dest = (me + 1 + b - panel * 7) & 7; }
}
__device__ __forceinline__ void image_of(int i, int me, bool& isA, int& panel, int& j) {
    constexpr int J0 = 8 - NX, I0 = 26 * J0, I1 = I0 + 18 * NX;
    int g = 14;
    if (i < I0) { j = i / 26; g = i - j * 26; panel = g - 14; }
    else if (i < I1) { j = (i - I0) / 18; g = i - I0 - j * 18; j += J0; panel = 3 * (g - 14); }
    else if (i < 208) { j = J0 + ((i - I1) >> 3); panel = (((i - I1) >> 1) & 3) * 3 + 1 + ((i - I1) & 1); }
    else if (i < 240) { j = (i - 208) >> 2; panel = 12 + (i & 3); }
    else { j = (i - 240) >> 1; g = (i & 1) - 2; }
    isA = g < 14;
    if (isA) panel = (2 * me + 2 + g) & 15;
}
__device__ __forceinline__ int late_peer(int x, int k) { return (k + 3 * x) % 7; }
__device__ __forceinline__ unsigned late_order(int a) {
    unsigned o = 0u; int n = 0;
    for (int x = 0; x < 8; ++x) if ((a + 7 * 4 - 3 * x) % 7 < 4) o |= (unsigned)x << (4 * n++);
    for (int x = 0; x < 8; ++x) if ((a + 7 * 4 - 3 * x) % 7 >= 4) o |= (unsigned)x << (4 * n++);
    return o;
}
__device__ __forceinline__ void unit_of(int u, unsigned late, int& tile, int& k) {
    const int h = (u >> 3) & 1;
    if (u < 256) { k = (u >> 6) * 8 + (u & 7); tile = h * 16 + 3 * ((u >> 4) & 3); }
    else if (u < 768) { k = ((u - 256) >> 7) * 8 + (u & 7); tile = h * 16 + ((u >> 5) & 3) * 3 + 1 + ((u >> 4) & 1); }
    else {
        const int t = u < 896 ? (u - 768) >> 5 : 4 + ((u >> 3) & 3), x = (late >> t * 4) & 7;
        k = u < 896 ? u & 31 : ((u - 896) >> 5) * 8 + (u & 7); tile = (x & 1) * 16 + 12 + (x >> 1);
    }
}
__device__ __forceinline__ void job_loads(f4 (&L)[8], const char* x, const char* w, bool isA, int dest, int panel, int j, int e, unsigned vxe, unsigned vl) {
    const char* b = isA ? x + ((size_t)(panel * 256 + e * 32) * KS + j * 64) * 4 : w + ((size_t)(j * 64 + e * 8) * NG + dest * NS + panel * 256) * 4;
    const unsigned ve = isA ? vxe : vl, vo = isA ? (vxe ^ 128u) : vl;
    const unsigned s2 = isA ? 8 * KS * 4 : 2 * NG * 4, s1 = isA ? 16 : NG * 4;
#pragma unroll
    for (int r = 0; r < 8; ++r) L[r] = *(const gf4*)(b + ((r >> 1) * s2 + (r & 1) * s1) + ((r & 2) ? vo : ve));
}
__device__ __forceinline__ void job_pack(const f4 (&L)[8], u4 (&Q)[4], bool isA) {
    if (isA) {
#pragma unroll
        for (int u = 0; u < 4; ++u) { Q[u].x = pk(L[2 * u].x, L[2 * u].y); Q[u].y = pk(L[2 * u].z, L[2 * u].w); Q[u].z = pk(L[2 * u + 1].x, L[2 * u + 1].y); Q[u].w = pk(L[2 * u + 1].z, L[2 * u + 1].w); }
    } else {
#pragma unroll
        for (int t = 0; t < 4; ++t) { Q[t].x = pk(L[0][t], L[1][t]); Q[t].y = pk(L[2][t], L[3][t]); Q[t].z = pk(L[4][t], L[5][t]); Q[t].w = pk(L[6][t], L[7][t]); }
    }
}

struct Com {
    gu32* ctl; char* me_ws; const char* src_ws; char* out; LAS unsigned char* ring; LAS unsigned char* lines; LAS unsigned* word;
    unsigned slot, call, vl, vo, late; int rank, src, lane;
};
__device__ __forceinline__ int item_word(const Com& C, unsigned n) {
    if (n < NI_IN) { bool isA; int panel, j; image_of(n >> 2, C.rank, isA, panel, j); return W_AVAIL + ((((C.call & 1) * 8 + C.src) * 2 + (isA ? 0 : 1)) * 16 + panel) * 32 + j * 4 + (n & 3); }
    if (n < NI) { int tile, k; unit_of(n - NI_IN, C.late, tile, k); return W_UNIT + (C.src * 32 + tile) * 32 + k; }
    return W_TMO;
}
__device__ __forceinline__ int h_byte(int idx, int lane) {
    return ((idx >> 5) * 2 + ((idx >> 2) & 1)) * 1024 + lane * 16 + (idx & 3) * 4 + ((idx >> 3) & 3);
}
#define DPP(x, ctrl, rmask) (unsigned)__builtin_amdgcn_update_dpp(0, (int)(x), (ctrl), (rmask), 0xf, false)
__device__ __forceinline__ unsigned wave_sum(unsigned x) {
    x += DPP(x, 0x111, 0xf); x += DPP(x, 0x112, 0xf); x += DPP(x, 0x114, 0xf); x += DPP(x, 0x118, 0xf);
    x += DPP(x, 0x142, 0xa); x += DPP(x, 0x143, 0xc);
    return __builtin_amdgcn_readlane(x, 63);
}
__device__ __forceinline__ unsigned wave_max(unsigned x) {
    x = max(x, DPP(x, 0x111, 0xf)); x = max(x, DPP(x, 0x112, 0xf)); x = max(x, DPP(x, 0x114, 0xf)); x = max(x, DPP(x, 0x118, 0xf));
    x = max(x, DPP(x, 0x142, 0xa)); x = max(x, DPP(x, 0x143, 0xc));
    return __builtin_amdgcn_readlane(x, 63);
}
__device__ __forceinline__ unsigned lane_rank(u64 m) { return __builtin_amdgcn_mbcnt_hi((unsigned)(m >> 32), __builtin_amdgcn_mbcnt_lo((unsigned)m, 0u)); }
__device__ __forceinline__ unsigned lanes_below(unsigned c, unsigned& total) {
    unsigned p = 0u; total = 0u;
#pragma unroll
    for (int b = 0; b < 5; ++b) { const u64 m = __ballot((c >> b) & 1u); p += lane_rank(m) << b; total += (unsigned)__popcll(m) << b; }
    return p;
}
__device__ __forceinline__ void list_put(LAS unsigned* l, unsigned first, u64 nb) {
    const unsigned sh = (first & 7u) * 4u; l += first >> 3;
    __hip_atomic_fetch_or(l, (unsigned)(nb << sh), __ATOMIC_RELAXED, __HIP_MEMORY_SCOPE_WORKGROUP);
    __hip_atomic_fetch_or(l + 1, (unsigned)((nb << sh) >> 32), __ATOMIC_RELAXED, __HIP_MEMORY_SCOPE_WORKGROUP);
    __hip_atomic_fetch_or(l + 2, (unsigned)((nb >> 1) >> (63u - sh)), __ATOMIC_RELAXED, __HIP_MEMORY_SCOPE_WORKGROUP);
}
__device__ __forceinline__ u64 list_get(const LAS unsigned char* head, LAS unsigned char* more, unsigned cnt, int lane) {
    if (lane < 16) *(LAS unsigned*)(more - 64 + lane * 4) = *(const LAS unsigned*)(head + lane * 4);
    CFENCE;
    unsigned total; const unsigned first = lanes_below(cnt, total), sh = (first & 7u) * 4u;
    const LAS unsigned* l = (const LAS unsigned*)(more - 64) + min(first >> 3, 80u);
    return ((((u64)l[1] << 32) | l[0]) >> sh) | (((u64)l[2] << 1) << (63u - sh));
}
__device__ __forceinline__ unsigned piece_off(bool isA, int q, int p) {
    return isA ? (unsigned)(q * 8192 + p * 1024) : (unsigned)((2 * q + (p >> 2)) * 2048 + ((p >> 1) & 1) * HALF_B + (p & 1) * 1024);
}
#define SETTLE() do { if (owed) { WAIT_V(0); if (C.lane == 0) __hip_atomic_fetch_add(C.ctl + owed, 1u, __ATOMIC_RELAXED, __HIP_MEMORY_SCOPE_AGENT); owed = 0; } } while (0)
__device__ __forceinline__ void put_quarter(const Com& C, unsigned n, unsigned hd, LAS unsigned char* z, const LAS unsigned char* head, int& owed) {
    bool isA; int panel, j; image_of(n >> 2, C.rank, isA, panel, j);
    const char* db = C.me_ws + (C.slot + (isA ? O_A : O_B) + (unsigned)((panel * NKT + 8 + ((C.src - C.rank - 1) & 7) + j * 7) * TILE_B));
    LAS unsigned char* zl = z + C.lane * 16;
    if (hd & H_RAW) {
        SETTLE();
#pragma unroll
        for (int r = 0; r < 8; ++r) st4(C.vl, *(const LAS u4*)(zl + r * 1024), db + piece_off(isA, n & 3, r));
    } else {
        u4 lo[4], pz[3], h[4];
#pragma unroll
        for (int r = 0; r < 4; ++r) lo[r] = *(const LAS u4*)(zl + r * 1024);
#pragma unroll
        for (int r = 0; r < 3; ++r) pz[r] = *(const LAS u4*)(zl + 4096 + r * 1024);
        const unsigned bq = ((((hd >> 8) & 31u) << 2) | 0x80u) * 0x01010101u;
        unsigned m0 = 0u, m1 = 0u;
#pragma unroll
        for (int i = 0; i < 16; ++i) {
            const int g = i >> 2, k = i & 3;
            const unsigned f = g < 3 ? pz[g][k] : ((pz[0][k] >> 6) & 0x03030303u) | ((pz[1][k] >> 4) & 0x0c0c0c0cu) | ((pz[2][k] >> 2) & 0x30303030u);
            const unsigned c = f & 0x1c1c1c1cu, es = ((c + 0x04040404u) << 2) & 0x80808080u;
            h[g][k] = ((f << 2) & 0x80808080u) | ((bq - c) & 0x7c7c7c7cu & ~bytemask(es)) | (f & 0x03030303u);
            if (i < 8) m0 |= es >> (7 - i); else m1 |= es >> (15 - i);
        }
        asm volatile("" : "+v"(lo[0]), "+v"(lo[1]), "+v"(lo[2]), "+v"(lo[3]), "+v"(m0), "+v"(m1) :: "memory");
#pragma unroll
        for (int r = 0; r < 4; ++r) *(LAS u4*)(zl + r * 1024) = h[r];
        u64 m = ((u64)m1 << 32) | m0, nb = list_get(head, z + 7168, (unsigned)__popcll(m), C.lane);
        while (__any(m != 0ull)) {
            if (m != 0ull) {
                const int o = h_byte(__builtin_ctzll(m), C.lane);
                const unsigned q4 = (unsigned)nb & 15u, e = (((hd >> 8) & 31u) + 2u - (q4 < 2u ? q4 : q4 + 7u)) & 31u;
                __hip_atomic_fetch_or((LAS unsigned*)(z + (o & ~3)), e << (2 + 8 * (o & 3)), __ATOMIC_RELAXED, __HIP_MEMORY_SCOPE_WORKGROUP);
                m &= m - 1ull; nb >>= 4;
            }
        }
        CFENCE;
#pragma unroll
        for (int r = 0; r < 4; ++r) h[r] = *(const LAS u4*)(zl + r * 1024);
        SETTLE();
#pragma unroll
        for (int p = 0; p < 8; ++p) {
            const unsigned ha = h[p >> 1][2 * (p & 1)], hb = h[p >> 1][2 * (p & 1) + 1], la = lo[p >> 1][2 * (p & 1)], lb = lo[p >> 1][2 * (p & 1) + 1];
            st4(C.vl, (u4){PERM(ha, la, 0x05010400u), PERM(ha, la, 0x07030602u), PERM(hb, lb, 0x05010400u), PERM(hb, lb, 0x07030602u)}, db + piece_off(isA, n & 3, p));
        }
    }
    owed = W_LAND + (((isA ? 0 : 16) + panel) * 8 + C.src) * 32 + j;
}
__device__ __forceinline__ const char* unit_rows(const Com& C, unsigned n) {
    int tile, k; unit_of(n - NI_IN, C.late, tile, k);
    const int g = k >> 3, w = k & 7;
    return C.out + ((size_t)((tile >> 4) * 256 + (g >> 1) * 128 + (w >> 2) * 64 + (g & 1) * 32) * NG + C.src * NS + (tile & 15) * 256 + (w & 3) * 64) * 4;
}
__device__ __forceinline__ void put_unit(const Com& C, unsigned n, unsigned hd, LAS unsigned char* z, const LAS unsigned char* head) {
    const char* ob = unit_rows(C, n);
    LAS unsigned char* zl = z + C.lane * 16;
    if (hd & H_RAW) {
#pragma unroll
        for (int q = 0; q < 8; ++q) st4(C.vo, *(const LAS u4*)(zl + q * 1024), ob + (size_t)((q >> 2) * 16 + (q & 3)) * (NG * 4));
        return;
    }
    u4 Z[6], E[2]; unsigned pc[3], c[8], m = 0u;
#pragma unroll
    for (int r = 0; r < 6; ++r) Z[r] = *(const LAS u4*)(zl + r * 1024);
#pragma unroll
    for (int r = 0; r < 3; ++r) pc[r] = *(const LAS unsigned*)(z + 6144 + r * 256 + C.lane * 4);
    c[0] = pc[0]; c[1] = pc[0] >> 3; c[2] = ((pc[0] >> 6) & 0x03030303u) | (pc[1] << 2 & 0x04040404u); c[3] = pc[1] >> 1; c[4] = pc[1] >> 4;
    c[5] = ((pc[1] >> 7) & 0x01010101u) | (pc[2] << 1 & 0x06060606u); c[6] = pc[2] >> 2; c[7] = pc[2] >> 5;
    const unsigned bp4 = ((hd >> 8) & 0xffu) * 0x01010101u, bn4 = (hd & 0xffu) * 0x01010101u;
#pragma unroll
    for (int q = 0; q < 8; ++q) {
        const unsigned cc = c[q] & 0x07070707u, es = ((cc + 0x01010101u) << 4) & 0x80808080u, keep = ~bytemask(es), mk = bytemask(Z[4 + (q >> 2)][q & 3] & 0x80808080u);
        E[q >> 2][q & 3] = ((bn4 & mk) | (bp4 & ~mk)) - (cc & keep);
        m |= es >> (7 - q);
    }
    asm volatile("" : "+v"(Z[0]), "+v"(Z[1]), "+v"(m) :: "memory");
    *(LAS u4*)zl = E[0]; *(LAS u4*)(zl + 1024) = E[1];
    u64 nb = list_get(head, z + 6912, (unsigned)__popc(m), C.lane);
    while (__any(m != 0u)) {
        if (m != 0u) {
            const int o = h_byte(__builtin_ctz(m), C.lane);
            __hip_atomic_fetch_sub((LAS unsigned*)(z + (o & ~3)), (7u + ((unsigned)nb & 15u)) << (8 * (o & 3)), __ATOMIC_RELAXED, __HIP_MEMORY_SCOPE_WORKGROUP);
            m &= m - 1u; nb >>= 4;
        }
    }
    CFENCE;
    E[0] = *(const LAS u4*)zl; E[1] = *(const LAS u4*)(zl + 1024);
#pragma unroll
    for (int q = 0; q < 8; ++q) {
        const unsigned e = E[q >> 2][q & 3], sm = Z[4 + (q >> 2)][q & 3];
        const unsigned tp = (sm & 0x80808080u) | ((e >> 1) & 0x7f7f7f7fu), md = (sm & 0x7f7f7f7fu) | ((e << 7) & 0x80808080u);
        const unsigned h01 = PERM(tp, md, 0x05010400u), h23 = PERM(tp, md, 0x07030602u), l01 = Z[q >> 1][2 * (q & 1)], l23 = Z[q >> 1][2 * (q & 1) + 1];
        st4(C.vo, (u4){PERM(h01, l01, 0x05040100u), PERM(h01, l01, 0x07060302u), PERM(h23, l23, 0x05040100u), PERM(h23, l23, 0x07060302u)},
            ob + (size_t)((q >> 2) * 16 + (q & 3)) * (NG * 4));
    }
}
#define LDS_LOAD(p) __hip_atomic_load((p), __ATOMIC_RELAXED, __HIP_MEMORY_SCOPE_WORKGROUP)
#define LDS_STORE(p, v) __hip_atomic_store((p), (v), __ATOMIC_RELAXED, __HIP_MEMORY_SCOPE_WORKGROUP)
constexpr unsigned IT_DEAD = 0x80000000u;
constexpr int PF = 2;
constexpr unsigned TP = (NI - NSGL) / 2, NT = TP + NSGL;
__device__ __forceinline__ unsigned first_item(unsigned t) { return t < TP ? 2 * t : t + TP; }
__device__ __forceinline__ void load_items(const Com& C, gu32* tick, unsigned tbase, unsigned cur, unsigned nxt) {
    const unsigned want = C.call & 0x1fffu, ring = (unsigned)(size_t)C.ring;
    unsigned k = 0u, nl = 0u, h0 = (want ^ 1u) << 19, h1 = 0u; bool pend = false, dead = false;
    while (cur < NT) {
        const unsigned n0 = first_item(cur), np = n0 & ~1u, n1 = cur < TP ? n0 + 1u : n0;
        const bool in = n0 < NI_IN;
        unsigned so;
        if (in) { bool isA; int panel, j; image_of(np >> 2, C.rank, isA, panel, j); so = (unsigned)((((isA ? C.src : C.rank) * 16 + panel) * 8 + j) * TILE_B) + (np & 3) * 8192; }
        else { int tile, u; unit_of(np - NI_IN, C.late, tile, u); so = (O_OUT - O_X) + (unsigned)((C.rank * 32 + tile) * 32 + u) * UNIT_B; }
        if (!dead && ((h0 >> 19) != want || (h1 >> 19) != want)) {
            if (pend) { WAIT_V(0); if (C.lane == 0) LDS_STORE(C.word, k); pend = false; }
            dead = header_wait(C.ctl, item_word(C, np), h0, h1, C.call, C.lane);
        }
        unsigned tv = 0; u64 fv = 0, ex;
        asm volatile("s_mov_b64 %1, exec\n\ts_mov_b64 exec, 1\n\tglobal_atomic_add %0, %2, %3, off sc0\n\ts_mov_b64 exec, %1"
                     : "+v"(tv), "=&s"(ex) : "v"(tick), "v"(nxt < NT ? 1u : 0u) : "memory");
        if (dead) { WAIT_V(0); asm volatile("" : "+v"(tv)); }
        const unsigned nw = (unsigned)item_word(C, first_item(nxt) & ~1u) * 4u, line = (nl & 3u) * 128u;
        for (unsigned n = n0; n <= n1 && !(dead && in); ++n) {
            const unsigned hd = (n & 1u) ? h1 : h0;
            while (k >= (unsigned)RFL(LDS_LOAD(C.word + 1)) + 4u) __builtin_amdgcn_s_sleep(1);
            if (C.lane == 0) { C.word[4 + 2 * (k & 3)] = n | ((line + (n & 1u) * 64u) << 16) | (dead ? IT_DEAD : 0u); C.word[5 + 2 * (k & 3)] = hd; }
            if (!dead) {
                const char* sb = C.src_ws + (C.slot + O_X + so + (n & 1u) * 8192u);
                const unsigned lo = ring + (k & 3) * 8192, more = min((hd >> 16) & 3u, 2u) * 8u;
                const bool raw = hd & H_RAW;
                if (n == n0) dma_part(C.vl, C.src_ws + (C.slot + O_LIN + (so >> 14) * 128u), (unsigned)(size_t)C.lines + line, 8u);
                static_assert(PF <= 2, "the header load must be older than load 6 - PF");
#pragma unroll
                for (int r = 0; r < 6; ++r) {
                    if (r < PF) { if (n == n0) wait_v<PF + 1>(); else wait_v<PF - 1>(); } else if (r > 3 && r <= 3 + PF) wait_v<PF>(); else wait_v<PF - 1>();
                    if (r == PF && pend && C.lane == 0) LDS_STORE(C.word, k);
                    dma(C.vl, sb + r * 1024, lo + r * 1024);
                    if (r == 3) asm volatile("global_load_dwordx2 %0, %1, %2 sc0 sc1" : "+v"(fv) : "v"(nw), "s"(C.ctl) : "memory");
                }
                wait_v<PF - 1>();
                asm volatile("" : "+v"(tv), "+v"(fv));
                dma_part(C.vl, sb + 6144, lo + 6144, in || raw ? 64u : 48u + more);
                if (raw || (in && more)) { wait_v<PF - 1>(); dma_part(C.vl, sb + 7168, lo + 7168, raw ? 64u : more); }
                pend = true; ++k;
            } else { ++k; CFENCE; if (C.lane == 0) LDS_STORE(C.word, k); }
        }
        const unsigned nn = nxt < NT ? (unsigned)RFL(tv) - tbase : NT;
        cur = nxt; nxt = nn; h0 = RFL((unsigned)fv); h1 = RFL((unsigned)(fv >> 32)); ++nl;
    }
    WAIT_V(0);
    while (k >= (unsigned)RFL(LDS_LOAD(C.word + 1)) + 4u) __builtin_amdgcn_s_sleep(1);
    if (C.lane == 0) { C.word[4 + 2 * (k & 3)] = ~0u; CFENCE; LDS_STORE(C.word, k + 1u); }
}
__device__ __forceinline__ void unpack_items(const Com& C) {
    int owed = 0;
    for (unsigned k = 0u;; ++k) {
        while ((unsigned)RFL(LDS_LOAD(C.word)) <= k) { SETTLE(); __builtin_amdgcn_s_sleep(1); }
        CFENCE;
        const unsigned it = RFL(C.word[4 + 2 * (k & 3)]), hd = RFL(C.word[5 + 2 * (k & 3)]), n = it & 0xffffu;
        if (n >= NI_IN) SETTLE();
        if (it == ~0u) break;
        LAS unsigned char* z = C.ring + (k & 3) * 8192;
        const LAS unsigned char* head = C.lines + ((it >> 16) & 510u);
        if (it & IT_DEAD) {
            const char* ob = unit_rows(C, n); const unsigned qn = 0x7fc00000u;
#pragma unroll
            for (int q = 0; q < 8; ++q) st4(C.vo, (u4){qn, qn, qn, qn}, ob + (size_t)((q >> 2) * 16 + (q & 3)) * (NG * 4));
        } else if (n < NI_IN) put_quarter(C, n, hd, z, head, owed);
        else put_unit(C, n, hd, z, head);
        WAIT_L(0);
        if (C.lane == 0) LDS_STORE(C.word + 1, k + 1u);
    }
}

__global__ __launch_bounds__(NTHR, 2) void gemm_rs_kernel(const Params P) {
    __shared__ __attribute__((aligned(16))) unsigned char smem[LDS_BYTES];
    LAS unsigned char* lds = (LAS unsigned char*)smem;
    LAS unsigned* lctl = (LAS unsigned*)(lds + LDS_CTL);
    const int tid = threadIdx.x, lane = tid & 63, wid = RFL(tid >> 6);
    const int bid = blockIdx.x;
    int rank = P.rank; char* myws = P.myws; char* peerws = P.peer_ws[(bid >> 3) & 7]; const char* xb = (const char*)P.x; const char* wb = (const char*)P.w;
    asm volatile("" : "+s"(myws), "+s"(peerws), "+s"(xb), "+s"(wb), "+s"(rank));
    gu32* ctl = (gu32*)myws;
    const unsigned vl = (unsigned)lane * 16u;
    const int gid = bid - NCOM;
    const unsigned vxe = (unsigned)((lane >> 3) * (KS * 4) + (((lane & 7) ^ (lane >> 4)) * 32));
    gu32* callp = ctl + W_CALL + bid * 32;
#define TAKE_CALL() CFENCE; const unsigned call = RFL(__hip_atomic_load(callp, __ATOMIC_RELAXED, __HIP_MEMORY_SCOPE_AGENT)) + 1u; \
    if (tid < 64) lctl[tid] = 0u; \
    asm volatile("s_waitcnt lgkmcnt(0)" :: "s"(call) : "memory");        \
    BAR; CFENCE; \
    if (tid == 0) __hip_atomic_store(callp, call, __ATOMIC_RELAXED, __HIP_MEMORY_SCOPE_AGENT); \
    const unsigned slot = CTL_B + (call & 1u) * SLOT_B

    if (__builtin_expect(gid < 0, 1)) {
        const int peer = (rank + 1 + (bid >> 3)) & 7, q0 = (bid & 7) * 4 + (wid & 3);
        const int qp = wid >= 4 ? q0 : 32 + (bid & 7) * PUSHX + wid, panel0 = (2 * peer + 2 + (qp >> 2)) & 15;
        const bool push = wid >= 4 || wid < PUSHX;
        f4 L0[8], L1[8];
        if (push) { job_loads(L0, xb, wb, true, 0, panel0, 0, 2 * (qp & 3), vxe, vl); job_loads(L1, xb, wb, true, 0, panel0, 0, 2 * (qp & 3) + 1, vxe, vl); }
        TAKE_CALL();
        Com C;
        C.ctl = ctl; C.me_ws = myws; C.rank = rank; C.src = peer; C.src_ws = peerws; C.out = (char*)P.out;
        C.ring = lds + (wid & 3) * 32768; C.lines = lds + LDS_LIN + (wid & 3) * 512; C.word = lctl + (wid & 3) * 16; C.slot = slot; C.call = call; C.vl = vl; C.lane = lane;
        C.vo = (unsigned)((lane >> 4) * 4 * (NG * 4) + (lane & 15) * 16);
        if (push) {
            u4 D[8];
            job_pack(L0, *(u4 (*)[4])&D[0], true); job_pack(L1, *(u4 (*)[4])&D[4], true);
            const char* db = C.src_ws + (slot + O_A + (unsigned)((panel0 * NKT + 8 + ((rank - peer - 1) & 7)) * TILE_B + (qp & 3) * 8192));
#pragma unroll
            for (int p = 0; p < 8; ++p) { if (p >= 2) WAIT_V(1); st4(vl, D[p], db + p * 1024); }
            WAIT_V(0);
            if (lane == 0) __hip_atomic_fetch_add((gu32*)C.src_ws + W_LAND + (panel0 * 8 + rank) * 32, 1u, __ATOMIC_RELAXED, __HIP_MEMORY_SCOPE_SYSTEM);
        }
        C.late = late_order((rank - peer - 1) & 7);
        if (wid >= 4) {
            unpack_items(C);
        } else {
            constexpr unsigned F0 = 16 + 4 * PUSHX;
            const unsigned tbase = (call - 1u) * (NT - F0 - 64 + 32) - (F0 + 64);
            load_items(C, ctl + W_TICK + (bid >> 3) * 32, tbase, F0 + q0, F0 + 32u + q0);
        }
        return;
    }

    f4 L0[8], L1[8];
    const int qlate = 1600 + (gid - 168) * 8 + wid;
    const int q0 = gid < 196 ? gid * 8 + wid : qlate;
    {
        bool isA; int dest, panel, j; conv_of(q0 >> 2, rank, isA, dest, panel, j);
        for (int i = gid < 50 ? gid >> 3 : 10 + ((gid - 48) >> 3) * 3; i > 0; --i) __builtin_amdgcn_s_sleep(8);
        CFENCE;
        job_loads(L0, xb, wb, isA, dest, panel, j, 2 * (q0 & 3), vxe, vl); job_loads(L1, xb, wb, isA, dest, panel, j, 2 * (q0 & 3) + 1, vxe, vl);
    }
    TAKE_CALL();
    gu32* tmo = ctl + W_TMO;
    {
        gu32* wsl = (gu32*)P.ws[lane & 7];
        asm volatile("" : "+v"(wsl));
        LAS unsigned char* z = lds + wid * 16384;
        for (int qn = q0, it = 0; qn < 4640; ++it) {
            bool isA; int dest, panel, j; conv_of(qn >> 2, rank, isA, dest, panel, j);
            const int qx = gid < 168 ? 4640 : qn < 1600 ? qlate : qn + 256;
            const int q = qn & 3;
            u4 D[8];
            job_pack(L0, *(u4 (*)[4])&D[0], isA); job_pack(L1, *(u4 (*)[4])&D[4], isA);
            const bool own = isA || dest == rank, peers = isA || dest != rank;
            const char* db = myws + (slot + (isA ? O_A : O_B) + (unsigned)((panel * NKT + j) * TILE_B));
            if (own) {
#pragma unroll
                for (int p = 0; p < 8; ++p) st4(vl, D[p], db + piece_off(isA, q, p));
            }
            unsigned hd = call << 19;
            if (peers) {
                const unsigned xo = (unsigned)((((isA ? rank : dest) * 16 + panel) * 8 + j) * TILE_B) + q * 8192;
                const char* xp = myws + (slot + O_X + xo);
                u4 lo[4], h[4]; unsigned se = 0u;
#pragma unroll
                for (int i = 0; i < 16; ++i) {
                    const unsigned d0 = D[i >> 1][2 * (i & 1)], d1 = D[i >> 1][2 * (i & 1) + 1];
                    lo[i >> 2][i & 3] = PERM(d1, d0, 0x06040200u); h[i >> 2][i & 3] = PERM(d1, d0, 0x07050301u);
                    se = __builtin_amdgcn_sad_u8(h[i >> 2][i & 3] & 0x7c7c7c7cu, 0u, se);
                }
#pragma unroll
                for (int r = 0; r < 4; ++r) *(LAS u4*)(z + r * 1024 + lane * 16) = h[r];
                *(LAS u4*)(z + 4096 + lane * 16) = (u4){0u, 0u, 0u, 0u};
                const unsigned base = min(max((wave_sum(se) + 47500u) >> 14, 6u), 31u);
                const unsigned bq = ((base << 2) | 0x80u) * 0x01010101u;
                u4 f[4]; unsigned m0 = 0u, m1 = 0u;
#pragma unroll
                for (int i = 0; i < 16; ++i) {
                    const unsigned hh = h[i >> 2][i & 3], x = (bq - (hh & 0x7c7c7c7cu)) ^ 0x80808080u;
                    const unsigned es = (((x & 0x7f7f7f7fu) + 0x64646464u) | x) & 0x80808080u;
                    f[i >> 2][i & 3] = ((hh >> 2) & 0x20202020u) | ((x | bytemask(es)) & 0x1c1c1c1cu) | (hh & 0x03030303u);
                    if (i < 8) m0 |= es >> (7 - i); else m1 |= es >> (15 - i);
                }
                CFENCE;
                u64 m = ((u64)m1 << 32) | m0; unsigned cnt = 0u; bool far = __popcll(m) > 16;
                if (!__any(far)) {
                    const unsigned first = lanes_below((unsigned)__popcll(m), cnt);
                    u64 nb = 0ull; unsigned hb = z[h_byte(__builtin_ctzll(m | (1ull << 63)), lane)];
                    for (unsigned sh = 0u; __any(m != 0ull); sh += 4u) {
                        const unsigned d = base + 2u - ((hb >> 2) & 31u);
                        if (m != 0ull) { far |= d > 22u; nb |= (u64)((d < 2u ? d : d - 7u) & 15u) << sh; }
                        m &= m - 1ull;
                        hb = z[h_byte(__builtin_ctzll(m | (1ull << 63)), lane)];
                    }
                    list_put((LAS unsigned*)(z + 4096), first, nb);
                }
                CFENCE;
                if (cnt <= 640u && !__any(far)) {
                    hd |= (((cnt + 127u) >> 8) << 16) | (base << 8);
#pragma unroll
                    for (int r = 0; r < 4; ++r) st4(vl, lo[r], xp + r * 1024);
#pragma unroll
                    for (int r = 0; r < 3; ++r) st4(vl, f[r] | ((f[3] << (6 - 2 * r)) & 0xc0c0c0c0u), xp + 4096 + r * 1024);
                    if (lane < 16) st1((unsigned)lane * 4u, *(const LAS unsigned*)(z + 4096 + lane * 4), myws + (slot + O_LIN + (xo >> 13) * 64u));
                    if (cnt > 128u) st1((unsigned)lane * 4u, *(const LAS unsigned*)(z + 4160 + lane * 4), xp + 7168);
                } else {
                    hd |= H_RAW;
#pragma unroll
                    for (int p = 0; p < 8; ++p) st4(vl, D[p], xp + p * 1024);
                }
            }
            if (qx < 4640) {
                bool nA; int nd, np, nj; conv_of(qx >> 2, rank, nA, nd, np, nj);
                job_loads(L0, xb, wb, nA, nd, np, nj, 2 * (qx & 3), vxe, vl); job_loads(L1, xb, wb, nA, nd, np, nj, 2 * (qx & 3) + 1, vxe, vl);
                wait_v<16>();
            } else WAIT_V(0);
            const int im = it * 2 + (wid >> 2);
            unsigned old = 0u;
            if (lane == 0) { lctl[L_HD + im * 4 + q] = hd; old = __hip_atomic_fetch_add(lctl + L_CNT + im, 1u, __ATOMIC_RELAXED, __HIP_MEMORY_SCOPE_WORKGROUP); }
            if ((unsigned)RFL(old) == 3u && lane < NR && (isA || lane == dest)) {
                CFENCE;
                if (lane == rank) __hip_atomic_fetch_add(ctl + W_LAND + (((isA ? 0 : 16) + panel) * 8 + rank) * 32 + j, 4u, __ATOMIC_RELAXED, __HIP_MEMORY_SCOPE_AGENT);
                else { const u4 h4 = *(const LAS u4*)(lctl + L_HD + im * 4);
                       asm volatile("global_store_dwordx4 %0, %1, off sc0 sc1\n\ts_nop 1" :: "v"(wsl + W_AVAIL + ((((call & 1) * 8 + rank) * 2 + (isA ? 0 : 1)) * 16 + panel) * 32 + j * 4), "v"(h4) : "memory"); }
            }
            qn = qx;
        }
        WAIT_V(0); WAIT_L(0);
    }

    const int wr = wid >> 2, wc = wid & 3;
    const int xcd = gid & 7, idx = gid >> 3, xh = xcd & 1, xq = xcd >> 1;
    for (int round = 0; round < 2; ++round) {
        int pm, pn;
        if (round == 0 && idx < 21) { const int b = idx / 7; pn = 3 * xq + b; pm = 2 * ((rank + 1 + idx - b * 7) & 7) + xh; }
        else if (round == 0 || idx < 3) { pn = 12 + xq; pm = 2 * ((rank + 1 + late_peer(xcd, round == 0 ? idx - 21 : idx + 4)) & 7) + xh; }
        else if (idx < 7) { pm = 2 * rank + xh; pn = 4 * xq + idx - 3; }
        else break;
        pm = RFL(pm); pn = RFL(pn);
        int tz = threadIdx.x; asm volatile("" : "+v"(tz));
        const int ln = tz & 63, fr = ln & 15, fq = ln >> 4;
        gu32* fa = ctl + W_LAND + (pm * 8 + (ln & 7)) * 32 + (ln >> 3);
        gu32* fb = ctl + W_LAND + ((16 + pn) * 8 + (ln & 7)) * 32 + (ln >> 3);
        const u64 own = 0x0101010101010101ull << rank;
        u64 ra = 0, rb = 0;
        bool bad = false;
#define STEP_WAIT(need, last) do { if (!bad && ((last) || (ra & rb & (need)) != (need))) bad = wave_wait(fa, fb, (need), ra, rb, call * 4u, tmo, call, ln); } while (0)
        if (wid == 0) { STEP_WAIT(own, false); WAIT_V(0); }
        CFENCE; BAR; CFENCE;

        const char* cA = myws + (slot + O_A + (unsigned)(pm * NKT) * TILE_B);
        const char* cB = myws + (slot + O_B + (unsigned)(pn * NKT) * TILE_B);
        const unsigned vst = (unsigned)tz * 16u;
        const unsigned ldsw = (unsigned)wid * 1024u;
        const int a0 = (wr * 64 + fr) * 128 + ((fq ^ (fr >> 1)) * 16), a1 = a0 ^ 64;
        const int b0 = fq * 2048 + (wc * 16 + fr) * 16;
#define SA(b, h) (((b) * 2 + (h)) * HALF_B)
#define SB(b, h) ((4 + (b) * 2 + (h)) * HALF_B)
#define STAGE(bufoff, gbase) do { _Pragma("unroll") for (int _i = 0; _i < 2; ++_i) \
        __builtin_amdgcn_global_load_lds((const unsigned*)((gbase) + vst + _i * 8192), (LAS unsigned*)(lds + (bufoff) + ldsw + _i * 8192), 16, 0, 17); } while (0)
#define LDA(dst, b, h) do { _Pragma("unroll") for (int m = 0; m < 4; ++m) { dst[m][0] = *(const LAS h8*)(lds + SA(b, h) + a0 + m * 2048); dst[m][1] = *(const LAS h8*)(lds + SA(b, h) + a1 + m * 2048); } } while (0)
#define LDB(dst, b, h) do { _Pragma("unroll") for (int n = 0; n < 2; ++n) _Pragma("unroll") for (int k = 0; k < 2; ++k) dst[n][k] = *(const LAS h8*)(lds + SB(b, h) + b0 + k * 8192 + n * 1024); } while (0)
#define MMA(ai, bj, At, Bt) do { __builtin_amdgcn_s_setprio(1); _Pragma("unroll") for (int m = 0; m < 4; ++m) _Pragma("unroll") for (int n = 0; n < 2; ++n) _Pragma("unroll") for (int k = 0; k < 2; ++k) \
        acc[ai][bj][m][n] = __builtin_amdgcn_mfma_f32_16x16x32_f16(At[m][k], Bt[n][k], acc[ai][bj][m][n], 0, 0, 0); __builtin_amdgcn_s_setprio(0); } while (0)
        f4 acc[2][2][4][2];
#pragma unroll
        for (int a = 0; a < 2; ++a)
#pragma unroll
            for (int b = 0; b < 2; ++b)
#pragma unroll
                for (int m = 0; m < 4; ++m)
#pragma unroll
                    for (int n = 0; n < 2; ++n) acc[a][b][m][n] = (f4){0.f, 0.f, 0.f, 0.f};
        h8 At[4][2], B0[2][2], B1[2][2];
        STAGE(SB(0, 0), cB); STAGE(SA(0, 0), cA); STAGE(SB(0, 1), cB + HALF_B); STAGE(SA(0, 1), cA + HALF_B);
        if (wr == 1) BAR;
        WAIT_V(4); BAR;
        STAGE(SB(1, 0), cB + TILE_B); STAGE(SA(1, 0), cA + TILE_B); STAGE(SB(1, 1), cB + TILE_B + HALF_B);
        WAIT_V(6); BAR;
        int jn = 0, bnd = 8;
        for (int t = 0; t < NKT; t += 2) {
            const int t2 = (t + 2) & (NKT - 1);
            const char* a1p = cA + (size_t)(t + 1) * TILE_B;
            const char* a2 = cA + (size_t)t2 * TILE_B; const char* b2 = cB + (size_t)t2 * TILE_B;
            const char* a3 = a2 + TILE_B; const char* b3 = b2 + TILE_B;
            if (t + 3 >= bnd && jn < 8) {
                if (wid == 0) {
                    STEP_WAIT((0xFFull << (8 * jn)) & ~own, jn == 7);
                    if (jn == 7 && ln == 0) lctl[L_BAD] = bad ? 1u : 0u;
                }
                CFENCE; BAR; CFENCE;
                ++jn; bnd += 7;
            }
            LDB(B0, 0, 0); SCHED; LDA(At, 0, 0); STAGE(SA(1, 1), a1p + HALF_B);
            WAIT_L(8); BAR; WAIT_L(0); MMA(0, 0, At, B0); BAR; SCHED;
            LDB(B1, 0, 1); STAGE(SB(0, 0), b2);
            BAR; WAIT_L(0); MMA(0, 1, At, B1); BAR;
            LDA(At, 0, 1); STAGE(SA(0, 0), a2);
            BAR; WAIT_L(0); MMA(1, 0, At, B0); BAR; SCHED;
            STAGE(SB(0, 1), b2 + HALF_B);
            WAIT_V(6); BAR; MMA(1, 1, At, B1); BAR;
            LDB(B0, 1, 0); SCHED; LDA(At, 1, 0); STAGE(SA(0, 1), a2 + HALF_B);
            WAIT_L(8); BAR; WAIT_L(0); MMA(0, 0, At, B0); BAR; SCHED;
            LDB(B1, 1, 1); STAGE(SB(1, 0), b3);
            BAR; WAIT_L(0); MMA(0, 1, At, B1); BAR;
            LDA(At, 1, 1); STAGE(SA(1, 0), a3);
            BAR; WAIT_L(0); MMA(1, 0, At, B0); BAR; SCHED;
            STAGE(SB(1, 1), b3 + HALF_B);
            WAIT_V(6); BAR; MMA(1, 1, At, B1); BAR;
        }
        WAIT_V(0);
        if (wr == 0) BAR;
        CFENCE; BAR; CFENCE;

        const unsigned dest = ((unsigned)pm >> 1) & 7u;
        int ty = threadIdx.x; asm volatile("" : "+v"(ty));
        const int l2 = ty & 63;
        const unsigned vo = (unsigned)((l2 >> 4) * 4 * (NG * 4) + (l2 & 15) * 16);
        const bool poison = lctl[L_BAD] != 0u;
        const float qnan = __builtin_nanf("");
#define ROW4(ai, m, r) (poison ? (f4){qnan, qnan, qnan, qnan} : (f4){gelu(acc[ai][0][m][0][r]), gelu(acc[ai][0][m][1][r]), gelu(acc[ai][1][m][0][r]), gelu(acc[ai][1][m][1][r])})
        if (dest == (unsigned)rank) {
            const char* ob = (const char*)P.out + ((size_t)((pm & 1) * 256 + wr * 64) * NG + rank * NS + pn * 256 + wc * 64) * 4;
#pragma unroll
            for (int ai = 0; ai < 2; ++ai)
#pragma unroll
                for (int m = 0; m < 4; ++m)
#pragma unroll
                    for (int r = 0; r < 4; ++r) { const f4 v = ROW4(ai, m, r); st4(vo, __builtin_bit_cast(u4, v), ob + (size_t)(ai * 128 + m * 16 + r) * (NG * 4)); }
        } else {
            const int tile = (pm & 1) * 16 + pn;
            const unsigned zo = (O_OUT - O_X) + (unsigned)((dest * 32 + tile) * 32 + wid) * UNIT_B;
            gu32* uf = (gu32*)P.ws[dest] + W_UNIT + (rank * 32 + tile) * 32;
            LAS unsigned char* z = lds + wid * 16384;
            unsigned hprev = 0u;
#define ANNOUNCE(gg) do { const int ig = round * 4 + (gg); unsigned old = 0u; \
                if (l2 == 0) { lctl[L_HD + 104 + ig * 8 + wid] = hprev; old = __hip_atomic_fetch_add(lctl + L_CNT + 26 + ig, 1u, __ATOMIC_RELAXED, __HIP_MEMORY_SCOPE_WORKGROUP); } \
                if ((unsigned)RFL(old) == 7u && l2 < 8) { CFENCE; __hip_atomic_store(uf + (gg) * 8 + l2, lctl[L_HD + 104 + ig * 8 + l2], __ATOMIC_RELAXED, __HIP_MEMORY_SCOPE_SYSTEM); } } while (0)
#pragma unroll
            for (int g = 0; g < 4; ++g) {
                const int ai = g >> 1;
                const char* zp = myws + (slot + O_X + zo + g * 8 * UNIT_B);
                unsigned sm[8], mp = 0u, mn = 0u; u4 E[2];
#pragma unroll
                for (int i = 0; i < 4; ++i) {
                    u4 lo;
#pragma unroll
                    for (int e = 0; e < 2; ++e) {
                        const int q = 2 * i + e, m = (g & 1) * 2 + (q >> 2), r = q & 3;
                        const u4 u = __builtin_bit_cast(u4, ROW4(ai, m, r));
                        lo[2 * e] = PERM(u.y, u.x, 0x05040100u); lo[2 * e + 1] = PERM(u.w, u.z, 0x05040100u);
                        const unsigned h01 = PERM(u.y, u.x, 0x07060302u), h23 = PERM(u.w, u.z, 0x07060302u);
                        sm[q] = (PERM(h23, h01, 0x06040200u) & 0x7f7f7f7fu) | (PERM(h23, h01, 0x07050301u) & 0x80808080u);
                        E[q >> 2][q & 3] = PERM(h23 << 1, h01 << 1, 0x07050301u);
                        mp = pmax_i16(mp, pmax_i16(h01, h23)); mn = pmax_u16(mn, pmax_u16(h01, h23));
                    }
                    *(LAS u4*)(z + 8192 + i * 1024 + l2 * 16) = lo;
                    st4((unsigned)l2 * 16u, lo, zp + i * 1024);
                }
                if (g >= 2) { WAIT_V(4); ANNOUNCE(g - 1); }
                *(LAS u4*)(z + 2048 + l2 * 16) = (u4){0u, 0u, 0u, 0u};
                const unsigned nm = wave_max(max(mn & 0xffffu, mn >> 16)), bn = (nm & 0x8000u) ? (nm >> 7) & 0xffu : 0u, bp = wave_max(max((mp >> 7) & 0xffu, mp >> 23));
                const unsigned bp4 = bp * 0x01010101u, bn4 = bn * 0x01010101u;
                unsigned c[8], m = 0u, cnt = 0u, hd = call << 19; u4 D[2]; bool far = false;
#pragma unroll
                for (int q = 0; q < 8; ++q) {
                    const unsigned mk = bytemask(sm[q] & 0x80808080u), d = ((bn4 & mk) | (bp4 & ~mk)) - E[q >> 2][q & 3];
                    const unsigned es = (((d & 0x7f7f7f7fu) + 0x79797979u) | d) & 0x80808080u;
                    c[q] = (d | bytemask(es)) & 0x07070707u; D[q >> 2][q & 3] = d;
                    m |= es >> (7 - q);
                }
                *(LAS u4*)(z + l2 * 16) = D[0]; *(LAS u4*)(z + 1024 + l2 * 16) = D[1];
                CFENCE;
                far = __popc(m) > 16;
                if (!__any(far)) {
                    const unsigned first = lanes_below((unsigned)__popc(m), cnt);
                    u64 nb = 0ull; unsigned d = z[h_byte(__builtin_ctz(m | 0x80000000u), l2)];
                    for (unsigned sh = 0u; __any(m != 0u); sh += 4u) {
                        if (m != 0u) { far |= d > 22u; nb |= (u64)((d - 7u) & 15u) << sh; }
                        m &= m - 1u;
                        d = z[h_byte(__builtin_ctz(m | 0x80000000u), l2)];
                    }
                    list_put((LAS unsigned*)(z + 2048), first, nb);
                }
                CFENCE;
                if (cnt <= 640u && !__any(far)) {
                    hd |= (((cnt + 127u) >> 8) << 16) | (bp << 8) | bn;
                    st4((unsigned)l2 * 16u, (u4){sm[0], sm[1], sm[2], sm[3]}, zp + 4096); st4((unsigned)l2 * 16u, (u4){sm[4], sm[5], sm[6], sm[7]}, zp + 5120);
                    st1((unsigned)l2 * 4u, c[0] | (c[1] << 3) | (c[2] << 6 & 0xc0c0c0c0u), zp + 6144);
                    st1((unsigned)l2 * 4u, (c[2] >> 2 & 0x01010101u) | (c[3] << 1) | (c[4] << 4) | (c[5] << 7 & 0x80808080u), zp + 6400);
                    st1((unsigned)l2 * 4u, (c[5] >> 1 & 0x03030303u) | (c[6] << 2) | (c[7] << 5), zp + 6656);
                    if (l2 < 16) st1((unsigned)l2 * 4u, *(const LAS unsigned*)(z + 2048 + l2 * 4), myws + (slot + O_LIN + ((zo >> 13) + g * 8) * 64u));
                    if (cnt > 128u) st1((unsigned)l2 * 4u, *(const LAS unsigned*)(z + 2112 + l2 * 4), zp + 6912);
                } else {
                    hd |= H_RAW;
                    WAIT_V(0);
#pragma unroll
                    for (int i = 0; i < 4; ++i) {
                        const u4 lo = *(const LAS u4*)(z + 8192 + i * 1024 + l2 * 16);
#pragma unroll
                        for (int e = 0; e < 2; ++e) {
                            const int q = 2 * i + e;
                            const unsigned ee = E[q >> 2][q & 3], tp = (sm[q] & 0x80808080u) | ((ee >> 1) & 0x7f7f7f7fu), md = (sm[q] & 0x7f7f7f7fu) | ((ee << 7) & 0x80808080u);
                            const unsigned h01 = PERM(tp, md, 0x05010400u), h23 = PERM(tp, md, 0x07030602u);
                            st4((unsigned)l2 * 16u, (u4){PERM(h01, lo[2 * e], 0x05040100u), PERM(h01, lo[2 * e], 0x07060302u), PERM(h23, lo[2 * e + 1], 0x05040100u), PERM(h23, lo[2 * e + 1], 0x07060302u)},
                                zp + q * 1024);
                        }
                    }
                }
                hprev = hd;
                if (g == 0 || g == 3) { WAIT_V(0); ANNOUNCE(g); }
            }
        }
        WAIT_V(0); WAIT_L(0); BAR; CFENCE;
    }
}

}

extern "C" void kernel_launch(int rank, int nranks, ncclComm_t comm, hipStream_t stream,
                              void* const* d_in, const int* in_sizes, int n_in,
                              void* d_out, int out_size, const int* in_dtypes, int out_dtype,
                              void* const* d_out_all, void* const* d_ws_all, size_t ws_bytes) {
    (void)comm; (void)in_dtypes; (void)out_dtype; (void)d_out_all;
    if (nranks != NR || (unsigned)rank >= (unsigned)NR || n_in < 2 || ws_bytes < WS_NEED || in_sizes[0] != MG * KS || in_sizes[1] != KS * NG || out_size != MS * NG) {
        fprintf(stderr, "kernel_launch: unexpected problem shape\n");
        return;
    }
    Params P;
    P.x = (const float*)d_in[0]; P.w = (const float*)d_in[1];
    for (int p = 0; p < NR; ++p) P.ws[p] = (char*)d_ws_all[p];
    for (int p = 0; p < NR; ++p) P.peer_ws[p] = P.ws[(rank + 1 + p) & 7];
    P.myws = P.ws[rank]; P.out = (float*)d_out;
    P.rank = rank; P.pad = 0;
    hipLaunchKernelGGL(gemm_rs_kernel, dim3(NWG), dim3(NTHR), 0, stream, P);
    HIP_CHECK(hipGetLastError());
}
```

```cpp
#include <hip/hip_runtime.h>
#include <rccl/rccl.h>
#include <cstdio>

#define HIP_CHECK(expr) do { hipError_t _e = (expr); if (_e != hipSuccess) { \
    fprintf(stderr, "HIP error %s at %s:%d\n", hipGetErrorString(_e), __FILE__, __LINE__); \
    return; } } while (0)

namespace {

typedef _Float16 h8 __attribute__((ext_vector_type(8)));
typedef _Float16 h2 __attribute__((ext_vector_type(2)));
typedef float f4 __attribute__((ext_vector_type(4)));
typedef unsigned u4 __attribute__((ext_vector_type(4)));
typedef short s2v __attribute__((ext_vector_type(2)));
typedef unsigned short us2v __attribute__((ext_vector_type(2)));
typedef unsigned long long u64;
typedef __attribute__((address_space(1))) unsigned gu32;
typedef __attribute__((address_space(1))) f4 gf4;
#define LAS __attribute__((address_space(3)))

constexpr int NR = 8;
constexpr int MG = 4096, KG = 4096, NG = 32768;
constexpr int KS = KG / NR;
constexpr int NS = NG / NR;
constexpr int MS = MG / NR;
constexpr int NWG = 256, NTHR = 512;
constexpr int NCOM = 56;
constexpr int NX = 4;
constexpr int PUSHX = 1;
constexpr unsigned NSGL = 128;
constexpr int POLL_SLEEP = 16;
constexpr int TILE_B = 256 * 64 * 2;
constexpr int HALF_B = TILE_B / 2;
constexpr int NKT = KG / 64;
constexpr unsigned NI_IN = 1024, NI = 2048;
constexpr unsigned OP_B = 16u * NKT * TILE_B;
constexpr unsigned UNIT_B = 8192;
constexpr unsigned CTL_B = 1u << 20;
constexpr unsigned O_A = 0, O_B = OP_B, O_X = 2 * OP_B, O_OUT = 3 * OP_B, O_LIN = O_OUT + NR * 32 * 32 * UNIT_B,
                   SLOT_B = O_LIN + (1u << 20);
constexpr size_t WS_NEED = (size_t)CTL_B + 2 * (size_t)SLOT_B;
constexpr int W_TMO = 0;
constexpr int W_LAND = 32;
constexpr int W_AVAIL = W_LAND + 256 * 32;
constexpr int W_UNIT = W_AVAIL + 512 * 32;
constexpr unsigned H_RAW = 1u << 18;
constexpr int W_TICK = W_UNIT + 256 * 32;
constexpr int W_CALL = W_TICK + 8 * 32;
static_assert((W_CALL + NWG * 32) * 4 <= CTL_B, "control layout");
constexpr u64 DEADLINE = 40000000ull;

constexpr int LDS_CTL = 8 * HALF_B;
constexpr int LDS_LIN = LDS_CTL + 1024, LDS_BYTES = LDS_LIN + 2048;
constexpr int L_BAD = 0, L_CNT = 2, L_HD = 64;

struct Params {
    const float* x; const float* w; char* myws; float* out;
    int rank; int pad;
    char* ws[NR];
    char* peer_ws[NR];
};

#define WAIT_V(n) asm volatile("s_waitcnt vmcnt(" #n ")" ::: "memory")
#define WAIT_L(n) asm volatile("s_waitcnt lgkmcnt(" #n ")" ::: "memory")
#define BAR __builtin_amdgcn_s_barrier()
#define SCHED __builtin_amdgcn_sched_barrier(0)
#define CFENCE asm volatile("" ::: "memory")
#define RFL(x) __builtin_amdgcn_readfirstlane(x)
#define PERM(hi, lo, sel) __builtin_amdgcn_perm((hi), (lo), (sel))
template <int N> __device__ __forceinline__ void wait_v() { asm volatile("s_waitcnt vmcnt(%0)" :: "i"(N) : "memory"); }

__device__ __forceinline__ void st4(unsigned voff, u4 v, const void* sbase) {
    asm volatile("global_store_dwordx4 %0, %1, %2 sc0 sc1\n\ts_nop 1" :: "v"(voff), "v"(v), "s"(sbase) : "memory");
}
__device__ __forceinline__ void st1(unsigned voff, unsigned v, const void* sbase) {
    asm volatile("global_store_dword %0, %1, %2 sc0 sc1" :: "v"(voff), "v"(v), "s"(sbase) : "memory");
}
__device__ __forceinline__ void dma(unsigned voff, const void* sbase, unsigned ldsoff) {
    unsigned keep;
    asm volatile("s_mov_b32 %0, m0\n\ts_mov_b32 m0, %3\n\ts_nop 0\n\tglobal_load_lds_dwordx4 %1, %2 sc0 sc1\n\ts_mov_b32 m0, %0"
                 : "=&s"(keep) : "v"(voff), "s"(sbase), "s"(ldsoff) : "memory");
}
__device__ __forceinline__ void dma_part(unsigned voff, const void* sbase, unsigned ldsoff, unsigned lanes) {
    unsigned keep; u64 ex;
    asm volatile("s_mov_b32 %0, m0\n\ts_mov_b32 m0, %4\n\ts_mov_b64 %1, exec\n\ts_mov_b64 exec, %5\n\tglobal_load_lds_dwordx4 %2, %3 sc0 sc1\n\ts_mov_b64 exec, %1\n\ts_mov_b32 m0, %0"
                 : "=&s"(keep), "=&s"(ex) : "v"(voff), "s"(sbase), "s"(ldsoff), "s"(~0ull >> (64u - lanes)) : "memory");
}
__device__ __forceinline__ unsigned pmax_i16(unsigned a, unsigned b) { return __builtin_bit_cast(unsigned, __builtin_elementwise_max(__builtin_bit_cast(s2v, a), __builtin_bit_cast(s2v, b))); }
__device__ __forceinline__ unsigned pmax_u16(unsigned a, unsigned b) { return __builtin_bit_cast(unsigned, __builtin_elementwise_max(__builtin_bit_cast(us2v, a), __builtin_bit_cast(us2v, b))); }
__device__ __forceinline__ unsigned bytemask(unsigned s) { return (s - (s >> 7)) | s; }
__device__ __forceinline__ unsigned pk(float a, float b) {
    h2 r; r.x = (_Float16)a; r.y = (_Float16)b; return __builtin_bit_cast(unsigned, r);
}
__device__ __forceinline__ float gelu(float y) {
    const float k1 = -2.0f * 0.7978845608028654f * 1.4426950408889634f, k2 = k1 * 0.044715f;
    const float e = __builtin_amdgcn_exp2f(y * (k1 + k2 * y * y));
    return y * __builtin_amdgcn_rcpf(1.0f + e);
}

__device__ __forceinline__ bool wave_wait(gu32* pa, gu32* pb, u64 need, u64& ra, u64& rb, unsigned target, gu32* tmo, unsigned call, int lane) {
    const u64 t0 = __builtin_amdgcn_s_memrealtime();
    for (;;) {
        const unsigned va = __hip_atomic_load(pa, __ATOMIC_RELAXED, __HIP_MEMORY_SCOPE_SYSTEM);
        const unsigned vb = __hip_atomic_load(pb, __ATOMIC_RELAXED, __HIP_MEMORY_SCOPE_SYSTEM);
        const unsigned vt = __hip_atomic_load(tmo, __ATOMIC_RELAXED, __HIP_MEMORY_SCOPE_AGENT);
        if ((unsigned)RFL(vt) == call) return true;
        ra = __ballot((int)(va - target) >= 0); rb = __ballot((int)(vb - target) >= 0);
        if ((ra & rb & need) == need) return false;
        if (__builtin_amdgcn_s_memrealtime() - t0 > DEADLINE) {
            if (lane == 0) __hip_atomic_store(tmo, call, __ATOMIC_RELAXED, __HIP_MEMORY_SCOPE_AGENT);
            return true;
        }
        __builtin_amdgcn_s_sleep(POLL_SLEEP);
    }
}
__device__ __forceinline__ bool header_wait(gu32* ctl, int word, unsigned& h0, unsigned& h1, unsigned call, int lane) {
    const u64 t0 = __builtin_amdgcn_s_memrealtime();
    for (;;) {
        const unsigned v = __hip_atomic_load(ctl + (lane < 2 ? word + lane : W_TMO), __ATOMIC_RELAXED, __HIP_MEMORY_SCOPE_SYSTEM);
        h0 = RFL(v); h1 = __builtin_amdgcn_readlane(v, 1);
        if ((unsigned)__builtin_amdgcn_readlane(v, 2) == call) return true;
        if ((h0 >> 19) == (call & 0x1fffu) && (h1 >> 19) == (call & 0x1fffu)) return false;
        if (__builtin_amdgcn_s_memrealtime() - t0 > DEADLINE) {
            if (lane == 0) __hip_atomic_store(ctl + W_TMO, call, __ATOMIC_RELAXED, __HIP_MEMORY_SCOPE_AGENT);
            return true;
        }
        __builtin_amdgcn_s_sleep(POLL_SLEEP / 2);
    }
}

__device__ __forceinline__ void conv_of(int c, int me, bool& isA, int& dest, int& panel, int& j) {
    int b = -1;
    isA = false; dest = me;
    if (c < 200) { j = c >= 100; const int g = c - j * 100; isA = g < 16; panel = g; b = g - 16; }
    else if (c < 296) { isA = true; j = 2 + ((c - 200) >> 4); panel = (c - 200) & 15; }
    else if (c < 400) { j = (c - 296) / 12; panel = c - 296 - j * 12; }
    else if (c < 904) { j = (c - 400) / 84; b = c - 400 - j * 84; j += 2; }
    else { const int q = c - 904; j = q >> 5; panel = 12 + ((q >> 3) & 3); dest = q & 7; }
    if (b >= 0) { panel = b / 7; dest = (me + 1 + b - panel * 7) & 7; }
}
__device__ __forceinline__ void image_of(int i, int me, bool& isA, int& panel, int& j) {
    constexpr int J0 = 8 - NX, I0 = 26 * J0, I1 = I0 + 18 * NX;
    int g = 14;
    if (i < I0) { j = i / 26; g = i - j * 26; panel = g - 14; }
    else if (i < I1) { j = (i - I0) / 18; g = i - I0 - j * 18; j += J0; panel = 3 * (g - 14); }
    else if (i < 208) { j = J0 + ((i - I1) >> 3); panel = (((i - I1) >> 1) & 3) * 3 + 1 + ((i - I1) & 1); }
    else if (i < 240) { j = (i - 208) >> 2; panel = 12 + (i & 3); }
    else { j = (i - 240) >> 1; g = (i & 1) - 2; }
    isA = g < 14;
    if (isA) panel = (2 * me + 2 + g) & 15;
}
__device__ __forceinline__ int late_peer(int x, int k) { return (k + 3 * x) % 7; }
__device__ __forceinline__ unsigned late_order(int a) {
    unsigned o = 0u; int n = 0;
    for (int x = 0; x < 8; ++x) if ((a + 7 * 4 - 3 * x) % 7 < 4) o |= (unsigned)x << (4 * n++);
    for (int x = 0; x < 8; ++x) if ((a + 7 * 4 - 3 * x) % 7 >= 4) o |= (unsigned)x << (4 * n++);
    return o;
}
__device__ __forceinline__ void unit_of(int u, unsigned late, int& tile, int& k) {
    const int h = (u >> 3) & 1;
    if (u < 256) { k = (u >> 6) * 8 + (u & 7); tile = h * 16 + 3 * ((u >> 4) & 3); }
    else if (u < 768) { k = ((u - 256) >> 7) * 8 + (u & 7); tile = h * 16 + ((u >> 5) & 3) * 3 + 1 + ((u >> 4) & 1); }
    else {
        const int t = u < 896 ? (u - 768) >> 5 : 4 + ((u >> 3) & 3), x = (late >> t * 4) & 7;
        k = u < 896 ? u & 31 : ((u - 896) >> 5) * 8 + (u & 7); tile = (x & 1) * 16 + 12 + (x >> 1);
    }
}
__device__ __forceinline__ void job_loads(f4 (&L)[8], const char* x, const char* w, bool isA, int dest, int panel, int j, int e, unsigned vxe, unsigned vl) {
    const char* b = isA ? x + ((size_t)(panel * 256 + e * 32) * KS + j * 64) * 4 : w + ((size_t)(j * 64 + e * 8) * NG + dest * NS + panel * 256) * 4;
    const unsigned ve = isA ? vxe : vl, vo = isA ? (vxe ^ 128u) : vl;
    const unsigned s2 = isA ? 8 * KS * 4 : 2 * NG * 4, s1 = isA ? 16 : NG * 4;
#pragma unroll
    for (int r = 0; r < 8; ++r) L[r] = *(const gf4*)(b + ((r >> 1) * s2 + (r & 1) * s1) + ((r & 2) ? vo : ve));
}
__device__ __forceinline__ void job_pack(const f4 (&L)[8], u4 (&Q)[4], bool isA) {
    if (isA) {
#pragma unroll
        for (int u = 0; u < 4; ++u) { Q[u].x = pk(L[2 * u].x, L[2 * u].y); Q[u].y = pk(L[2 * u].z, L[2 * u].w); Q[u].z = pk(L[2 * u + 1].x, L[2 * u + 1].y); Q[u].w = pk(L[2 * u + 1].z, L[2 * u + 1].w); }
    } else {
#pragma unroll
        for (int t = 0; t < 4; ++t) { Q[t].x = pk(L[0][t], L[1][t]); Q[t].y = pk(L[2][t], L[3][t]); Q[t].z = pk(L[4][t], L[5][t]); Q[t].w = pk(L[6][t], L[7][t]); }
    }
}

struct Com {
    gu32* ctl; char* me_ws; const char* src_ws; char* out; LAS unsigned char* ring; LAS unsigned char* lines; LAS unsigned* word;
    unsigned slot, call, vl, vo, late; int rank, src, lane;
};
__device__ __forceinline__ int item_word(const Com& C, unsigned n) {
    if (n < NI_IN) { bool isA; int panel, j; image_of(n >> 2, C.rank, isA, panel, j); return W_AVAIL + ((((C.call & 1) * 8 + C.src) * 2 + (isA ? 0 : 1)) * 16 + panel) * 32 + j * 4 + (n & 3); }
    if (n < NI) { int tile, k; unit_of(n - NI_IN, C.late, tile, k); return W_UNIT + (C.src * 32 + tile) * 32 + k; }
    return W_TMO;
}
__device__ __forceinline__ int h_byte(int idx, int lane) {
    return ((idx >> 5) * 2 + ((idx >> 2) & 1)) * 1024 + lane * 16 + (idx & 3) * 4 + ((idx >> 3) & 3);
}
#define DPP(x, ctrl, rmask) (unsigned)__builtin_amdgcn_update_dpp(0, (int)(x), (ctrl), (rmask), 0xf, false)
__device__ __forceinline__ unsigned wave_sum(unsigned x) {
    x += DPP(x, 0x111, 0xf); x += DPP(x, 0x112, 0xf); x += DPP(x, 0x114, 0xf); x += DPP(x, 0x118, 0xf);
    x += DPP(x, 0x142, 0xa); x += DPP(x, 0x143, 0xc);
    return __builtin_amdgcn_readlane(x, 63);
}
__device__ __forceinline__ unsigned wave_max(unsigned x) {
    x = max(x, DPP(x, 0x111, 0xf)); x = max(x, DPP(x, 0x112, 0xf)); x = max(x, DPP(x, 0x114, 0xf)); x = max(x, DPP(x, 0x118, 0xf));
    x = max(x, DPP(x, 0x142, 0xa)); x = max(x, DPP(x, 0x143, 0xc));
    return __builtin_amdgcn_readlane(x, 63);
}
__device__ __forceinline__ unsigned lane_rank(u64 m) { return __builtin_amdgcn_mbcnt_hi((unsigned)(m >> 32), __builtin_amdgcn_mbcnt_lo((unsigned)m, 0u)); }
__device__ __forceinline__ unsigned lanes_below(unsigned c, unsigned& total) {
    unsigned p = 0u; total = 0u;
#pragma unroll
    for (int b = 0; b < 5; ++b) { const u64 m = __ballot((c >> b) & 1u); p += lane_rank(m) << b; total += (unsigned)__popcll(m) << b; }
    return p;
}
__device__ __forceinline__ void list_put(LAS unsigned* l, unsigned first, u64 nb) {
    const unsigned sh = (first & 7u) * 4u; l += first >> 3;
    __hip_atomic_fetch_or(l, (unsigned)(nb << sh), __ATOMIC_RELAXED, __HIP_MEMORY_SCOPE_WORKGROUP);
    __hip_atomic_fetch_or(l + 1, (unsigned)((nb << sh) >> 32), __ATOMIC_RELAXED, __HIP_MEMORY_SCOPE_WORKGROUP);
    __hip_atomic_fetch_or(l + 2, (unsigned)((nb >> 1) >> (63u - sh)), __ATOMIC_RELAXED, __HIP_MEMORY_SCOPE_WORKGROUP);
}
__device__ __forceinline__ u64 list_get(const LAS unsigned char* head, LAS unsigned char* more, int skip, unsigned cnt, int lane) {
    if (lane < 16) *(LAS unsigned*)(more - 64 + lane * 4) = *(const LAS unsigned*)(head + lane * 4);
    CFENCE;
    unsigned total; const unsigned first = lanes_below(cnt, total), sh = (first & 7u) * 4u;
    const LAS unsigned* l = (const LAS unsigned*)(more - 64 + skip) + min(first >> 3, 80u);
    return ((((u64)l[1] << 32) | l[0]) >> sh) | (((u64)l[2] << 1) << (63u - sh));
}
__device__ __forceinline__ unsigned piece_off(bool isA, int q, int p) {
    return isA ? (unsigned)(q * 8192 + p * 1024) : (unsigned)((2 * q + (p >> 2)) * 2048 + ((p >> 1) & 1) * HALF_B + (p & 1) * 1024);
}
#define SETTLE() do { if (owed) { WAIT_V(0); if (C.lane == 0) __hip_atomic_fetch_add(C.ctl + owed, 1u, __ATOMIC_RELAXED, __HIP_MEMORY_SCOPE_AGENT); owed = 0; } } while (0)
__device__ __forceinline__ void put_quarter(const Com& C, unsigned n, unsigned hd, LAS unsigned char* z, const LAS unsigned char* head, int& owed) {
    bool isA; int panel, j; image_of(n >> 2, C.rank, isA, panel, j);
    const char* db = C.me_ws + (C.slot + (isA ? O_A : O_B) + (unsigned)((panel * NKT + 8 + ((C.src - C.rank - 1) & 7) + j * 7) * TILE_B));
    LAS unsigned char* zl = z + C.lane * 16;
    if (hd & H_RAW) {
        SETTLE();
#pragma unroll
        for (int r = 0; r < 8; ++r) st4(C.vl, *(const LAS u4*)(zl + r * 1024), db + piece_off(isA, n & 3, r));
    } else {
        u4 lo[4], pz[3], h[4];
#pragma unroll
        for (int r = 0; r < 4; ++r) lo[r] = *(const LAS u4*)(zl + r * 1024);
#pragma unroll
        for (int r = 0; r < 3; ++r) pz[r] = *(const LAS u4*)(zl + 4096 + r * 1024);
        const unsigned bq = ((((hd >> 8) & 31u) << 2) | 0x80u) * 0x01010101u;
        unsigned m0 = 0u, m1 = 0u;
#pragma unroll
        for (int i = 0; i < 16; ++i) {
            const int g = i >> 2, k = i & 3;
            const unsigned f = g < 3 ? pz[g][k] : ((pz[0][k] >> 6) & 0x03030303u) | ((pz[1][k] >> 4) & 0x0c0c0c0cu) | ((pz[2][k] >> 2) & 0x30303030u);
            const unsigned c = f & 0x1c1c1c1cu, es = ((c + 0x04040404u) << 2) & 0x80808080u;
            h[g][k] = ((f << 2) & 0x80808080u) | ((bq - c) & 0x7c7c7c7cu & ~bytemask(es)) | (f & 0x03030303u);
            if (i < 8) m0 |= es >> (7 - i); else m1 |= es >> (15 - i);
        }
        asm volatile("" : "+v"(lo[0]), "+v"(lo[1]), "+v"(lo[2]), "+v"(lo[3]), "+v"(m0), "+v"(m1) :: "memory");
#pragma unroll
        for (int r = 0; r < 4; ++r) *(LAS u4*)(zl + r * 1024) = h[r];
        u64 m = ((u64)m1 << 32) | m0, nb = list_get(head, z + 7168, 0, (unsigned)__popcll(m), C.lane);
        while (__any(m != 0ull)) {
            if (m != 0ull) {
                const int o = h_byte(__builtin_ctzll(m), C.lane);
                const unsigned q4 = (unsigned)nb & 15u, e = (((hd >> 8) & 31u) + 2u - (q4 < 2u ? q4 : q4 + 7u)) & 31u;
                __hip_atomic_fetch_or((LAS unsigned*)(z + (o & ~3)), e << (2 + 8 * (o & 3)), __ATOMIC_RELAXED, __HIP_MEMORY_SCOPE_WORKGROUP);
                m &= m - 1ull; nb >>= 4;
            }
        }
        CFENCE;
#pragma unroll
        for (int r = 0; r < 4; ++r) h[r] = *(const LAS u4*)(zl + r * 1024);
        SETTLE();
#pragma unroll
        for (int p = 0; p < 8; ++p) {
            const unsigned ha = h[p >> 1][2 * (p & 1)], hb = h[p >> 1][2 * (p & 1) + 1], la = lo[p >> 1][2 * (p & 1)], lb = lo[p >> 1][2 * (p & 1) + 1];
            st4(C.vl, (u4){PERM(ha, la, 0x05010400u), PERM(ha, la, 0x07030602u), PERM(hb, lb, 0x05010400u), PERM(hb, lb, 0x07030602u)}, db + piece_off(isA, n & 3, p));
        }
    }
    owed = W_LAND + (((isA ? 0 : 16) + panel) * 8 + C.src) * 32 + j;
}
__device__ __forceinline__ const char* unit_rows(const Com& C, unsigned n) {
    int tile, k; unit_of(n - NI_IN, C.late, tile, k);
    const int g = k >> 3, w = k & 7;
    return C.out + ((size_t)((tile >> 4) * 256 + (g >> 1) * 128 + (w >> 2) * 64 + (g & 1) * 32) * NG + C.src * NS + (tile & 15) * 256 + (w & 3) * 64) * 4;
}
__device__ __forceinline__ void put_unit(const Com& C, unsigned n, unsigned hd, LAS unsigned char* z, const LAS unsigned char* head) {
    const char* ob = unit_rows(C, n);
    LAS unsigned char* zl = z + C.lane * 16;
    if (hd & H_RAW) {
#pragma unroll
        for (int q = 0; q < 8; ++q) st4(C.vo, *(const LAS u4*)(zl + q * 1024), ob + (size_t)((q >> 2) * 16 + (q & 3)) * (NG * 4));
        return;
    }
    u4 Z[6]; unsigned pc[2], es[8], cnt = 0u;
#pragma unroll
    for (int r = 0; r < 2; ++r) pc[r] = *(const LAS unsigned*)(z + 6144 + r * 256 + C.lane * 4);
#pragma unroll
    for (int r = 0; r < 6; ++r) Z[(r + 4) % 6] = *(const LAS u4*)(zl + ((r + 4) % 6) * 1024);
#pragma unroll
    for (int q = 0; q < 8; ++q) {
        es[q] = Z[4 + (q >> 2)][q & 3] & (pc[q >> 2] << (7 - 2 * (q & 3))) & (pc[q >> 2] << (6 - 2 * (q & 3))) & 0x80808080u;
        cnt += (unsigned)__popc(es[q]);
    }
    asm volatile("" : "+v"(cnt), "+v"(Z[0]), "+v"(Z[1]), "+v"(Z[2]), "+v"(Z[3]) :: "memory");
    u64 nb = list_get(head, z + 6656, 16, cnt, C.lane);
    const unsigned bp = (hd >> 8) & 0xffu, bn = hd & 0xffu, bp4 = bp * 0x01010101u, bn4 = bn * 0x01010101u;
    unsigned lut[4];
#pragma unroll
    for (int i = 0; i < 4; ++i) {
        lut[i] = 0u;
#pragma unroll
        for (int b = 0; b < 4; ++b) { const int v = 4 * i + b; lut[i] |= ((v < 9 ? bp + (v < 2 ? 2 - v : -2 - v) : bn + 6 - v) & 0xffu) << (8 * b); }
    }
    u4 E[2]; unsigned m15 = 0u;
#pragma unroll
    for (int q = 0; q < 8; ++q) {
        const unsigned sm = Z[4 + (q >> 2)][q & 3], t = sm & 0x80808080u, c = (pc[q >> 2] >> (2 * (q & 3))) & 0x03030303u, km = bytemask(es[q]), mk = bytemask(t);
        unsigned w = (unsigned)nb & 0xffffu; w = (w | (w << 8)) & 0x00ff00ffu; w = (w | (w << 4)) & 0x0f0f0f0fu;
        const unsigned v = PERM(0u, w, (es[q] >> 7) * 0x01010100u), v7 = v & 0x07070707u, hi = bytemask((v << 4) & 0x80808080u);
        nb >>= 4u * (unsigned)__popc(es[q]);
        E[q >> 2][q & 3] = ((((bn4 & mk) | (bp4 & ~mk)) - (c & ~km)) & ~km) | (((PERM(lut[3], lut[2], v7) & hi) | (PERM(lut[1], lut[0], v7) & ~hi)) & km);
        Z[4 + (q >> 2)][q & 3] = (sm ^ es[q]) | (((v + 0x07070707u) << 3) & es[q]);
        m15 |= (((v + 0x01010101u) << 3) & es[q]) >> (7 - q);
    }
    if (__any(m15 != 0u)) {
        *(LAS u4*)zl = E[0]; *(LAS u4*)(zl + 1024) = E[1]; *(LAS u4*)(zl + 2048) = Z[4]; *(LAS u4*)(zl + 3072) = Z[5];
        CFENCE;
        for (unsigned base = 0u;;) {
            const u64 act = __ballot(m15 != 0u);
            if (act == 0ull) break;
            if (m15 != 0u) {
                const unsigned x = *(const LAS unsigned short*)(head + ((base + lane_rank(act)) & 7u) * 2u);
                const int o = h_byte(__builtin_ctz(m15), C.lane);
                z[o] = (unsigned char)x; z[o + 2048] = (unsigned char)((z[o + 2048] & 0x7fu) | ((x >> 8) << 7));
                m15 &= m15 - 1u;
            }
            base += (unsigned)__popcll(act);
        }
        CFENCE;
        E[0] = *(const LAS u4*)zl; E[1] = *(const LAS u4*)(zl + 1024); Z[4] = *(const LAS u4*)(zl + 2048); Z[5] = *(const LAS u4*)(zl + 3072);
    }
#pragma unroll
    for (int q = 0; q < 8; ++q) {
        const unsigned e = E[q >> 2][q & 3], sm = Z[4 + (q >> 2)][q & 3];
        const unsigned tp = (sm & 0x80808080u) | ((e >> 1) & 0x7f7f7f7fu), md = (sm & 0x7f7f7f7fu) | ((e << 7) & 0x80808080u);
        const unsigned h01 = PERM(tp, md, 0x05010400u), h23 = PERM(tp, md, 0x07030602u), l01 = Z[q >> 1][2 * (q & 1)], l23 = Z[q >> 1][2 * (q & 1) + 1];
        st4(C.vo, (u4){PERM(h01, l01, 0x05040100u), PERM(h01, l01, 0x07060302u), PERM(h23, l23, 0x05040100u), PERM(h23, l23, 0x07060302u)},
            ob + (size_t)((q >> 2) * 16 + (q & 3)) * (NG * 4));
    }
}
#define LDS_LOAD(p) __hip_atomic_load((p), __ATOMIC_RELAXED, __HIP_MEMORY_SCOPE_WORKGROUP)
#define LDS_STORE(p, v) __hip_atomic_store((p), (v), __ATOMIC_RELAXED, __HIP_MEMORY_SCOPE_WORKGROUP)
constexpr unsigned IT_DEAD = 0x80000000u;
constexpr int PF = 2;
constexpr unsigned TP = (NI - NSGL) / 2, NT = TP + NSGL;
__device__ __forceinline__ unsigned first_item(unsigned t) { return t < TP ? 2 * t : t + TP; }
__device__ __forceinline__ void load_items(const Com& C, gu32* tick, unsigned tbase, unsigned cur, unsigned nxt) {
    const unsigned want = C.call & 0x1fffu, ring = (unsigned)(size_t)C.ring;
    unsigned k = 0u, nl = 0u, h0 = (want ^ 1u) << 19, h1 = 0u; bool pend = false, dead = false;
    while (cur < NT) {
        const unsigned n0 = first_item(cur), np = n0 & ~1u, n1 = cur < TP ? n0 + 1u : n0;
        const bool in = n0 < NI_IN;
        unsigned so;
        if (in) { bool isA; int panel, j; image_of(np >> 2, C.rank, isA, panel, j); so = (unsigned)((((isA ? C.src : C.rank) * 16 + panel) * 8 + j) * TILE_B) + (np & 3) * 8192; }
        else { int tile, u; unit_of(np - NI_IN, C.late, tile, u); so = (O_OUT - O_X) + (unsigned)((C.rank * 32 + tile) * 32 + u) * UNIT_B; }
        if (!dead && ((h0 >> 19) != want || (h1 >> 19) != want)) {
            if (pend) { WAIT_V(0); if (C.lane == 0) LDS_STORE(C.word, k); pend = false; }
            dead = header_wait(C.ctl, item_word(C, np), h0, h1, C.call, C.lane);
        }
        unsigned tv = 0; u64 fv = 0, ex;
        asm volatile("s_mov_b64 %1, exec\n\ts_mov_b64 exec, 1\n\tglobal_atomic_add %0, %2, %3, off sc0\n\ts_mov_b64 exec, %1"
                     : "+v"(tv), "=&s"(ex) : "v"(tick), "v"(nxt < NT ? 1u : 0u) : "memory");
        if (dead) { WAIT_V(0); asm volatile("" : "+v"(tv)); }
        const unsigned nw = (unsigned)item_word(C, first_item(nxt) & ~1u) * 4u, line = (nl & 3u) * 128u;
        for (unsigned n = n0; n <= n1 && !(dead && in); ++n) {
            const unsigned hd = (n & 1u) ? h1 : h0;
            while (k >= (unsigned)RFL(LDS_LOAD(C.word + 1)) + 4u) __builtin_amdgcn_s_sleep(1);
            if (C.lane == 0) { C.word[4 + 2 * (k & 3)] = n | ((line + (n & 1u) * 64u) << 16) | (dead ? IT_DEAD : 0u); C.word[5 + 2 * (k & 3)] = hd; }
            if (!dead) {
                const char* sb = C.src_ws + (C.slot + O_X + so + (n & 1u) * 8192u);
                const unsigned lo = ring + (k & 3) * 8192, more = min((hd >> 16) & 3u, 2u) * 8u;
                const bool raw = hd & H_RAW;
                if (n == n0) dma_part(C.vl, C.src_ws + (C.slot + O_LIN + (so >> 14) * 128u), (unsigned)(size_t)C.lines + line, 8u);
                static_assert(PF <= 2, "the header load must be older than load 6 - PF");
#pragma unroll
                for (int r = 0; r < 6; ++r) {
                    if (r < PF) { if (n == n0) wait_v<PF + 1>(); else wait_v<PF - 1>(); } else if (r > 3 && r <= 3 + PF) wait_v<PF>(); else wait_v<PF - 1>();
                    if (r == PF && pend && C.lane == 0) LDS_STORE(C.word, k);
                    dma(C.vl, sb + r * 1024, lo + r * 1024);
                    if (r == 3) asm volatile("global_load_dwordx2 %0, %1, %2 sc0 sc1" : "+v"(fv) : "v"(nw), "s"(C.ctl) : "memory");
                }
                wait_v<PF - 1>();
                asm volatile("" : "+v"(tv), "+v"(fv));
                dma_part(C.vl, sb + 6144, lo + 6144, in || raw ? 64u : 32u + more);
                if (raw || (in && more)) { wait_v<PF - 1>(); dma_part(C.vl, sb + 7168, lo + 7168, raw ? 64u : more); }
                pend = true; ++k;
            } else { ++k; CFENCE; if (C.lane == 0) LDS_STORE(C.word, k); }
        }
        const unsigned nn = nxt < NT ? (unsigned)RFL(tv) - tbase : NT;
        cur = nxt; nxt = nn; h0 = RFL((unsigned)fv); h1 = RFL((unsigned)(fv >> 32)); ++nl;
    }
    WAIT_V(0);
    while (k >= (unsigned)RFL(LDS_LOAD(C.word + 1)) + 4u) __builtin_amdgcn_s_sleep(1);
    if (C.lane == 0) { C.word[4 + 2 * (k & 3)] = ~0u; CFENCE; LDS_STORE(C.word, k + 1u); }
}
__device__ __forceinline__ void unpack_items(const Com& C) {
    int owed = 0;
    for (unsigned k = 0u;; ++k) {
        while ((unsigned)RFL(LDS_LOAD(C.word)) <= k) { SETTLE(); __builtin_amdgcn_s_sleep(1); }
        CFENCE;
        const unsigned it = RFL(C.word[4 + 2 * (k & 3)]), hd = RFL(C.word[5 + 2 * (k & 3)]), n = it & 0xffffu;
        if (n >= NI_IN) SETTLE();
        if (it == ~0u) break;
        LAS unsigned char* z = C.ring + (k & 3) * 8192;
        const LAS unsigned char* head = C.lines + ((it >> 16) & 510u);
        if (it & IT_DEAD) {
            const char* ob = unit_rows(C, n); const unsigned qn = 0x7fc00000u;
#pragma unroll
            for (int q = 0; q < 8; ++q) st4(C.vo, (u4){qn, qn, qn, qn}, ob + (size_t)((q >> 2) * 16 + (q & 3)) * (NG * 4));
        } else if (n < NI_IN) put_quarter(C, n, hd, z, head, owed);
        else put_unit(C, n, hd, z, head);
        WAIT_L(0);
        if (C.lane == 0) LDS_STORE(C.word + 1, k + 1u);
    }
}

__global__ __launch_bounds__(NTHR, 2) void gemm_rs_kernel(const Params P) {
    __shared__ __attribute__((aligned(16))) unsigned char smem[LDS_BYTES];
    LAS unsigned char* lds = (LAS unsigned char*)smem;
    LAS unsigned* lctl = (LAS unsigned*)(lds + LDS_CTL);
    const int tid = threadIdx.x, lane = tid & 63, wid = RFL(tid >> 6);
    const int bid = blockIdx.x;
    int rank = P.rank; char* myws = P.myws; char* peerws = P.peer_ws[(bid >> 3) & 7]; const char* xb = (const char*)P.x; const char* wb = (const char*)P.w;
    asm volatile("" : "+s"(myws), "+s"(peerws), "+s"(xb), "+s"(wb), "+s"(rank));
    gu32* ctl = (gu32*)myws;
    const unsigned vl = (unsigned)lane * 16u;
    const int gid = bid - NCOM;
    const unsigned vxe = (unsigned)((lane >> 3) * (KS * 4) + (((lane & 7) ^ (lane >> 4)) * 32));
    gu32* callp = ctl + W_CALL + bid * 32;
#define TAKE_CALL() CFENCE; const unsigned call = RFL(__hip_atomic_load(callp, __ATOMIC_RELAXED, __HIP_MEMORY_SCOPE_AGENT)) + 1u; \
    if (tid < 64) lctl[tid] = 0u; \
    asm volatile("s_waitcnt lgkmcnt(0)" :: "s"(call) : "memory");        \
    BAR; CFENCE; \
    if (tid == 0) __hip_atomic_store(callp, call, __ATOMIC_RELAXED, __HIP_MEMORY_SCOPE_AGENT); \
    const unsigned slot = CTL_B + (call & 1u) * SLOT_B

    if (__builtin_expect(gid < 0, 1)) {
        const int peer = (rank + 1 + (bid >> 3)) & 7, q0 = (bid & 7) * 4 + (wid & 3);
        const int qp = wid >= 4 ? q0 : 32 + (bid & 7) * PUSHX + wid, panel0 = (2 * peer + 2 + (qp >> 2)) & 15;
        const bool push = wid >= 4 || wid < PUSHX;
        f4 L0[8], L1[8];
        if (push) { job_loads(L0, xb, wb, true, 0, panel0, 0, 2 * (qp & 3), vxe, vl); job_loads(L1, xb, wb, true, 0, panel0, 0, 2 * (qp & 3) + 1, vxe, vl); }
        TAKE_CALL();
        Com C;
        C.ctl = ctl; C.me_ws = myws; C.rank = rank; C.src = peer; C.src_ws = peerws; C.out = (char*)P.out;
        C.ring = lds + (wid & 3) * 32768; C.lines = lds + LDS_LIN + (wid & 3) * 512; C.word = lctl + (wid & 3) * 16; C.slot = slot; C.call = call; C.vl = vl; C.lane = lane;
        C.vo = (unsigned)((lane >> 4) * 4 * (NG * 4) + (lane & 15) * 16);
        if (push) {
            u4 D[8];
            job_pack(L0, *(u4 (*)[4])&D[0], true); job_pack(L1, *(u4 (*)[4])&D[4], true);
            const char* db = C.src_ws + (slot + O_A + (unsigned)((panel0 * NKT + 8 + ((rank - peer - 1) & 7)) * TILE_B + (qp & 3) * 8192));
#pragma unroll
            for (int p = 0; p < 8; ++p) { if (p >= 2) WAIT_V(1); st4(vl, D[p], db + p * 1024); }
            WAIT_V(0);
            if (lane == 0) __hip_atomic_fetch_add((gu32*)C.src_ws + W_LAND + (panel0 * 8 + rank) * 32, 1u, __ATOMIC_RELAXED, __HIP_MEMORY_SCOPE_SYSTEM);
        }
        C.late = late_order((rank - peer - 1) & 7);
        if (wid >= 4) {
            unpack_items(C);
        } else {
            constexpr unsigned F0 = 16 + 4 * PUSHX;
            const unsigned tbase = (call - 1u) * (NT - F0 - 64 + 32) - (F0 + 64);
            load_items(C, ctl + W_TICK + (bid >> 3) * 32, tbase, F0 + q0, F0 + 32u + q0);
        }
        return;
    }

    f4 L0[8], L1[8];
    const int qlate = 1600 + (gid - 168) * 8 + wid;
    const int q0 = gid < 196 ? gid * 8 + wid : qlate;
    {
        bool isA; int dest, panel, j; conv_of(q0 >> 2, rank, isA, dest, panel, j);
        for (int i = gid < 50 ? gid >> 3 : 10 + ((gid - 48) >> 3) * 3; i > 0; --i) __builtin_amdgcn_s_sleep(8);
        CFENCE;
        job_loads(L0, xb, wb, isA, dest, panel, j, 2 * (q0 & 3), vxe, vl); job_loads(L1, xb, wb, isA, dest, panel, j, 2 * (q0 & 3) + 1, vxe, vl);
    }
    TAKE_CALL();
    gu32* tmo = ctl + W_TMO;
    {
        gu32* wsl = (gu32*)P.ws[lane & 7];
        asm volatile("" : "+v"(wsl));
        LAS unsigned char* z = lds + wid * 16384;
        for (int qn = q0, it = 0; qn < 4640; ++it) {
            bool isA; int dest, panel, j; conv_of(qn >> 2, rank, isA, dest, panel, j);
            const int qx = gid < 168 ? 4640 : qn < 1600 ? qlate : qn + 256;
            const int q = qn & 3;
            u4 D[8];
            job_pack(L0, *(u4 (*)[4])&D[0], isA); job_pack(L1, *(u4 (*)[4])&D[4], isA);
            const bool own = isA || dest == rank, peers = isA || dest != rank;
            const char* db = myws + (slot + (isA ? O_A : O_B) + (unsigned)((panel * NKT + j) * TILE_B));
            if (own) {
#pragma unroll
                for (int p = 0; p < 8; ++p) st4(vl, D[p], db + piece_off(isA, q, p));
            }
            unsigned hd = call << 19;
            if (peers) {
                const unsigned xo = (unsigned)((((isA ? rank : dest) * 16 + panel) * 8 + j) * TILE_B) + q * 8192;
                const char* xp = myws + (slot + O_X + xo);
                u4 lo[4], h[4]; unsigned se = 0u;
#pragma unroll
                for (int i = 0; i < 16; ++i) {
                    const unsigned d0 = D[i >> 1][2 * (i & 1)], d1 = D[i >> 1][2 * (i & 1) + 1];
                    lo[i >> 2][i & 3] = PERM(d1, d0, 0x06040200u); h[i >> 2][i & 3] = PERM(d1, d0, 0x07050301u);
                    se = __builtin_amdgcn_sad_u8(h[i >> 2][i & 3] & 0x7c7c7c7cu, 0u, se);
                }
#pragma unroll
                for (int r = 0; r < 4; ++r) *(LAS u4*)(z + r * 1024 + lane * 16) = h[r];
                *(LAS u4*)(z + 4096 + lane * 16) = (u4){0u, 0u, 0u, 0u};
                const unsigned base = min(max((wave_sum(se) + 47500u) >> 14, 6u), 31u);
                const unsigned bq = ((base << 2) | 0x80u) * 0x01010101u;
                u4 f[4]; unsigned m0 = 0u, m1 = 0u;
#pragma unroll
                for (int i = 0; i < 16; ++i) {
                    const unsigned hh = h[i >> 2][i & 3], x = (bq - (hh & 0x7c7c7c7cu)) ^ 0x80808080u;
                    const unsigned es = (((x & 0x7f7f7f7fu) + 0x64646464u) | x) & 0x80808080u;
                    f[i >> 2][i & 3] = ((hh >> 2) & 0x20202020u) | ((x | bytemask(es)) & 0x1c1c1c1cu) | (hh & 0x03030303u);
                    if (i < 8) m0 |= es >> (7 - i); else m1 |= es >> (15 - i);
                }
                CFENCE;
                u64 m = ((u64)m1 << 32) | m0; unsigned cnt = 0u; bool far = __popcll(m) > 16;
                if (!__any(far)) {
                    const unsigned first = lanes_below((unsigned)__popcll(m), cnt);
                    u64 nb = 0ull; unsigned hb = z[h_byte(__builtin_ctzll(m | (1ull << 63)), lane)];
                    for (unsigned sh = 0u; __any(m != 0ull); sh += 4u) {
                        const unsigned d = base + 2u - ((hb >> 2) & 31u);
                        if (m != 0ull) { far |= d > 22u; nb |= (u64)((d < 2u ? d : d - 7u) & 15u) << sh; }
                        m &= m - 1ull;
                        hb = z[h_byte(__builtin_ctzll(m | (1ull << 63)), lane)];
                    }
                    list_put((LAS unsigned*)(z + 4096), first, nb);
                }
                CFENCE;
                if (cnt <= 640u && !__any(far)) {
                    hd |= (((cnt + 127u) >> 8) << 16) | (base << 8);
#pragma unroll
                    for (int r = 0; r < 4; ++r) st4(vl, lo[r], xp + r * 1024);
#pragma unroll
                    for (int r = 0; r < 3; ++r) st4(vl, f[r] | ((f[3] << (6 - 2 * r)) & 0xc0c0c0c0u), xp + 4096 + r * 1024);
                    if (lane < 16) st1((unsigned)lane * 4u, *(const LAS unsigned*)(z + 4096 + lane * 4), myws + (slot + O_LIN + (xo >> 13) * 64u));
                    if (cnt > 128u) st1((unsigned)lane * 4u, *(const LAS unsigned*)(z + 4160 + lane * 4), xp + 7168);
                } else {
                    hd |= H_RAW;
#pragma unroll
                    for (int p = 0; p < 8; ++p) st4(vl, D[p], xp + p * 1024);
                }
            }
            if (qx < 4640) {
                bool nA; int nd, np, nj; conv_of(qx >> 2, rank, nA, nd, np, nj);
                job_loads(L0, xb, wb, nA, nd, np, nj, 2 * (qx & 3), vxe, vl); job_loads(L1, xb, wb, nA, nd, np, nj, 2 * (qx & 3) + 1, vxe, vl);
                wait_v<16>();
            } else WAIT_V(0);
            const int im = it * 2 + (wid >> 2);
            unsigned old = 0u;
            if (lane == 0) { lctl[L_HD + im * 4 + q] = hd; old = __hip_atomic_fetch_add(lctl + L_CNT + im, 1u, __ATOMIC_RELAXED, __HIP_MEMORY_SCOPE_WORKGROUP); }
            if ((unsigned)RFL(old) == 3u && lane < NR && (isA || lane == dest)) {
                CFENCE;
                if (lane == rank) __hip_atomic_fetch_add(ctl + W_LAND + (((isA ? 0 : 16) + panel) * 8 + rank) * 32 + j, 4u, __ATOMIC_RELAXED, __HIP_MEMORY_SCOPE_AGENT);
                else { const u4 h4 = *(const LAS u4*)(lctl + L_HD + im * 4);
                       asm volatile("global_store_dwordx4 %0, %1, off sc0 sc1\n\ts_nop 1" :: "v"(wsl + W_AVAIL + ((((call & 1) * 8 + rank) * 2 + (isA ? 0 : 1)) * 16 + panel) * 32 + j * 4), "v"(h4) : "memory"); }
            }
            qn = qx;
        }
        WAIT_V(0); WAIT_L(0);
    }

    const int wr = wid >> 2, wc = wid & 3;
    const int xcd = gid & 7, idx = gid >> 3, xh = xcd & 1, xq = xcd >> 1;
    for (int round = 0; round < 2; ++round) {
        int pm, pn;
        if (round == 0 && idx < 21) { const int b = idx / 7; pn = 3 * xq + b; pm = 2 * ((rank + 1 + idx - b * 7) & 7) + xh; }
        else if (round == 0 || idx < 3) { pn = 12 + xq; pm = 2 * ((rank + 1 + late_peer(xcd, round == 0 ? idx - 21 : idx + 4)) & 7) + xh; }
        else if (idx < 7) { pm = 2 * rank + xh; pn = 4 * xq + idx - 3; }
        else break;
        pm = RFL(pm); pn = RFL(pn);
        int tz = threadIdx.x; asm volatile("" : "+v"(tz));
        const int ln = tz & 63, fr = ln & 15, fq = ln >> 4;
        gu32* fa = ctl + W_LAND + (pm * 8 + (ln & 7)) * 32 + (ln >> 3);
        gu32* fb = ctl + W_LAND + ((16 + pn) * 8 + (ln & 7)) * 32 + (ln >> 3);
        const u64 own = 0x0101010101010101ull << rank;
        u64 ra = 0, rb = 0;
        bool bad = false;
#define STEP_WAIT(need, last) do { if (!bad && ((last) || (ra & rb & (need)) != (need))) bad = wave_wait(fa, fb, (need), ra, rb, call * 4u, tmo, call, ln); } while (0)
        if (wid == 0) { STEP_WAIT(own, false); WAIT_V(0); }
        CFENCE; BAR; CFENCE;

        const char* cA = myws + (slot + O_A + (unsigned)(pm * NKT) * TILE_B);
        const char* cB = myws + (slot + O_B + (unsigned)(pn * NKT) * TILE_B);
        const unsigned vst = (unsigned)tz * 16u;
        const unsigned ldsw = (unsigned)wid * 1024u;
        const int a0 = (wr * 64 + fr) * 128 + ((fq ^ (fr >> 1)) * 16), a1 = a0 ^ 64;
        const int b0 = fq * 2048 + (wc * 16 + fr) * 16;
#define SA(b, h) (((b) * 2 + (h)) * HALF_B)
#define SB(b, h) ((4 + (b) * 2 + (h)) * HALF_B)
#define STAGE(bufoff, gbase) do { _Pragma("unroll") for (int _i = 0; _i < 2; ++_i) \
        __builtin_amdgcn_global_load_lds((const unsigned*)((gbase) + vst + _i * 8192), (LAS unsigned*)(lds + (bufoff) + ldsw + _i * 8192), 16, 0, 17); } while (0)
#define LDA(dst, b, h) do { _Pragma("unroll") for (int m = 0; m < 4; ++m) { dst[m][0] = *(const LAS h8*)(lds + SA(b, h) + a0 + m * 2048); dst[m][1] = *(const LAS h8*)(lds + SA(b, h) + a1 + m * 2048); } } while (0)
#define LDB(dst, b, h) do { _Pragma("unroll") for (int n = 0; n < 2; ++n) _Pragma("unroll") for (int k = 0; k < 2; ++k) dst[n][k] = *(const LAS h8*)(lds + SB(b, h) + b0 + k * 8192 + n * 1024); } while (0)
#define MMA(ai, bj, At, Bt) do { __builtin_amdgcn_s_setprio(1); _Pragma("unroll") for (int m = 0; m < 4; ++m) _Pragma("unroll") for (int n = 0; n < 2; ++n) _Pragma("unroll") for (int k = 0; k < 2; ++k) \
        acc[ai][bj][m][n] = __builtin_amdgcn_mfma_f32_16x16x32_f16(At[m][k], Bt[n][k], acc[ai][bj][m][n], 0, 0, 0); __builtin_amdgcn_s_setprio(0); } while (0)
        f4 acc[2][2][4][2];
#pragma unroll
        for (int a = 0; a < 2; ++a)
#pragma unroll
            for (int b = 0; b < 2; ++b)
#pragma unroll
                for (int m = 0; m < 4; ++m)
#pragma unroll
                    for (int n = 0; n < 2; ++n) acc[a][b][m][n] = (f4){0.f, 0.f, 0.f, 0.f};
        h8 At[4][2], B0[2][2], B1[2][2];
        STAGE(SB(0, 0), cB); STAGE(SA(0, 0), cA); STAGE(SB(0, 1), cB + HALF_B); STAGE(SA(0, 1), cA + HALF_B);
        if (wr == 1) BAR;
        WAIT_V(4); BAR;
        STAGE(SB(1, 0), cB + TILE_B); STAGE(SA(1, 0), cA + TILE_B); STAGE(SB(1, 1), cB + TILE_B + HALF_B);
        WAIT_V(6); BAR;
        int jn = 0, bnd = 8;
        for (int t = 0; t < NKT; t += 2) {
            const int t2 = (t + 2) & (NKT - 1);
            const char* a1p = cA + (size_t)(t + 1) * TILE_B;
            const char* a2 = cA + (size_t)t2 * TILE_B; const char* b2 = cB + (size_t)t2 * TILE_B;
            const char* a3 = a2 + TILE_B; const char* b3 = b2 + TILE_B;
            if (t + 3 >= bnd && jn < 8) {
                if (wid == 0) {
                    STEP_WAIT((0xFFull << (8 * jn)) & ~own, jn == 7);
                    if (jn == 7 && ln == 0) lctl[L_BAD] = bad ? 1u : 0u;
                }
                CFENCE; BAR; CFENCE;
                ++jn; bnd += 7;
            }
            LDB(B0, 0, 0); SCHED; LDA(At, 0, 0); STAGE(SA(1, 1), a1p + HALF_B);
            WAIT_L(8); BAR; WAIT_L(0); MMA(0, 0, At, B0); BAR; SCHED;
            LDB(B1, 0, 1); STAGE(SB(0, 0), b2);
            BAR; WAIT_L(0); MMA(0, 1, At, B1); BAR;
            LDA(At, 0, 1); STAGE(SA(0, 0), a2);
            BAR; WAIT_L(0); MMA(1, 0, At, B0); BAR; SCHED;
            STAGE(SB(0, 1), b2 + HALF_B);
            WAIT_V(6); BAR; MMA(1, 1, At, B1); BAR;
            LDB(B0, 1, 0); SCHED; LDA(At, 1, 0); STAGE(SA(0, 1), a2 + HALF_B);
            WAIT_L(8); BAR; WAIT_L(0); MMA(0, 0, At, B0); BAR; SCHED;
            LDB(B1, 1, 1); STAGE(SB(1, 0), b3);
            BAR; WAIT_L(0); MMA(0, 1, At, B1); BAR;
            LDA(At, 1, 1); STAGE(SA(1, 0), a3);
            BAR; WAIT_L(0); MMA(1, 0, At, B0); BAR; SCHED;
            STAGE(SB(1, 1), b3 + HALF_B);
            WAIT_V(6); BAR; MMA(1, 1, At, B1); BAR;
        }
        WAIT_V(0);
        if (wr == 0) BAR;
        CFENCE; BAR; CFENCE;

        const unsigned dest = ((unsigned)pm >> 1) & 7u;
        int ty = threadIdx.x; asm volatile("" : "+v"(ty));
        const int l2 = ty & 63;
        const unsigned vo = (unsigned)((l2 >> 4) * 4 * (NG * 4) + (l2 & 15) * 16);
        const bool poison = lctl[L_BAD] != 0u;
        const float qnan = __builtin_nanf("");
#define ROW4(ai, m, r) (poison ? (f4){qnan, qnan, qnan, qnan} : (f4){gelu(acc[ai][0][m][0][r]), gelu(acc[ai][0][m][1][r]), gelu(acc[ai][1][m][0][r]), gelu(acc[ai][1][m][1][r])})
        if (dest == (unsigned)rank) {
            const char* ob = (const char*)P.out + ((size_t)((pm & 1) * 256 + wr * 64) * NG + rank * NS + pn * 256 + wc * 64) * 4;
#pragma unroll
            for (int ai = 0; ai < 2; ++ai)
#pragma unroll
                for (int m = 0; m < 4; ++m)
#pragma unroll
                    for (int r = 0; r < 4; ++r) { const f4 v = ROW4(ai, m, r); st4(vo, __builtin_bit_cast(u4, v), ob + (size_t)(ai * 128 + m * 16 + r) * (NG * 4)); }
        } else {
            const int tile = (pm & 1) * 16 + pn;
            const unsigned zo = (O_OUT - O_X) + (unsigned)((dest * 32 + tile) * 32 + wid) * UNIT_B;
            gu32* uf = (gu32*)P.ws[dest] + W_UNIT + (rank * 32 + tile) * 32;
            LAS unsigned char* z = lds + wid * 16384;
            unsigned hprev = 0u;
#define ANNOUNCE(gg) do { const int ig = round * 4 + (gg); unsigned old = 0u; \
                if (l2 == 0) { lctl[L_HD + 104 + ig * 8 + wid] = hprev; old = __hip_atomic_fetch_add(lctl + L_CNT + 26 + ig, 1u, __ATOMIC_RELAXED, __HIP_MEMORY_SCOPE_WORKGROUP); } \
                if ((unsigned)RFL(old) == 7u && l2 < 8) { CFENCE; __hip_atomic_store(uf + (gg) * 8 + l2, lctl[L_HD + 104 + ig * 8 + l2], __ATOMIC_RELAXED, __HIP_MEMORY_SCOPE_SYSTEM); } } while (0)
#pragma unroll
            for (int g = 0; g < 4; ++g) {
                const int ai = g >> 1;
                const char* zp = myws + (slot + O_X + zo + g * 8 * UNIT_B);
                unsigned sm[8], E[8], mn = 0u, se = 0u, np = 0u;
#pragma unroll
                for (int i = 0; i < 4; ++i) {
                    u4 lo;
#pragma unroll
                    for (int e = 0; e < 2; ++e) {
                        const int q = 2 * i + e, m = (g & 1) * 2 + (q >> 2), r = q & 3;
                        const u4 u = __builtin_bit_cast(u4, ROW4(ai, m, r));
                        lo[2 * e] = PERM(u.y, u.x, 0x05040100u); lo[2 * e + 1] = PERM(u.w, u.z, 0x05040100u);
                        const unsigned h01 = PERM(u.y, u.x, 0x07060302u), h23 = PERM(u.w, u.z, 0x07060302u), sg = PERM(h23, h01, 0x07050301u) & 0x80808080u;
                        sm[q] = (PERM(h23, h01, 0x06040200u) & 0x7f7f7f7fu) | sg;
                        E[q] = PERM(h23 << 1, h01 << 1, 0x07050301u);
                        mn = pmax_u16(mn, pmax_u16(h01, h23));
                        se = __builtin_amdgcn_sad_u8(E[q] & ~bytemask(sg), 0u, se); np += (unsigned)__popc(sg ^ 0x80808080u);
                    }
                    *(LAS u4*)(z + 8192 + i * 1024 + l2 * 16) = lo;
                    st4((unsigned)l2 * 16u, lo, zp + i * 1024);
                }
                if (g >= 2) { WAIT_V(4); ANNOUNCE(g - 1); }
                *(LAS u4*)(z + l2 * 16) = (u4){0u, 0u, 0u, 0u};
                *(LAS u4*)(z + 2048 + l2 * 16) = (u4){E[0], E[1], E[2], E[3]}; *(LAS u4*)(z + 3072 + l2 * 16) = (u4){E[4], E[5], E[6], E[7]};
                *(LAS u4*)(z + 4096 + l2 * 16) = (u4){sm[0], sm[1], sm[2], sm[3]}; *(LAS u4*)(z + 5120 + l2 * 16) = (u4){sm[4], sm[5], sm[6], sm[7]};
                const unsigned nm = wave_max(max(mn & 0xffffu, mn >> 16)), bn = (nm & 0x8000u) ? (nm >> 7) & 0xffu : 0u, sp = wave_sum((se << 12) | np);
                const unsigned bp = RFL(min((unsigned)((float)(sp >> 12) * __builtin_amdgcn_rcpf((float)max(sp & 0xfffu, 1u)) + 2.4f), 255u));
                const unsigned bp4 = bp * 0x01010101u, bn4 = bn * 0x01010101u;
                unsigned pc[2] = {0u, 0u}, sh = 0u, cnt, cnt2 = 0u, m15 = 0u, hd = call << 19; u64 nb = 0ull;
                CFENCE;
#pragma unroll
                for (int q = 0; q < 8; ++q) {
                    const unsigned sg = sm[q] & 0x80808080u, mk = bytemask(sg), bv = (bn4 & mk) | (bp4 & ~mk);
                    const unsigned d = ((bv | 0x80808080u) - (E[q] & 0x7f7f7f7fu)) ^ ((bv ^ ~E[q]) & 0x80808080u), y = d & 0xfcfcfcfcu;
                    const unsigned es = ((((y & 0x7f7f7f7fu) + 0x7f7f7f7fu) | y) & 0x80808080u) | ((d & (d >> 1)) << 7 & sg), km = bytemask(es);
                    pc[q >> 2] |= ((d | km) & 0x03030303u) << (2 * (q & 3));
                    sm[q] |= es;
                    const unsigned up = bytemask(d & 0x80808080u), k = (mk & 0x06060606u) | (~mk & ((up & 0x02020202u) | (~up & 0xfefefefeu)));
                    const unsigned t = ((d & 0x7f7f7f7fu) + (k & 0x7f7f7f7fu)) ^ ((d ^ k) & 0x80808080u);
                    const unsigned ext = (((t & 0x7f7f7f7fu) + ((mk & 0x71717171u) | (~mk & 0x77777777u))) | t | (d & sg)) & es;
                    const unsigned v = (t | bytemask(ext)) & km & 0x0f0f0f0fu, p4 = (es >> 7) * 0x04040400u;
                    const unsigned w = (v & 15u) | (((v >> 8) & 15u) << ((p4 >> 8) & 15u)) | (((v >> 16) & 15u) << ((p4 >> 16) & 15u)) | ((v >> 24) << (p4 >> 24));
                    nb |= (u64)w << min(sh, 63u); sh += 4u * (unsigned)__popc(es);
                    m15 |= ext >> (7 - q);
                }
                for (;;) {
                    const u64 act = __ballot(m15 != 0u);
                    if (act == 0ull || cnt2 > 8u) break;
                    if (m15 != 0u) {
                        const unsigned pos = cnt2 + lane_rank(act); const int o = h_byte(__builtin_ctz(m15), l2);
                        if (pos < 8u) *(LAS unsigned short*)(z + pos * 2u) = (unsigned short)(z[2048 + o] | ((z[4096 + o] >> 7) << 8));
                        m15 &= m15 - 1u;
                    }
                    cnt2 += (unsigned)__popcll(act);
                }
                bool packed = cnt2 <= 8u && !__any(sh > 64u);
                if (packed) {
                    const unsigned first = lanes_below(sh >> 2, cnt);
                    list_put((LAS unsigned*)(z + 16), first, nb);
                    packed = cnt <= 608u;
                }
                CFENCE;
                if (packed) {
                    hd |= (((cnt + 159u) >> 8) << 16) | (bp << 8) | bn;
                    st4((unsigned)l2 * 16u, (u4){sm[0], sm[1], sm[2], sm[3]}, zp + 4096); st4((unsigned)l2 * 16u, (u4){sm[4], sm[5], sm[6], sm[7]}, zp + 5120);
                    st1((unsigned)l2 * 4u, pc[0], zp + 6144); st1((unsigned)l2 * 4u, pc[1], zp + 6400);
                    if (l2 < 16) st1((unsigned)l2 * 4u, *(const LAS unsigned*)(z + l2 * 4), myws + (slot + O_LIN + ((zo >> 13) + g * 8) * 64u));
                    if (cnt > 96u) st1((unsigned)l2 * 4u, *(const LAS unsigned*)(z + 64 + l2 * 4), zp + 6656);
                } else {
                    hd |= H_RAW;
                    WAIT_V(0);
#pragma unroll
                    for (int i = 0; i < 4; ++i) {
                        const u4 lo = *(const LAS u4*)(z + 8192 + i * 1024 + l2 * 16);
#pragma unroll
                        for (int e = 0; e < 2; ++e) {
                            const int q = 2 * i + e, o = (q >> 2) * 1024 + l2 * 16 + (q & 3) * 4;
                            const unsigned ee = *(const LAS unsigned*)(z + 2048 + o), ss = *(const LAS unsigned*)(z + 4096 + o);
                            const unsigned tp = (ss & 0x80808080u) | ((ee >> 1) & 0x7f7f7f7fu), md = (ss & 0x7f7f7f7fu) | ((ee << 7) & 0x80808080u);
                            const unsigned h01 = PERM(tp, md, 0x05010400u), h23 = PERM(tp, md, 0x07030602u);
                            st4((unsigned)l2 * 16u, (u4){PERM(h01, lo[2 * e], 0x05040100u), PERM(h01, lo[2 * e], 0x07060302u), PERM(h23, lo[2 * e + 1], 0x05040100u), PERM(h23, lo[2 * e + 1], 0x07060302u)},
                                zp + q * 1024);
                        }
                    }
                }
                hprev = hd;
                if (g == 0 || g == 3) { WAIT_V(0); ANNOUNCE(g); }
            }
        }
        WAIT_V(0); WAIT_L(0); BAR; CFENCE;
    }
}

}

extern "C" void kernel_launch(int rank, int nranks, ncclComm_t comm, hipStream_t stream,
                              void* const* d_in, const int* in_sizes, int n_in,
                              void* d_out, int out_size, const int* in_dtypes, int out_dtype,
                              void* const* d_out_all, void* const* d_ws_all, size_t ws_bytes) {
    (void)comm; (void)in_dtypes; (void)out_dtype; (void)d_out_all;
    if (nranks != NR || (unsigned)rank >= (unsigned)NR || n_in < 2 || ws_bytes < WS_NEED || in_sizes[0] != MG * KS || in_sizes[1] != KS * NG || out_size != MS * NG) {
        fprintf(stderr, "kernel_launch: unexpected problem shape\n");
        return;
    }
    Params P;
    P.x = (const float*)d_in[0]; P.w = (const float*)d_in[1];
    for (int p = 0; p < NR; ++p) P.ws[p] = (char*)d_ws_all[p];
    for (int p = 0; p < NR; ++p) P.peer_ws[p] = P.ws[(rank + 1 + p) & 7];
    P.myws = P.ws[rank]; P.out = (float*)d_out;
    P.rank = rank; P.pad = 0;
    hipLaunchKernelGGL(gemm_rs_kernel, dim3(NWG), dim3(NTHR), 0, stream, P);
    HIP_CHECK(hipGetLastError());
}
```
